```python
import numpy as np

kernel_name = "gpu_ag_gemm_m4096_k4096_n4096_f32_qe4m3"


ABSMAX_RTOL = 0.09

COMPUTE_DTYPE = "float32"
EPILOGUE = "quant_e4m3"
IN_QUANT = None
OUT_QUANT = "e4m3"
IN_DTYPES = {"x": "float32", "w_mat": "float32"}
WORLD_SIZE = 8

def setup_inputs():

    x = np.random.randn(4096, 4096).astype(np.float32)
    w_mat = (np.random.randn(4096, 4096) / np.sqrt(4096)).astype(np.float32)
    return {"x": x, "w_mat": w_mat}

def _snap_fp8(x, mant_bits, max_abs, min_normal):
    a = np.abs(x.astype(np.float64))
    _m, _e = np.frexp(a)
    q = np.where(a >= min_normal, np.ldexp(1.0, _e - 1 - mant_bits),
                 min_normal * 2.0 ** -mant_bits)
    snapped = np.minimum(np.round(a / q) * q, max_abs)
    return (np.sign(x) * snapped).astype(np.float32)
def _quant_dequant(y, amax):
    scale = amax / 448.0
    q = _snap_fp8(y / scale, 3, 448.0, 2.0 ** -6)
    return (q * scale).astype(np.float32)
def _epilogue(y):
    return _quant_dequant(y, np.abs(y).max())

def reference(x, w_mat):
    y = (x @ w_mat).astype(np.float32)
    return _epilogue(y)
```

```cpp
#include <hip/hip_runtime.h>
#include <rccl/rccl.h>
#include <cstdio>
#include <cstdint>

#define HIP_CHECK(expr) do { hipError_t _e = (expr); if (_e != hipSuccess) { \
    fprintf(stderr, "HIP error %s at %s:%d\n", hipGetErrorString(_e), __FILE__, __LINE__); \
    return; } } while (0)

typedef unsigned u32x4 __attribute__((ext_vector_type(4)));
typedef unsigned u32x2 __attribute__((ext_vector_type(2)));
typedef float f32x4 __attribute__((ext_vector_type(4)));
typedef short bf16x8 __attribute__((ext_vector_type(8)));

constexpr int NR = 8, M = 4096, K = 4096, NL = 512, ML = 512;
constexpr unsigned long long DEADLINE = 40000000ull;
constexpr size_t OFF_CTL = 0;
constexpr size_t OFF_XG = 1 << 20;
constexpr size_t OFF_WT = OFF_XG + (size_t)M * K * 2;
constexpr size_t OFF_Y = OFF_WT + (size_t)NL * K * 2;
constexpr size_t WS_NEED = OFF_Y + (size_t)M * NL * 4;
constexpr int C_TICK0 = 0, C_TICK1 = 32, C_TICK2 = 64, C_TMO = 96, C_CNTX = 128, C_LMAX = 160, C_LCNT = 192, C_AMAX = 256;

struct Params { const float* x; const float* w; unsigned char* out; char* ws[NR]; int rank; int phase; };

__device__ __forceinline__ unsigned cvt_pk_bf16(float lo, float hi) { unsigned r; asm volatile("v_cvt_pk_bf16_f32 %0, %1, %2" : "=v"(r) : "v"(lo), "v"(hi)); return r; }
__device__ __forceinline__ void st16_sys(void* p, u32x4 v) { asm volatile("global_store_dwordx4 %0, %1, off sc0 sc1\n\ts_nop 1" :: "v"(p), "v"(v) : "memory"); }

__global__ void __launch_bounds__(256) k_main(Params p) {
    __shared__ __attribute__((aligned(16))) unsigned char lds[(128 + 64) * 72 * 2 + 64];
    unsigned* ctl = (unsigned*)(p.ws[p.rank] + OFF_CTL);
    const int tid = threadIdx.x, lane = tid & 63, wv = tid >> 6;
    unsigned* s_call = (unsigned*)(lds + (128 + 64) * 72 * 2);
    if (tid == 0) {
        unsigned t = __hip_atomic_fetch_add(ctl + (p.phase == 0 ? C_TICK0 : p.phase == 1 ? C_TICK1 : C_TICK2), 1u, __ATOMIC_RELAXED, __HIP_MEMORY_SCOPE_AGENT);
        s_call[0] = t / gridDim.x + 1; s_call[1] = 0;
    }
    __syncthreads();
    const unsigned call = s_call[0];
    if (p.phase == 0) {
        for (int i = blockIdx.x * 256 + tid; i < ML * K / 8; i += gridDim.x * 256) {
            const f32x4 a = *(const f32x4*)(p.x + (size_t)i * 8), b = *(const f32x4*)(p.x + (size_t)i * 8 + 4);
            u32x4 v = {cvt_pk_bf16(a[0], a[1]), cvt_pk_bf16(a[2], a[3]), cvt_pk_bf16(b[0], b[1]), cvt_pk_bf16(b[2], b[3])};
            for (int d = 0; d < NR; ++d) { const int q = (p.rank + d + blockIdx.x) & 7; st16_sys(p.ws[q] + OFF_XG + ((size_t)p.rank * ML * K + (size_t)i * 8) * 2, v); }
        }
        for (int i = blockIdx.x * 256 + tid; i < NL * K / 8; i += gridDim.x * 256) {
            const int n = i & 511, k8 = i >> 9; float v[8];
            for (int j = 0; j < 8; ++j) v[j] = p.w[(size_t)(k8 * 8 + j) * NL + n];
            u32x4 o = {cvt_pk_bf16(v[0], v[1]), cvt_pk_bf16(v[2], v[3]), cvt_pk_bf16(v[4], v[5]), cvt_pk_bf16(v[6], v[7])};
            *(u32x4*)(p.ws[p.rank] + OFF_WT + ((size_t)n * K + k8 * 8) * 2) = o;
        }
        asm volatile("s_waitcnt vmcnt(0)" ::: "memory");
        __syncthreads();
        if (tid < NR) __hip_atomic_fetch_add((unsigned*)(p.ws[tid] + OFF_CTL) + C_CNTX, 1u, __ATOMIC_RELAXED, __HIP_MEMORY_SCOPE_SYSTEM);
        return;
    }
    if (p.phase == 1) {
        if (tid == 0) {
            const unsigned want = call * NR * gridDim.x; unsigned long long t0 = __builtin_amdgcn_s_memrealtime();
            while (__hip_atomic_load(ctl + C_CNTX, __ATOMIC_RELAXED, __HIP_MEMORY_SCOPE_SYSTEM) < want) {
                if (__hip_atomic_load(ctl + C_TMO, __ATOMIC_RELAXED, __HIP_MEMORY_SCOPE_AGENT) == call || __builtin_amdgcn_s_memrealtime() - t0 > DEADLINE) { __hip_atomic_store(ctl + C_TMO, call, __ATOMIC_RELAXED, __HIP_MEMORY_SCOPE_AGENT); break; }
                __builtin_amdgcn_s_sleep(2);
            }
            __builtin_amdgcn_fence(__ATOMIC_ACQUIRE, "");
            asm volatile("s_waitcnt vmcnt(0)" ::: "memory");
        }
        __syncthreads();
        const int tm = blockIdx.x >> 3, tn = blockIdx.x & 7;
        const unsigned short* A = (const unsigned short*)(p.ws[p.rank] + OFF_XG) + (size_t)tm * 128 * K;
        const unsigned short* B = (const unsigned short*)(p.ws[p.rank] + OFF_WT) + (size_t)tn * 64 * K;
        unsigned short* sA = (unsigned short*)lds; unsigned short* sB = sA + 128 * 72;
        const int wr = wv >> 1, wc = wv & 1, fr = lane & 15, fq = lane >> 4;
        f32x4 acc[4][2] = {};
        for (int k0 = 0; k0 < K; k0 += 64) {
            u32x4 ra[4], rb[2];
            for (int j = 0; j < 4; ++j) { const int e = tid + j * 256, r = e >> 3, c = e & 7; ra[j] = *(const u32x4*)(A + (size_t)r * K + k0 + c * 8); }
            for (int j = 0; j < 2; ++j) { const int e = tid + j * 256, r = e >> 3, c = e & 7; rb[j] = *(const u32x4*)(B + (size_t)r * K + k0 + c * 8); }
            __syncthreads();
            for (int j = 0; j < 4; ++j) { const int e = tid + j * 256, r = e >> 3, c = e & 7; *(u32x4*)(sA + r * 72 + c * 8) = ra[j]; }
            for (int j = 0; j < 2; ++j) { const int e = tid + j * 256, r = e >> 3, c = e & 7; *(u32x4*)(sB + r * 72 + c * 8) = rb[j]; }
            __syncthreads();
            for (int kk = 0; kk < 2; ++kk) {
                bf16x8 af[4], bfr[2];
                for (int m = 0; m < 4; ++m) af[m] = *(const bf16x8*)(sA + (wr * 64 + m * 16 + fr) * 72 + kk * 32 + fq * 8);
                for (int n = 0; n < 2; ++n) bfr[n] = *(const bf16x8*)(sB + (wc * 32 + n * 16 + fr) * 72 + kk * 32 + fq * 8);
                for (int m = 0; m < 4; ++m) for (int n = 0; n < 2; ++n) acc[m][n] = __builtin_amdgcn_mfma_f32_16x16x32_bf16(bfr[n], af[m], acc[m][n], 0, 0, 0);
            }
        }
        float* Y = (float*)(p.ws[p.rank] + OFF_Y); float mx = 0.f;
        for (int m = 0; m < 4; ++m) for (int n = 0; n < 2; ++n) {
            const int row = tm * 128 + wr * 64 + m * 16 + fr, col = tn * 64 + wc * 32 + n * 16 + fq * 4;
            *(f32x4*)(Y + (size_t)row * NL + col) = acc[m][n];
            for (int j = 0; j < 4; ++j) mx = fmaxf(mx, fabsf(acc[m][n][j]));
        }
        for (int o = 32; o >= 1; o >>= 1) mx = fmaxf(mx, __shfl_xor(mx, o));
        if (lane == 0) __hip_atomic_fetch_max(s_call + 1, __float_as_uint(mx), __ATOMIC_RELAXED, __HIP_MEMORY_SCOPE_WORKGROUP);
        __syncthreads();
        if (tid == 0) {
            unsigned long long* lm = (unsigned long long*)(ctl + C_LMAX);
            __hip_atomic_fetch_max(lm, ((unsigned long long)call << 32) | s_call[1], __ATOMIC_RELAXED, __HIP_MEMORY_SCOPE_AGENT);
            asm volatile("s_waitcnt vmcnt(0)" ::: "memory");
            const unsigned t = __hip_atomic_fetch_add(ctl + C_LCNT, 1u, __ATOMIC_RELAXED, __HIP_MEMORY_SCOPE_AGENT);
            if ((t + 1) % gridDim.x == 0) {
                const unsigned long long v = __hip_atomic_load(lm, __ATOMIC_RELAXED, __HIP_MEMORY_SCOPE_AGENT);
                for (int q = 0; q < NR; ++q) __hip_atomic_store((unsigned long long*)((unsigned*)(p.ws[q] + OFF_CTL) + C_AMAX + 32 * p.rank), v, __ATOMIC_RELAXED, __HIP_MEMORY_SCOPE_SYSTEM);
            }
        }
        return;
    }
    if (tid == 0) {
        unsigned g = 0; bool bad = false; unsigned long long t0 = __builtin_amdgcn_s_memrealtime();
        for (int q = 0; q < NR && !bad; ++q) {
            unsigned long long v;
            while (((v = __hip_atomic_load((unsigned long long*)(ctl + C_AMAX + 32 * q), __ATOMIC_RELAXED, __HIP_MEMORY_SCOPE_SYSTEM)) >> 32) < call) {
                if (__hip_atomic_load(ctl + C_TMO, __ATOMIC_RELAXED, __HIP_MEMORY_SCOPE_AGENT) == call || __builtin_amdgcn_s_memrealtime() - t0 > DEADLINE) { __hip_atomic_store(ctl + C_TMO, call, __ATOMIC_RELAXED, __HIP_MEMORY_SCOPE_AGENT); bad = true; break; }
                __builtin_amdgcn_s_sleep(2);
            }
            g = max(g, (unsigned)v);
        }
        if (__hip_atomic_load(ctl + C_TMO, __ATOMIC_RELAXED, __HIP_MEMORY_SCOPE_AGENT) == call) bad = true;
        s_call[1] = g; s_call[2] = bad;
    }
    __syncthreads();
    const float amax = __uint_as_float(s_call[1]); const bool bad = s_call[2];
    const float scale = amax / 448.0f;
    const float* Y = (const float*)(p.ws[p.rank] + OFF_Y);
    for (int i = blockIdx.x * 256 + tid; i < M * NL / 8; i += gridDim.x * 256) {
        const f32x4 a = *(const f32x4*)(Y + (size_t)i * 8), b = *(const f32x4*)(Y + (size_t)i * 8 + 4);
        float v[8] = {a[0], a[1], a[2], a[3], b[0], b[1], b[2], b[3]};
        for (int j = 0; j < 8; ++j) v[j] = amax > 0.f ? fminf(fmaxf(v[j] / scale, -448.f), 448.f) : 0.f;
        int lo = __builtin_amdgcn_cvt_pk_fp8_f32(v[0], v[1], 0, false); lo = __builtin_amdgcn_cvt_pk_fp8_f32(v[2], v[3], lo, true);
        int hi = __builtin_amdgcn_cvt_pk_fp8_f32(v[4], v[5], 0, false); hi = __builtin_amdgcn_cvt_pk_fp8_f32(v[6], v[7], hi, true);
        u32x2 o = {(unsigned)lo, (unsigned)hi};
        if (bad) o = (u32x2){0x7f7f7f7fu, 0x7f7f7f7fu};
        *(u32x2*)(p.out + (size_t)i * 8) = o;
    }
}

extern "C" void kernel_launch(int rank, int nranks, ncclComm_t comm, hipStream_t stream,
                              void* const* d_in, const int* in_sizes, int n_in,
                              void* d_out, int out_size, const int* in_dtypes, int out_dtype,
                              void* const* d_out_all, void* const* d_ws_all, size_t ws_bytes) {
    if (nranks != NR || ws_bytes < WS_NEED || n_in < 2 || in_sizes[0] != ML * K || in_sizes[1] != K * NL || out_size != M * NL) { fprintf(stderr, "unexpected shapes\n"); return; }
    Params p; p.x = (const float*)d_in[0]; p.w = (const float*)d_in[1]; p.out = (unsigned char*)d_out;
    for (int i = 0; i < NR; ++i) p.ws[i] = (char*)d_ws_all[i];
    p.rank = rank;
    for (int ph = 0; ph < 3; ++ph) { p.phase = ph; k_main<<<256, 256, 0, stream>>>(p); }
    HIP_CHECK(hipGetLastError());
}
```

```cpp
#include <hip/hip_runtime.h>
#include <rccl/rccl.h>
#include <cstdio>
#include <cstdint>

#define HIP_CHECK(expr) do { hipError_t _e = (expr); if (_e != hipSuccess) { \
    fprintf(stderr, "HIP error %s at %s:%d\n", hipGetErrorString(_e), __FILE__, __LINE__); \
    return; } } while (0)

typedef unsigned u32;
typedef unsigned long long u64;
typedef u32 u32x4 __attribute__((ext_vector_type(4)));
typedef u32 u32x2 __attribute__((ext_vector_type(2)));
typedef float f32x4 __attribute__((ext_vector_type(4)));
typedef short bf16x8 __attribute__((ext_vector_type(8)));
#define LAS __attribute__((address_space(3)))
#define GAS __attribute__((address_space(1)))

constexpr int NR = 8, MG = 4096, KG = 4096, SH = 512;
constexpr int NT = 32;
constexpr int NTHR = 640;
constexpr u64 DEADLINE = 150000000ull;
constexpr u32 UNIT = 1536;
constexpr u32 OFF_CNT = 0;
constexpr u32 OFF_TMO = 0x8000;
constexpr u32 OFF_AMAX = 0x10000;
constexpr u32 OFF_STAMP = 0x20000;
constexpr u32 OFF_HA = 0x30000;
constexpr u32 OFF_HB = 0x40000;
constexpr u32 OFF_HY = 0x50000;
constexpr u32 OFF_IN = 2u << 20;
constexpr u32 IN_SRC = 2u << 20, IN_B = 768u << 10, IN_T = 16 * UNIT;
constexpr u32 OFF_Y = 18u << 20;
constexpr u32 Y_SRC = 1u << 20;
constexpr u32 OFF_OVF = 26u << 20;
constexpr size_t WS_NEED = 34u << 20;
constexpr int STAGE = 36864, LDS_B = 20480, LDS_MISC = 2 * STAGE, LDS_BYTES = 96 * 1024;

struct Params { const float* x; const float* w; unsigned char* out; char* ws[NR]; int rank; int pad; };

__device__ __forceinline__ u32 cvt_pk_bf16(float lo, float hi) { u32 r; asm("v_cvt_pk_bf16_f32 %0, %1, %2" : "=v"(r) : "v"(lo), "v"(hi)); return r; }
__device__ __forceinline__ void st16_sys(void* p, u32x4 v) { asm volatile("global_store_dwordx4 %0, %1, off sc0 sc1\n\ts_nop 1" :: "v"(p), "v"(v) : "memory"); }
__device__ __forceinline__ void st8_sys(void* p, u32x2 v) { asm volatile("global_store_dwordx2 %0, %1, off sc0 sc1\n\ts_nop 1" :: "v"(p), "v"(v) : "memory"); }
__device__ __forceinline__ u32x4 ld16_sync(const char* base, u32 off) { u32x4 r; asm volatile("global_load_dwordx4 %0, %1, %2 sc0 sc1\n\ts_waitcnt vmcnt(0)" : "=&v"(r) : "v"(off), "s"(base) : "memory"); return r; }
__device__ __forceinline__ u32 ld4_sync_agent(const void* p) { u32 r; asm volatile("global_load_dword %0, %1, off sc1\n\ts_waitcnt vmcnt(0)" : "=&v"(r) : "v"(p) : "memory"); return r; }
__device__ __forceinline__ void drain() { asm volatile("s_waitcnt vmcnt(0)" ::: "memory"); }
__device__ __forceinline__ u64 rtime() { return __builtin_amdgcn_s_memrealtime(); }
__device__ __forceinline__ u32 rfl(u32 v) { return __builtin_amdgcn_readfirstlane(v); }
__device__ __forceinline__ u32 rdl(u32 v, int l) { return __builtin_amdgcn_readlane(v, l); }
__device__ __forceinline__ u32 wave_max_u32(u32 v) {
    v = max(v, (u32)__builtin_amdgcn_update_dpp(0, (int)v, 0x111, 0xf, 0xf, false));
    v = max(v, (u32)__builtin_amdgcn_update_dpp(0, (int)v, 0x112, 0xf, 0xf, false));
    v = max(v, (u32)__builtin_amdgcn_update_dpp(0, (int)v, 0x114, 0xf, 0xf, false));
    v = max(v, (u32)__builtin_amdgcn_update_dpp(0, (int)v, 0x118, 0xf, 0xf, false));
    v = max(v, (u32)__builtin_amdgcn_update_dpp(0, (int)v, 0x142, 0xa, 0xf, false));
    v = max(v, (u32)__builtin_amdgcn_update_dpp(0, (int)v, 0x143, 0xc, 0xf, false));
    return rdl(v, 63);
}
#define BARRIER() do { asm volatile("s_waitcnt lgkmcnt(0)" ::: "memory"); __builtin_amdgcn_s_barrier(); asm volatile("" ::: "memory"); } while (0)

#define ROLE_OF(b) (((b) >> 3) < 4 ? 0 : ((b) >> 3) < 30 ? 1 : 2)
#define STAMP(ws, slot) do {} while (0)
#define STAMPD(ws, slot) do {} while (0)

__device__ __forceinline__ void pack_unit(const u32* d, float amax_lane, u32 seq16, u32x4& L, u32x2& N, u32x4& hdr) {
    u32 H[4];
#pragma unroll
    for (int q = 0; q < 4; ++q) { L[q] = __builtin_amdgcn_perm(d[2 * q + 1], d[2 * q], 0x06040200u); H[q] = __builtin_amdgcn_perm(d[2 * q + 1], d[2 * q], 0x07050301u); }
    const u32 mb = wave_max_u32(__float_as_uint(amax_lane));
    u32 h7 = (mb + 0x7FFFu + ((mb >> 16) & 1u)) >> 24;
    h7 = h7 > 127u ? 127u : h7;
    const u32 base = h7 > 7u ? h7 - 7u : 0u, brep = base * 0x01010101u;
    u32 b[4], X[4];
#pragma unroll
    for (int q = 0; q < 4; ++q) { const u32 t = (H[q] | 0x80808080u) - brep; b[q] = (t ^ 0x80808080u) & 0xF8F8F8F8u; X[q] = (H[q] & 0x80808080u) | (t & 0x07070707u); }
    N[0] = X[0] | __builtin_amdgcn_alignbit(X[1], X[1], 28); N[1] = X[2] | __builtin_amdgcn_alignbit(X[3], X[3], 28);
    u32 cnt = 0; u64 ea = 0, eb = 0;
    u64 m = __ballot((b[0] | b[1] | b[2] | b[3]) != 0);
    if (m) {
        u32 bm = 0;
#pragma unroll
        for (int q = 0; q < 4; ++q) bm |= (((((((b[q] & 0x7F7F7F7Fu) + 0x7F7F7F7Fu) | b[q]) & 0x80808080u) >> 7) * 0x01020408u) >> 24 & 0xFu) << (4 * q);
        while (m && cnt <= 4) {
            const int l = __builtin_ctzll(m); m &= m - 1;
            u32 mk = rdl(bm, l);
            const u32 h0 = rdl(H[0], l), h1 = rdl(H[1], l), h2 = rdl(H[2], l), h3 = rdl(H[3], l);
            while (mk) {
                const u32 j = (u32)__builtin_ctz(mk); mk &= mk - 1;
                const u32 hq = (j & 8u) ? ((j & 4u) ? h3 : h2) : ((j & 4u) ? h1 : h0);
                const u64 e = (u64)((u32)l * 16u + j) | ((u64)((hq >> (8u * (j & 3u))) & 0xFFu) << 11);
                if (cnt < 2) ea |= e << (26u + 19u * cnt); else if (cnt < 4) eb |= e << (16u + 19u * (cnt - 2u));
                ++cnt;
            }
        }
    }
    const u32 mode = cnt > 4 ? 7u : cnt;
    ea |= (u64)seq16 | ((u64)base << 16) | ((u64)mode << 23); eb |= (u64)seq16;
    hdr = (u32x4){(u32)ea, (u32)(ea >> 32), (u32)eb, (u32)(eb >> 32)};
}
__device__ __forceinline__ u32x4 decode_group(u32 LA, u32 LB, u32 n, u32 brep) {
    const u32 HA = (n & 0x87878787u) + brep, HB = (__builtin_amdgcn_alignbit(n, n, 4) & 0x87878787u) + brep;
    return (u32x4){__builtin_amdgcn_perm(HA, LA, 0x05010400u), __builtin_amdgcn_perm(HA, LA, 0x07030602u),
                   __builtin_amdgcn_perm(HB, LB, 0x05010400u), __builtin_amdgcn_perm(HB, LB, 0x07030602u)};
}
__device__ __forceinline__ u32 hdr_exc(u64 a, u64 b, u32 c) { return (u32)((c < 2 ? a >> (26u + 19u * c) : b >> (16u + 19u * (c - 2u)))) & 0x7FFFFu; }
__device__ __forceinline__ bool hdr_ok(const u32x4& h, u32 seq16) { return __ballot(((h.x & 0xFFFFu) == seq16) && ((h.z & 0xFFFFu) == seq16)) == ~0ull; }

struct Wait { GAS u32* tmo; u32 call; bool dead; };
__device__ __forceinline__ bool spin_on(Wait& wt, u64 t0, u32& n) {
    if (wt.dead) return true;
    if ((++n & 15u) == 0u) {
        if (rfl(ld4_sync_agent((const void*)wt.tmo)) == wt.call || rtime() - t0 > DEADLINE) {
            __hip_atomic_store(wt.tmo, wt.call, __ATOMIC_RELAXED, __HIP_MEMORY_SCOPE_AGENT); wt.dead = true; return true; }
    }
    __builtin_amdgcn_s_sleep(1);
    return false;
}

template <bool ISW>
__device__ __forceinline__ void pusher(const Params& p, int cu, int sub, int lane, u32 seq16) {
    const int q4 = ((cu >> 3) + sub) & 3, h = q4 >> 1, kh = q4 & 1, blk = cu & 7, fr = lane & 15, fq = lane >> 4;
    int d0 = ISW ? ((h << 1) | kh) : ((h << 2) | kh), d1 = d0 | (ISW ? 4 : 2);
    if ((cu ^ sub) & 1) { const int t = d0; d0 = d1; d1 = t; }
    char* const w0 = p.ws[d0 & 7]; char* const w1 = p.ws[d1 & 7];
    const GAS f32x4* src;
    if (ISW) src = (const GAS f32x4*)(p.w + (size_t)(2048 * kh + 32 * (blk & 1) + 8 * fq + 64 * sub) * SH + 256 * h + 64 * (blk >> 1) + 4 * fr);
    else     src = (const GAS f32x4*)(p.x + (size_t)(256 * h + 32 * blk + fr) * KG + 2048 * kh + 8 * fq + 64 * sub);
    auto ld = [&](f32x4 (&v)[8], int it) {
#pragma unroll
        for (int li = 0; li < 8; ++li)
            v[li] = ISW ? src[(size_t)(li * SH + it * 256 * SH) / 4] : src[(size_t)((li >> 2) * 16 * KG + ((li >> 1) & 1) * 32 + (li & 1) * 4 + it * 256) / 4];
    };
    const u32 un = (u32)p.rank * 16u + 2u * (u32)blk;
    const u32 pk_off = OFF_IN + (u32)p.rank * IN_SRC + (ISW ? IN_B : 0u) + 2u * (u32)blk * UNIT;
    const u32 ovf_off = OFF_OVF + ((u32)p.rank * 2u + (ISW ? 1u : 0u)) * (512u << 10) + 2u * (u32)blk * 1024u + lane * 16u;
    const u32 hdr_off = (ISW ? OFF_HB : OFF_HA) + un * 16u;
    f32x4 va[8], vb[8];
    ld(va, 0); ld(vb, 1);
    auto step = [&](f32x4 (&v)[8], int it) {
        u32 d[16]; float mx[2];
        if (ISW) {
#pragma unroll
            for (int u = 0; u < 2; ++u) { mx[u] = 0.f;
#pragma unroll
                for (int li = 0; li < 8; ++li) mx[u] = fmaxf(mx[u], fmaxf(fabsf(v[li][2 * u]), fabsf(v[li][2 * u + 1]))); }
        } else {
#pragma unroll
            for (int u = 0; u < 2; ++u) { mx[u] = 0.f;
#pragma unroll
                for (int li = 4 * u; li < 4 * u + 4; ++li) mx[u] = fmaxf(fmaxf(mx[u], fmaxf(fabsf(v[li][0]), fabsf(v[li][1]))), fmaxf(fabsf(v[li][2]), fabsf(v[li][3]))); }
        }
#pragma unroll
        for (int g = 0; g < 4; ++g)
#pragma unroll
            for (int q = 0; q < 4; ++q)
                d[g * 4 + q] = ISW ? cvt_pk_bf16(v[2 * q][g], v[2 * q + 1][g]) : cvt_pk_bf16(v[2 * g + (q >> 1)][(q & 1) * 2], v[2 * g + (q >> 1)][(q & 1) * 2 + 1]);
        asm volatile("" ::: "memory");
        if (it + 2 < 8) ld(v, it + 2);
        const u32 t = 4 * it + sub;
        u32x4 hd[2];
#pragma unroll
        for (int u = 0; u < 2; ++u) {
            u32x4 L; u32x2 N;
            pack_unit(d + 8 * u, mx[u], seq16, L, N, hd[u]);
            const u32 po = pk_off + t * IN_T + u * UNIT;
            if (((hd[u].x >> 23) & 7u) != 7u) {
                st16_sys(w0 + po + lane * 16, L); st16_sys(w1 + po + lane * 16, L);
                st8_sys(w0 + po + 1024 + lane * 8, N); st8_sys(w1 + po + 1024 + lane * 8, N);
            } else {
                const u32x4 r0 = (u32x4){d[8 * u], d[8 * u + 1], d[8 * u + 2], d[8 * u + 3]}, r1 = (u32x4){d[8 * u + 4], d[8 * u + 5], d[8 * u + 6], d[8 * u + 7]};
                st16_sys(w0 + po + lane * 16, r0); st16_sys(w1 + po + lane * 16, r0);
                st16_sys(w0 + ovf_off + t * 16384 + u * 1024, r1); st16_sys(w1 + ovf_off + t * 16384 + u * 1024, r1);
            }
        }
        drain();
        if (lane < 4) st16_sys(((lane & 2) ? w1 : w0) + hdr_off + t * 2048 + (lane & 1) * 16, (lane & 1) ? hd[1] : hd[0]);
    };
#pragma unroll 1
    for (int it = 0; it < 8; it += 2) {
        step(va, it); if (ISW && sub == 3 && it == 6) STAMP(p.ws[p.rank], 14); if (!ISW && sub == 0 && it == 0) STAMP(p.ws[p.rank], 1);
        step(vb, it + 1); if (ISW && sub == 3 && it == 6) STAMP(p.ws[p.rank], 15);
    }
}

struct Pay { u32x4 l0, l1, l2; u32x2 n0, n1, n2; };
#define LW(p, v) *(LAS u32x4*)(p) = (v)

template <bool EARLY>
__device__ __forceinline__ void decoder(const Params& p, LAS unsigned char* lds, int tm, int tn, int dw, int lane, Wait& wt, u32 seq16) {
    char* const ws = p.ws[p.rank];
    const __amdgpu_buffer_rsrc_t rs = __builtin_amdgcn_make_buffer_rsrc(ws, 0, (int)WS_NEED, 0x00020000);
    const u32 fr = lane & 15, fq = lane >> 4;
    u32 po[3], ov[3], hof[3], la0[3], la1[3];
    auto geo = [&](u32 s, u32& rb0, u32& rb1, u32& ck0, u32& ck1) {
        const u32 v = dw + 6 * s, b = v - 10;
        rb0 = v < 10 ? v * 2048u : LDS_B + ((b >> 2) * 4 + (b & 1) * 2) * 2048u; rb1 = v < 10 ? rb0 : rb0 + 2048u;
        ck0 = v < 10 ? 0u : ((b >> 1) & 1) * 4; ck1 = v < 10 ? 4u : ck0;
    };
#pragma unroll
    for (int s = 0; s < 3; ++s) {
        const u32 v = dw + 6 * s;
        if (v < 10) {
            const u32 am = min(10u * tm + v, 127u);
            po[s] = OFF_IN + (am >> 4) * IN_SRC + (am & 15) * UNIT; ov[s] = OFF_OVF + (am >> 4) * (1024u << 10) + (am & 15) * 1024u; hof[s] = OFF_HA + am * 16u;
        } else {
            const u32 bu = 8u * tn + v - 10;
            po[s] = OFF_IN + (bu >> 4) * IN_SRC + IN_B + (bu & 15) * UNIT; ov[s] = OFF_OVF + (bu >> 4) * (1024u << 10) + (512u << 10) + (bu & 15) * 1024u; hof[s] = OFF_HB + bu * 16u;
        }
        u32 rb0, rb1, ck0, ck1; geo(s, rb0, rb1, ck0, ck1);
        la0[s] = rb0 + fr * 128 + (((ck0 + fq) ^ (fr & 7)) << 4); la1[s] = rb1 + fr * 128 + (((ck1 + fq) ^ (fr & 7)) << 4);
    }
    const u32 hs = min((u32)lane >> 2, 2u);
    const u32 hoff = hs == 0 ? hof[0] : hs == 1 ? hof[1] : hof[2];
    u32 pr0, pr1, pc0, pc1; geo(hs, pr0, pr1, pc0, pc1);
    auto ldh = [&](int t) -> u32x4 { return __builtin_amdgcn_raw_buffer_load_b128(rs, hoff, t * 2048, 17); };
    auto ldp = [&](Pay& y, int t) {
        const u32 so = t * IN_T;
        y.l0 = __builtin_amdgcn_raw_buffer_load_b128(rs, lane * 16, po[0] + so, 17); y.n0 = __builtin_amdgcn_raw_buffer_load_b64(rs, lane * 8 + 1024, po[0] + so, 17);
        y.l1 = __builtin_amdgcn_raw_buffer_load_b128(rs, lane * 16, po[1] + so, 17); y.n1 = __builtin_amdgcn_raw_buffer_load_b64(rs, lane * 8 + 1024, po[1] + so, 17);
        y.l2 = __builtin_amdgcn_raw_buffer_load_b128(rs, lane * 16, po[2] + so, 17); y.n2 = __builtin_amdgcn_raw_buffer_load_b64(rs, lane * 8 + 1024, po[2] + so, 17);
    };
    auto waith = [&](u32x4& h, int t) {
        if (!hdr_ok(h, seq16)) { const u64 t0 = rtime(); u32 n = 0; do { if (spin_on(wt, t0, n)) break; h = ld16_sync(ws, hoff + t * 2048); } while (!hdr_ok(h, seq16)); }
    };
    auto dec = [&](const Pay& y, const u32x4& h, int t) {
        LAS unsigned char* st = lds + (t & 1) * STAGE;
        const u32 w0 = rdl(h.x, 0), w1 = rdl(h.x, 4), w2 = rdl(h.x, 8);
        const u32 m0 = (w0 >> 23) & 7u, m1 = (w1 >> 23) & 7u, m2 = (w2 >> 23) & 7u;
        if (m0 != 7u && m1 != 7u && m2 != 7u) {
            const u32 b0 = ((w0 >> 16) & 0x7Fu) * 0x01010101u, b1 = ((w1 >> 16) & 0x7Fu) * 0x01010101u, b2 = ((w2 >> 16) & 0x7Fu) * 0x01010101u;
            const u32x4 g0 = decode_group(y.l0.x, y.l0.y, y.n0.x, b0), g1 = decode_group(y.l0.z, y.l0.w, y.n0.y, b0);
            LW(st + la0[0], g0); LW(st + la1[0], g1);
            const u32x4 g2 = decode_group(y.l1.x, y.l1.y, y.n1.x, b1), g3 = decode_group(y.l1.z, y.l1.w, y.n1.y, b1);
            LW(st + la0[1], g2); LW(st + la1[1], g3);
            const u32x4 g4 = decode_group(y.l2.x, y.l2.y, y.n2.x, b2), g5 = decode_group(y.l2.z, y.l2.w, y.n2.y, b2);
            LW(st + la0[2], g4); LW(st + la1[2], g5);
        } else {
#pragma unroll
            for (int s = 0; s < 3; ++s) {
                const u32x4 l = s == 0 ? y.l0 : s == 1 ? y.l1 : y.l2; const u32x2 n = s == 0 ? y.n0 : s == 1 ? y.n1 : y.n2;
                const u32 w = s == 0 ? w0 : s == 1 ? w1 : w2;
                const u32x4 rw = ld16_sync(ws, ov[s] + t * 16384 + lane * 16);
                const bool raw = ((w >> 23) & 7u) == 7u;
                const u32 br = ((w >> 16) & 0x7Fu) * 0x01010101u;
                *(LAS u32x4*)(st + la0[s]) = raw ? l : decode_group(l.x, l.y, n.x, br); *(LAS u32x4*)(st + la1[s]) = raw ? rw : decode_group(l.z, l.w, n.y, br);
            }
        }
        if (m0 - 1u < 4u || m1 - 1u < 4u || m2 - 1u < 4u) {
            const u32 c = lane & 3, mode = (h.x >> 23) & 7u;
            const u64 a = (u64)h.x | ((u64)h.y << 32), b = (u64)h.z | ((u64)h.w << 32);
            const u32 e = (u32)((c < 2 ? a : b) >> ((c < 2 ? 26u : 16u) + 19u * (c & 1u))) & 0x7FFFFu;
            const u32 l = (e >> 4) & 63u, g = (e >> 3) & 1u, j = e & 7u, efr = l & 15u, efq = l >> 4;
            const u32 adr = (g ? pr1 : pr0) + efr * 128u + ((((g ? pc1 : pc0) + efq) ^ (efr & 7u)) << 4) + 2u * j + 1u;
            if (lane < 12 && mode != 7u && c < mode) st[adr] = (unsigned char)(e >> 11);
        }
    };
    Pay y0, y1, y2, y3; u32x4 h0, h1, h2, h3, s0, s1;
    h0 = ldh(0); h1 = ldh(1); h2 = ldh(2);
    waith(h0, 0); if (dw == 0) STAMP(ws, 1);
    ldp(y0, 0);
    waith(h1, 1); s1 = ldh(3); ldp(y1, 1);
    waith(h2, 2); s0 = ldh(4); ldp(y2, 2);
#define CYC_FROM 24
#define CYC_W0 0
#define CYC_W1 5
#define CT() 0
#define CYC_ACC2(u, hc, t0_, t1_, t2_, t3_) (void)t0_; (void)t1_; (void)t2_; (void)t3_
#define DEC_STEP(EARLY, u, hc, hn, sn, yc, yn) do { \
        const int t3 = min((u) + 3, NT - 1), t5 = min((u) + 5, NT - 1); \
        const u64 ta_ = CT(); \
        if (EARLY) { waith(sn, t3); hn = sn; sn = ldh(t5); ldp(yn, t3); } \
        const u64 tb_ = CT(); \
        dec(yc, hc, (u)); \
        const u64 tc_ = CT(); \
        if (!(EARLY)) { waith(sn, t3); hn = sn; sn = ldh(t5); ldp(yn, t3); } \
        const u64 t2_ = CT(); const u64 t0_ = 0, t1_ = (tb_ - ta_) + (t2_ - tc_); const u64 t2x_ = t1_ + (tc_ - tb_); \
        BARRIER(); const u64 t3_ = CT() - t2_ + t2x_; CYC_ACC2(u, hc, t0_, t1_, t2x_, t3_); } while (0)
#define DEC_LOOP(EARLY) _Pragma("unroll 1") for (int u = 0; u < NT; u += 4) { \
        DEC_STEP(EARLY, u, h0, h3, s1, y0, y3); \
        DEC_STEP(EARLY, u + 1, h1, h0, s0, y1, y0); \
        DEC_STEP(EARLY, u + 2, h2, h1, s1, y2, y1); \
        DEC_STEP(EARLY, u + 3, h3, h2, s0, y3, y2); \
        if (dw == 0 && (u & 7) == 4) STAMP(ws, 2 + (u >> 3)); }
    DEC_LOOP(EARLY)
#undef DEC_LOOP
#undef DEC_STEP
}

__device__ __forceinline__ void mma_wave(const Params& p, LAS unsigned char* lds, int tm, int tn, int wv, int lane, u32 seq16) {
    const int wr = wv >> 1, wc = wv & 1, fr = lane & 15, fq = lane >> 4;
    const u32 lp0 = fr * 128 + ((fq ^ (fr & 7)) << 4), lp1 = lp0 ^ 64u;
    f32x4 acc[5][4];
#pragma unroll
    for (int m = 0; m < 5; ++m)
#pragma unroll
        for (int n = 0; n < 4; ++n) acc[m][n] = (f32x4){0.f, 0.f, 0.f, 0.f};
    const u32 aoff = wr * 80 * 128, boff = LDS_B + wc * 64 * 128;
#pragma unroll 1
    for (int u = 0; u < NT; ++u) {
        BARRIER();
        LAS unsigned char* st = lds + (u & 1) * STAGE;
        bf16x8 af[2][5], bf[2][4];
#pragma unroll
        for (int kk = 0; kk < 2; ++kk) {
            const u32 lp = kk ? lp1 : lp0;
#pragma unroll
            for (int n = 0; n < 4; ++n) bf[kk][n] = *(const LAS bf16x8*)(st + boff + lp + n * 2048);
#pragma unroll
            for (int m = 0; m < 5; ++m) af[kk][m] = *(const LAS bf16x8*)(st + aoff + lp + m * 2048);
        }
        __builtin_amdgcn_sched_barrier(0);
#pragma unroll
        for (int kk = 0; kk < 2; ++kk)
#pragma unroll
            for (int m = 0; m < 5; ++m)
#pragma unroll
                for (int n = 0; n < 4; ++n) acc[m][n] = __builtin_amdgcn_mfma_f32_16x16x32_bf16(bf[kk][n], af[kk][m], acc[m][n], 0, 0, 0);
    }
    STAMP(p.ws[p.rank], 6);
    char* const dst = p.ws[(tn >> 1) & 7];
    const u32 cg4 = 2 * (tn & 1) + wc, r16 = 10 * tm + 5 * wr;
    const u32 ubase = OFF_Y + (u32)p.rank * Y_SRC + (cg4 * 128 + r16) * 2048u;
    u32x4 myhdr = (u32x4){0, 0, 0, 0};
#pragma unroll
    for (int m = 0; m < 5; ++m) {
        if (r16 + m < 128) {
            u32 d[8]; float mx = 0.f;
#pragma unroll
            for (int jj = 0; jj < 4; ++jj) {
                d[2 * jj] = cvt_pk_bf16(acc[m][0][jj], acc[m][1][jj]); d[2 * jj + 1] = cvt_pk_bf16(acc[m][2][jj], acc[m][3][jj]);
                mx = fmaxf(fmaxf(mx, fmaxf(fabsf(acc[m][0][jj]), fabsf(acc[m][1][jj]))), fmaxf(fabsf(acc[m][2][jj]), fabsf(acc[m][3][jj])));
            }
            u32x4 L, hdr; u32x2 N;
            pack_unit(d, mx, seq16, L, N, hdr);
            char* q = dst + ubase + m * 2048u;
            if (((hdr.x >> 23) & 7u) != 7u) { st16_sys(q + lane * 16, L); st8_sys(q + 1024 + lane * 8, N); }
            else { st16_sys(q + lane * 16, (u32x4){d[0], d[1], d[2], d[3]}); st16_sys(q + 1024 + lane * 16, (u32x4){d[4], d[5], d[6], d[7]}); }
            if (lane == m) myhdr = hdr;
        }
    }
    STAMP(p.ws[p.rank], 7);
    drain();
    if (lane < 5 && r16 + lane < 128) st16_sys(dst + OFF_HY + (((u32)p.rank * 4 + cg4) * 128 + r16 + lane) * 16u, myhdr);
    STAMP(p.ws[p.rank], 8);
}

__global__ void __launch_bounds__(NTHR, 3) k_main(Params p) {
    extern __shared__ __attribute__((aligned(16))) unsigned char smem[];
    LAS unsigned char* lds = (LAS unsigned char*)smem;
    const int b = blockIdx.x, wv = rfl(threadIdx.x >> 6), lane = threadIdx.x & 63;
    char* const ws = p.ws[p.rank];
    if (wv == 0) STAMP(ws, 0);
    GAS u32* cntp = (GAS u32*)(ws + OFF_CNT + b * 128);
    const u32 call = rfl(__hip_atomic_load(cntp, __ATOMIC_RELAXED, __HIP_MEMORY_SCOPE_AGENT)) + 1u;
    const u32 seq16 = call % 65535u + 1u;
    Wait wt{(GAS u32*)(ws + OFF_TMO), call, false};
    const int xg = b & 7, slot = b >> 3;
    if (slot < 4) {
        const int cu = slot * 8 + xg;
        if (wv < 8) { if (wv & 1) pusher<true>(p, cu, wv >> 1, lane, seq16); else pusher<false>(p, cu, wv >> 1, lane, seq16); }
    } else if (ROLE_OF(b) == 1) {
        const int s = slot - 4, tm = s >> 1, tn = 2 * xg + (s & 1);
        if (wv >= 4) { if (wv & 1) decoder<false>(p, lds, tm, tn, wv - 4, lane, wt, seq16); else decoder<true>(p, lds, tm, tn, wv - 4, lane, wt, seq16); } else mma_wave(p, lds, tm, tn, wv, lane, seq16);
    }
    const __amdgpu_buffer_rsrc_t rs = __builtin_amdgcn_make_buffer_rsrc(ws, 0, (int)WS_NEED, 0x00020000);
    const u32 ri = b >> 7, rj = (b >> 6) & 1, r16 = 2 * (b & 63) + ((wv >> 2) & 1), cg4 = wv & 3;
    const u32 snd0 = (ri << 2) | (rj << 1);
    float y[16];
    LAS u32* misc = (LAS u32*)(lds + LDS_MISC);
    if (wv < 8) {
        const u32 hoff = OFF_HY + (((snd0 + (lane & 1)) * 4 + cg4) * 128 + r16) * 16u;
        u32x4 h = __builtin_amdgcn_raw_buffer_load_b128(rs, hoff, 0, 17);
        if (!hdr_ok(h, seq16)) { const u64 t0 = rtime(); u32 n = 0; do { if (spin_on(wt, t0, n)) break; h = ld16_sync(ws, hoff); } while (!hdr_ok(h, seq16)); }
        STAMP(ws, 24 + wv);
        u32x4 l[2]; u32x2 n[2];
#pragma unroll
        for (int s = 0; s < 2; ++s) {
            const u32 ub = OFF_Y + (snd0 + s) * Y_SRC + (cg4 * 128 + r16) * 2048u;
            l[s] = __builtin_amdgcn_raw_buffer_load_b128(rs, lane * 16, ub, 17);
            n[s] = __builtin_amdgcn_raw_buffer_load_b64(rs, lane * 8 + 1024, ub, 17);
        }
#pragma unroll
        for (int s = 0; s < 2; ++s) {
            const u32 w0 = rdl(h.x, s), mode = (w0 >> 23) & 7u;
            u32 d[8];
            if (mode != 7u) {
                const u32 brep = ((w0 >> 16) & 0x7Fu) * 0x01010101u;
                const u32x4 g0 = decode_group(l[s].x, l[s].y, n[s].x, brep), g1 = decode_group(l[s].z, l[s].w, n[s].y, brep);
                d[0] = g0.x; d[1] = g0.y; d[2] = g0.z; d[3] = g0.w; d[4] = g1.x; d[5] = g1.y; d[6] = g1.z; d[7] = g1.w;
                if (mode) {
                    const u64 a = (u64)w0 | ((u64)rdl(h.y, s) << 32), bb = (u64)rdl(h.z, s) | ((u64)rdl(h.w, s) << 32);
                    for (u32 c = 0; c < mode; ++c) {
                        const u32 e = hdr_exc(a, bb, c), el = (e >> 4) & 63u, j = e & 15u, hb = e >> 11, sh = (j & 1u) * 16u + 8u;
#pragma unroll
                        for (int q = 0; q < 8; ++q) if (j >> 1 == (u32)q && (u32)lane == el) d[q] = (d[q] & ~(0xFFu << sh)) | (hb << sh);
                    }
                }
            } else {
                const u32 ub = OFF_Y + (snd0 + s) * Y_SRC + (cg4 * 128 + r16) * 2048u;
                const u32x4 hi = ld16_sync(ws, ub + lane * 16 + 1024);
                d[0] = l[s].x; d[1] = l[s].y; d[2] = l[s].z; d[3] = l[s].w; d[4] = hi.x; d[5] = hi.y; d[6] = hi.z; d[7] = hi.w;
            }
#pragma unroll
            for (int q = 0; q < 8; ++q) {
                const float lo = __uint_as_float(d[q] << 16), hi = __uint_as_float(d[q] & 0xFFFF0000u);
                if (s == 0) { y[2 * q] = lo; y[2 * q + 1] = hi; } else { y[2 * q] += lo; y[2 * q + 1] += hi; }
            }
        }
        float mx = 0.f;
#pragma unroll
        for (int q = 0; q < 16; ++q) mx = fmaxf(mx, fabsf(y[q]));
        const u32 wm = wave_max_u32(__float_as_uint(mx));
        if (lane == 0) misc[wv] = wm;
        STAMPD(ws, 16 + wv);
    } else {
#pragma unroll
        for (int q = 0; q < 16; ++q) y[q] = 0.f;
    }
    BARRIER();
    if (wv == 0) {
        const u32 v = wave_max_u32(misc[lane & 7]);
        if (lane < NR) __hip_atomic_store((GAS u64*)(p.ws[lane] + OFF_AMAX + ((u32)p.rank * 256 + b) * 8), ((u64)call << 32) | v, __ATOMIC_RELAXED, __HIP_MEMORY_SCOPE_SYSTEM);
        STAMPD(ws, 11);
    }
    if (wv < 8) {
        const u32 go = OFF_AMAX + wv * 2048 + lane * 32;
        u32x4 g0 = __builtin_amdgcn_raw_buffer_load_b128(rs, go, 0, 17), g1 = __builtin_amdgcn_raw_buffer_load_b128(rs, go + 16, 0, 17);
        auto ok = [&]() { return __ballot(g0.y == call && g0.w == call && g1.y == call && g1.w == call) == ~0ull; };
        if (!ok()) { const u64 t0 = rtime(); u32 n = 0; do { if (spin_on(wt, t0, n)) break;
            asm volatile("global_load_dwordx4 %0, %2, %3 sc0 sc1\n\tglobal_load_dwordx4 %1, %2, %3 offset:16 sc0 sc1\n\ts_waitcnt vmcnt(0)" : "=&v"(g0), "=&v"(g1) : "v"(go), "s"(ws) : "memory"); } while (!ok()); }
        const u32 gm = wave_max_u32(max(max(g0.x, g0.z), max(g1.x, g1.z)));
        if (lane == 0) misc[8 + wv] = gm;
    }
    BARRIER();
    if (wv == 0) STAMP(ws, 12);
    if (wv < 8) {
        const float amax = __uint_as_float(wave_max_u32(misc[8 + (lane & 7)]));
        const bool bad = rfl(__hip_atomic_load(wt.tmo, __ATOMIC_RELAXED, __HIP_MEMORY_SCOPE_AGENT)) == call;
        const float inv = amax > 0.f ? 448.0f / amax : 0.f;
        u32 o[4];
#pragma unroll
        for (int q = 0; q < 4; ++q) {
            float v[4];
#pragma unroll
            for (int e = 0; e < 4; ++e) v[e] = __builtin_amdgcn_fmed3f(y[4 * q + e] * inv, -448.f, 448.f);
            int t = __builtin_amdgcn_cvt_pk_fp8_f32(v[0], v[1], 0, false); t = __builtin_amdgcn_cvt_pk_fp8_f32(v[2], v[3], t, true); o[q] = (u32)t;
        }
        const u32 R = r16 * 16 + (lane & 15), row = 512 * (R >> 8) + 256 * ri + (R & 255), col = 256 * rj + 64 * cg4 + 16 * (lane >> 4);
        const u32x4 ov = bad ? (u32x4){0x7f7f7f7fu, 0x7f7f7f7fu, 0x7f7f7f7fu, 0x7f7f7f7fu} : (u32x4){o[0], o[1], o[2], o[3]};
        *(GAS u32x4*)(p.out + (size_t)row * SH + col) = ov;
    }
    if (threadIdx.x == 0) __hip_atomic_store(cntp, call, __ATOMIC_RELAXED, __HIP_MEMORY_SCOPE_AGENT);
    if (wv == 0) STAMP(ws, 13);
}

extern "C" void kernel_launch(int rank, int nranks, ncclComm_t comm, hipStream_t stream,
                              void* const* d_in, const int* in_sizes, int n_in,
                              void* d_out, int out_size, const int* in_dtypes, int out_dtype,
                              void* const* d_out_all, void* const* d_ws_all, size_t ws_bytes) {
    if (nranks != NR || (unsigned)rank >= (unsigned)NR || ws_bytes < WS_NEED || n_in < 2 || in_sizes[0] != SH * KG || in_sizes[1] != KG * SH || out_size != MG * SH) {
        fprintf(stderr, "kernel_launch: unexpected shapes\n"); return; }
    HIP_CHECK(hipFuncSetAttribute((const void*)k_main, hipFuncAttributeMaxDynamicSharedMemorySize, LDS_BYTES));
    Params p;
    p.x = (const float*)d_in[0]; p.w = (const float*)d_in[1]; p.out = (unsigned char*)d_out;
    for (int i = 0; i < NR; ++i) p.ws[i] = (char*)d_ws_all[i];
    p.rank = rank; p.pad = 0;
    k_main<<<256, NTHR, LDS_BYTES, stream>>>(p);
    HIP_CHECK(hipGetLastError());
}
```

```cpp
#include <hip/hip_runtime.h>
#include <rccl/rccl.h>
#include <cstdio>
#include <cstdint>

#define HIP_CHECK(expr) do { hipError_t _e = (expr); if (_e != hipSuccess) { \
    fprintf(stderr, "HIP error %s at %s:%d\n", hipGetErrorString(_e), __FILE__, __LINE__); \
    return; } } while (0)

typedef unsigned u32;
typedef unsigned long long u64;
typedef u32 u32x4 __attribute__((ext_vector_type(4)));
typedef u32 u32x2 __attribute__((ext_vector_type(2)));
typedef float f32x4 __attribute__((ext_vector_type(4)));
typedef float f32x16 __attribute__((ext_vector_type(16)));
typedef short bf16x8 __attribute__((ext_vector_type(8)));
#define LAS __attribute__((address_space(3)))
#define GAS __attribute__((address_space(1)))

constexpr int NR = 8, MG = 4096, KG = 4096, SH = 512;
constexpr int NT = 32;
constexpr int NTHR = 640;
constexpr u64 DEADLINE = 150000000ull;
constexpr u32 UNIT = 1536;
constexpr u32 OFF_CNT = 0;
constexpr u32 OFF_TMO = 0x8000;
constexpr u32 OFF_TICK = 0x8100;
constexpr u32 OFF_LMAX = 0x10000;
constexpr u32 OFF_GMAX = 0x14000;
constexpr u32 OFF_STAMP = 0x20000;
constexpr u32 OFF_HA = 0x30000;
constexpr u32 OFF_HB = 0x40000;
constexpr u32 OFF_HY = 0x50000;
constexpr u32 OFF_IN = 2u << 20;
constexpr u32 IN_SRC = 2u << 20, IN_B = 768u << 10, IN_T = 16 * UNIT;
constexpr u32 OFF_Y = 18u << 20;
constexpr u32 Y_SRC = 1u << 20;
constexpr u32 OFF_OVF = 26u << 20;
constexpr size_t WS_NEED = 34u << 20;
constexpr int STAGE = 36864, NSTAGE = 4, LDS_B = 20480, LDS_MISC = NSTAGE * STAGE, LDS_BYTES = 148 * 1024;

struct Params { const float* x; const float* w; unsigned char* out; char* ws[NR]; int rank; int pad; };

__device__ __forceinline__ u32 cvt_pk_bf16(float lo, float hi) { u32 r; asm("v_cvt_pk_bf16_f32 %0, %1, %2" : "=v"(r) : "v"(lo), "v"(hi)); return r; }
__device__ __forceinline__ void st16_sys(void* p, u32x4 v) { asm volatile("global_store_dwordx4 %0, %1, off sc0 sc1\n\ts_nop 1" :: "v"(p), "v"(v) : "memory"); }
__device__ __forceinline__ void st8_sys(void* p, u32x2 v) { asm volatile("global_store_dwordx2 %0, %1, off sc0 sc1\n\ts_nop 1" :: "v"(p), "v"(v) : "memory"); }
__device__ __forceinline__ u32x4 ld16_sync(const char* base, u32 off) { u32x4 r; asm volatile("global_load_dwordx4 %0, %1, %2 sc0 sc1\n\ts_waitcnt vmcnt(0)" : "=&v"(r) : "v"(off), "s"(base) : "memory"); return r; }
__device__ __forceinline__ u32x2 ld8_sync(const char* base, u32 off) { u32x2 r; asm volatile("global_load_dwordx2 %0, %1, %2 sc0 sc1\n\ts_waitcnt vmcnt(0)" : "=&v"(r) : "v"(off), "s"(base) : "memory"); return r; }
__device__ __forceinline__ u32 ld4_sync_agent(const void* p) { u32 r; asm volatile("global_load_dword %0, %1, off sc1\n\ts_waitcnt vmcnt(0)" : "=&v"(r) : "v"(p) : "memory"); return r; }
__device__ __forceinline__ void drain() { asm volatile("s_waitcnt vmcnt(0)" ::: "memory"); }
__device__ __forceinline__ u64 rtime() { return __builtin_amdgcn_s_memrealtime(); }
__device__ __forceinline__ u32 rfl(u32 v) { return __builtin_amdgcn_readfirstlane(v); }
__device__ __forceinline__ u32 rdl(u32 v, int l) { return __builtin_amdgcn_readlane(v, l); }
__device__ __forceinline__ u32 wave_max_u32(u32 v) {
    v = max(v, (u32)__builtin_amdgcn_update_dpp(0, (int)v, 0x111, 0xf, 0xf, false));
    v = max(v, (u32)__builtin_amdgcn_update_dpp(0, (int)v, 0x112, 0xf, 0xf, false));
    v = max(v, (u32)__builtin_amdgcn_update_dpp(0, (int)v, 0x114, 0xf, 0xf, false));
    v = max(v, (u32)__builtin_amdgcn_update_dpp(0, (int)v, 0x118, 0xf, 0xf, false));
    v = max(v, (u32)__builtin_amdgcn_update_dpp(0, (int)v, 0x142, 0xa, 0xf, false));
    v = max(v, (u32)__builtin_amdgcn_update_dpp(0, (int)v, 0x143, 0xc, 0xf, false));
    return rdl(v, 63);
}
#define BARRIER() do { asm volatile("s_waitcnt lgkmcnt(0)" ::: "memory"); __builtin_amdgcn_s_barrier(); asm volatile("" ::: "memory"); } while (0)

#define ROLE_OF(b) (((b) >> 3) < 4 ? 0 : ((b) >> 3) < 30 ? 1 : 2)
#define STAMP(ws, slot) do {} while (0)
#define STAMPD(ws, slot) do {} while (0)

constexpr int LEADER = 248;

__device__ __forceinline__ void pack_unit(const u32* d, float amax_lane, u32 seq16, u32x4& L, u32x2& N, u32x4& hdr) {
    u32 H[4];
#pragma unroll
    for (int q = 0; q < 4; ++q) { L[q] = __builtin_amdgcn_perm(d[2 * q + 1], d[2 * q], 0x06040200u); H[q] = __builtin_amdgcn_perm(d[2 * q + 1], d[2 * q], 0x07050301u); }
    const u32 mb = wave_max_u32(__float_as_uint(amax_lane));
    u32 h7 = (mb + 0x7FFFu + ((mb >> 16) & 1u)) >> 24;
    h7 = h7 > 127u ? 127u : h7;
    const u32 base = h7 > 7u ? h7 - 7u : 0u, brep = base * 0x01010101u;
    u32 b[4], X[4];
#pragma unroll
    for (int q = 0; q < 4; ++q) { const u32 t = (H[q] | 0x80808080u) - brep; b[q] = (t ^ 0x80808080u) & 0xF8F8F8F8u; X[q] = (H[q] & 0x80808080u) | (t & 0x07070707u); }
    N[0] = X[0] | __builtin_amdgcn_alignbit(X[1], X[1], 28); N[1] = X[2] | __builtin_amdgcn_alignbit(X[3], X[3], 28);
    u32 cnt = 0; u64 ea = 0, eb = 0;
    u64 m = __ballot((b[0] | b[1] | b[2] | b[3]) != 0);
    if (m) {
        u32 bm = 0;
#pragma unroll
        for (int q = 0; q < 4; ++q) bm |= (((((((b[q] & 0x7F7F7F7Fu) + 0x7F7F7F7Fu) | b[q]) & 0x80808080u) >> 7) * 0x01020408u) >> 24 & 0xFu) << (4 * q);
        while (m && cnt <= 4) {
            const int l = __builtin_ctzll(m); m &= m - 1;
            u32 mk = rdl(bm, l);
            const u32 h0 = rdl(H[0], l), h1 = rdl(H[1], l), h2 = rdl(H[2], l), h3 = rdl(H[3], l);
            while (mk) {
                const u32 j = (u32)__builtin_ctz(mk); mk &= mk - 1;
                const u32 hq = (j & 8u) ? ((j & 4u) ? h3 : h2) : ((j & 4u) ? h1 : h0);
                const u64 e = (u64)((u32)l * 16u + j) | ((u64)((hq >> (8u * (j & 3u))) & 0xFFu) << 11);
                if (cnt < 2) ea |= e << (26u + 19u * cnt); else if (cnt < 4) eb |= e << (16u + 19u * (cnt - 2u));
                ++cnt;
            }
        }
    }
    const u32 mode = cnt > 4 ? 7u : cnt;
    ea |= (u64)seq16 | ((u64)base << 16) | ((u64)mode << 23); eb |= (u64)seq16;
    hdr = (u32x4){(u32)ea, (u32)(ea >> 32), (u32)eb, (u32)(eb >> 32)};
}
__device__ __forceinline__ u32x4 decode_group(u32 LA, u32 LB, u32 n, u32 brep) {
    const u32 HA = (n & 0x87878787u) + brep, HB = (__builtin_amdgcn_alignbit(n, n, 4) & 0x87878787u) + brep;
    return (u32x4){__builtin_amdgcn_perm(HA, LA, 0x05010400u), __builtin_amdgcn_perm(HA, LA, 0x07030602u),
                   __builtin_amdgcn_perm(HB, LB, 0x05010400u), __builtin_amdgcn_perm(HB, LB, 0x07030602u)};
}
__device__ __forceinline__ u32 hdr_exc(u64 a, u64 b, u32 c) { return (u32)((c < 2 ? a >> (26u + 19u * c) : b >> (16u + 19u * (c - 2u)))) & 0x7FFFFu; }
__device__ __forceinline__ bool hdr_ok(const u32x4& h, u32 seq16) { return __ballot(((h.x & 0xFFFFu) == seq16) && ((h.z & 0xFFFFu) == seq16)) == ~0ull; }

__device__ __forceinline__ void lds_signal(LAS u32* c) {
    asm volatile("" ::: "memory");
    if ((threadIdx.x & 63) == 0) __hip_atomic_fetch_add(c, 1u, __ATOMIC_RELAXED, __HIP_MEMORY_SCOPE_WORKGROUP);
    asm volatile("" ::: "memory");
}
__device__ __forceinline__ void lds_wait(LAS u32* c, u32 target) {
    while (rfl(__hip_atomic_load(c, __ATOMIC_RELAXED, __HIP_MEMORY_SCOPE_WORKGROUP)) < target) __builtin_amdgcn_s_sleep(1);
    asm volatile("" ::: "memory");
}

struct Wait { GAS u32* tmo; u32 call; bool dead; };
__device__ __forceinline__ bool spin_on(Wait& wt, u64 t0, u32& n) {
    if (wt.dead) return true;
    if ((++n & 15u) == 0u) {
        if (rfl(ld4_sync_agent((const void*)wt.tmo)) == wt.call || rtime() - t0 > DEADLINE) {
            __hip_atomic_store(wt.tmo, wt.call, __ATOMIC_RELAXED, __HIP_MEMORY_SCOPE_AGENT); wt.dead = true; return true; }
    }
    __builtin_amdgcn_s_sleep(1);
    return false;
}

#define PW 2
constexpr u32 PWAVES = 32 * PW;
template <bool ISW>
__device__ __forceinline__ void pusher(const Params& p, LAS unsigned char* lds, int wvi, int gw, int lane, u32 callv) {
    const int fr = lane & 15, fq = lane >> 4;
    GAS u32* const tick = (GAS u32*)(p.ws[p.rank] + OFF_TICK + (ISW ? 128 : 0));
    const GAS char* const src = (const GAS char*)(ISW ? p.w : p.x);
    const u32 loff = ISW ? (u32)(8 * fq * SH + 4 * fr) * 4u : (u32)(fr * KG + 8 * fq) * 4u;
    const u32 lbase = (u32)(uintptr_t)lds + (u32)wvi * 16384u;
    auto ld = [&](u32 buf, u32 tau) {
        const u32 t = min(tau >> 5, (u32)NT - 1u), h = (tau >> 4) & 1u, kh = (tau >> 3) & 1u, blk = tau & 7u;
        const u32 so = ISW ? ((2048u * kh + 64u * t + 32u * (blk & 1u)) * SH + 256u * h + 64u * (blk >> 1)) * 4u : ((256u * h + 32u * blk) * KG + 2048u * kh + 64u * t) * 4u;
        const GAS char* const sb = src + so;
        const u32 l0 = lbase + buf * 8192u;
        u32 keep;
#define GL(i, off) "s_mov_b32 m0, %" #i "\n\ts_nop 0\n\tglobal_load_lds_dwordx4 " off "\n\t"
        if (ISW) {
            asm volatile("s_mov_b32 %0, m0\n\t"
                         GL(6, "%1, %5") GL(7, "%2, %5") GL(8, "%3, %5") GL(9, "%4, %5") GL(10, "%14, %5") GL(11, "%15, %5") GL(12, "%16, %5") GL(13, "%17, %5")
                         "s_mov_b32 m0, %0"
                         : "=&s"(keep)
                         : "v"(loff), "v"(loff + 2048u), "v"(loff + 4096u), "v"(loff + 6144u), "s"(sb),
                           "s"(l0), "s"(l0 + 1024u), "s"(l0 + 2048u), "s"(l0 + 3072u), "s"(l0 + 4096u), "s"(l0 + 5120u), "s"(l0 + 6144u), "s"(l0 + 7168u),
                           "v"(loff + 8192u), "v"(loff + 10240u), "v"(loff + 12288u), "v"(loff + 14336u) : "memory");
        } else {
            asm volatile("s_mov_b32 %0, m0\n\t"
                         GL(6, "%1, %5") GL(7, "%2, %5") GL(8, "%3, %5") GL(9, "%4, %5") GL(10, "%14, %5") GL(11, "%15, %5") GL(12, "%16, %5") GL(13, "%17, %5")
                         "s_mov_b32 m0, %0"
                         : "=&s"(keep)
                         : "v"(loff), "v"(loff + 16u), "v"(loff + 128u), "v"(loff + 144u), "s"(sb),
                           "s"(l0), "s"(l0 + 1024u), "s"(l0 + 2048u), "s"(l0 + 3072u), "s"(l0 + 4096u), "s"(l0 + 5120u), "s"(l0 + 6144u), "s"(l0 + 7168u),
                           "v"(loff + 16u * KG * 4u), "v"(loff + 16u * KG * 4u + 16u), "v"(loff + 16u * KG * 4u + 128u), "v"(loff + 16u * KG * 4u + 144u) : "memory");
        }
#undef GL
    };
    u32 ta = gw, tb = gw + PWAVES, tc = gw + 2 * PWAVES;
    ld(0, ta); ld(1, tb);
    asm volatile("s_waitcnt vmcnt(0)" ::: "memory");
    const u32 call = rfl(callv) + 1u, seq16 = call % 65535u + 1u;
    const u32 tbase = (call - 1u) * 1024u - 3u * PWAVES;
    u32x4 ph = (u32x4){0, 0, 0, 0}; char* pa = nullptr; bool have = false;
    auto step = [&](u32 buf, u32& tau) {
        f32x4 v[8];
#pragma unroll
        for (int li = 0; li < 8; ++li) v[li] = *(const LAS f32x4*)(lds + wvi * 16384 + buf * 8192 + li * 1024 + lane * 16);
        u32 d[16]; float mx[2];
        if (ISW) {
#pragma unroll
            for (int u = 0; u < 2; ++u) { mx[u] = 0.f;
#pragma unroll
                for (int li = 0; li < 8; ++li) mx[u] = fmaxf(mx[u], fmaxf(fabsf(v[li][2 * u]), fabsf(v[li][2 * u + 1]))); }
        } else {
#pragma unroll
            for (int u = 0; u < 2; ++u) { mx[u] = 0.f;
#pragma unroll
                for (int li = 4 * u; li < 4 * u + 4; ++li) mx[u] = fmaxf(fmaxf(mx[u], fmaxf(fabsf(v[li][0]), fabsf(v[li][1]))), fmaxf(fabsf(v[li][2]), fabsf(v[li][3]))); }
        }
#pragma unroll
        for (int g = 0; g < 4; ++g)
#pragma unroll
            for (int q = 0; q < 4; ++q)
                d[g * 4 + q] = ISW ? cvt_pk_bf16(v[2 * q][g], v[2 * q + 1][g]) : cvt_pk_bf16(v[2 * g + (q >> 1)][(q & 1) * 2], v[2 * g + (q >> 1)][(q & 1) * 2 + 1]);
        u32 drawn = 0;
        if (lane == 0) asm volatile("global_atomic_add %0, %1, %2, off sc0" : "=v"(drawn) : "v"(tick), "v"(1u) : "memory");
        const u32 cur = tau;
        const u32 t = cur >> 5, h = (cur >> 4) & 1u, kh = (cur >> 3) & 1u, blk = cur & 7u;
        u32 d0 = ISW ? ((h << 1) | kh) : ((h << 2) | kh), d1 = d0 | (ISW ? 4u : 2u);
        if ((cur ^ (cur >> 5)) & 1u) { const u32 x = d0; d0 = d1; d1 = x; }
        char* const w0 = p.ws[d0 & 7u]; char* const w1 = p.ws[d1 & 7u];
        const u32 pk = OFF_IN + (u32)p.rank * IN_SRC + (ISW ? IN_B : 0u) + t * IN_T + 2u * blk * UNIT;
        u32x4 hd[2];
#pragma unroll
        for (int u = 0; u < 2; ++u) {
            u32x4 L; u32x2 N;
            pack_unit(d + 8 * u, mx[u], seq16, L, N, hd[u]);
            const u32 po = pk + u * UNIT;
            if (((hd[u].x >> 23) & 7u) != 7u) {
                st16_sys(w0 + po + lane * 16, L); st16_sys(w1 + po + lane * 16, L);
                st8_sys(w0 + po + 1024 + lane * 8, N); st8_sys(w1 + po + 1024 + lane * 8, N);
            } else {
                const u32 ov = OFF_OVF + ((u32)p.rank * 2u + (ISW ? 1u : 0u)) * (512u << 10) + t * 16384u + (2u * blk + u) * 1024u + lane * 16u;
                const u32x4 r0 = (u32x4){d[8 * u], d[8 * u + 1], d[8 * u + 2], d[8 * u + 3]}, r1 = (u32x4){d[8 * u + 4], d[8 * u + 5], d[8 * u + 6], d[8 * u + 7]};
                st16_sys(w0 + po + lane * 16, r0); st16_sys(w1 + po + lane * 16, r0);
                st16_sys(w0 + ov, r1); st16_sys(w1 + ov, r1);
            }
        }
        asm volatile("s_waitcnt lgkmcnt(0)" ::: "memory");
        ld(buf, tc); tau = tc;
        asm volatile("s_waitcnt vmcnt(16)" : "+v"(drawn) :: "memory");
        if (have && lane < 4) st16_sys(pa, ph);
        pa = ((lane & 2) ? w1 : w0) + (ISW ? OFF_HB : OFF_HA) + t * 2048u + ((u32)p.rank * 16u + 2u * blk + (lane & 1)) * 16u;
        ph = (lane & 1) ? hd[1] : hd[0]; have = true;
        tc = rfl(drawn) - tbase;
    };
#pragma unroll 1
    while (true) {
        if (ta >= 1024u) break;
        step(0, ta);
        if (tb >= 1024u) break;
        step(1, tb);
    }
    drain();
    if (lane < 4) st16_sys(pa, ph);
    STAMP(p.ws[p.rank], 15);
}

struct Pay { u32x4 l0, l1, l2; u32x2 n0, n1, n2; };
#define LW(p, v) *(LAS u32x4*)(p) = (v)

__device__ __forceinline__ void decoder(const Params& p, LAS unsigned char* lds, int tm, int tn, int dw, int lane, Wait& wt, u32 seq16) {
    char* const ws = p.ws[p.rank];
    LAS u32* const cnt = (LAS u32*)(lds + LDS_MISC);
    const __amdgpu_buffer_rsrc_t rs = __builtin_amdgcn_make_buffer_rsrc(ws, 0, (int)WS_NEED, 0x00020000);
    const u32 fr = lane & 15, fq = lane >> 4, cls = dw % 3, v0 = 9 * (dw / 3);
    auto uaddr = [&](u32 v, u32 g, u32 efr, u32 efq) -> u32 {
        const u32 b = v - 10u; const bool isA = v < 10u;
        const u32 row = isA ? 16u * v + efr : 64u * (b >> 2) + 32u * (efr >> 3) + 8u * (efr & 3u) + 4u * ((efr >> 2) & 1u) + 2u * (b & 1u) + g;
        const u32 ck = (isA ? 4u * g : 4u * ((b >> 1) & 1u)) + efq;
        return (isA ? 0u : (u32)LDS_B) + row * 128u + ((ck ^ (row & 7u) ^ ((row >> 3) & 3u)) << 4);
    };
    auto uidx = [&](u32 v) -> u32 { return v < 10u ? min(10u * tm + v, 127u) : 8u * tn + v - 10u; };
    u32 po[9], la0[9], la1[9];
#pragma unroll
    for (int s = 0; s < 9; ++s) {
        const u32 v = v0 + s, x = uidx(v);
        po[s] = OFF_IN + (x >> 4) * IN_SRC + (v < 10u ? 0u : IN_B) + (x & 15) * UNIT;
        la0[s] = uaddr(v, 0, fr, fq); la1[s] = uaddr(v, 1, fr, fq);
    }
    const u32 hv = v0 + min((u32)lane >> 2, 8u);
    const u32 hoff = (hv < 10u ? OFF_HA : OFF_HB) + uidx(hv) * 16u;
    struct Pay9 { u32x4 l[9]; u32x2 n[9]; };
    auto ldh = [&](int t) -> u32x4 { return __builtin_amdgcn_raw_buffer_load_b128(rs, hoff, t * 2048, 17); };
    auto ldp = [&](Pay9& y, int t) {
        const u32 so = t * IN_T;
#pragma unroll
        for (int s = 0; s < 9; ++s) {
            y.l[s] = __builtin_amdgcn_raw_buffer_load_b128(rs, lane * 16, po[s] + so, 17);
            y.n[s] = __builtin_amdgcn_raw_buffer_load_b64(rs, lane * 8 + 1024, po[s] + so, 17);
        }
    };
    auto waith = [&](u32x4& h, int t) {
        if (!hdr_ok(h, seq16)) { const u64 t0 = rtime(); u32 n = 0; do { if (spin_on(wt, t0, n)) break; h = ld16_sync(ws, hoff + t * 2048); } while (!hdr_ok(h, seq16)); }
    };
    auto dec = [&](const Pay9& y, const u32x4& h, int t) {
        LAS unsigned char* st = lds + (t & 3) * STAGE;
        if (t >= NSTAGE) lds_wait(cnt + 24 + (t & 3), 4u * (u32)(t >> 2));
        const u32 mode = (h.x >> 23) & 7u, brl = ((h.x >> 16) & 0x7Fu) * 0x01010101u;
        if (__ballot(mode == 7u) == 0ull) {
#pragma unroll
            for (int s = 0; s < 9; ++s) {
                const u32 br = rdl(brl, 4 * s);
                const u32x4 g0 = decode_group(y.l[s].x, y.l[s].y, y.n[s].x, br), g1 = decode_group(y.l[s].z, y.l[s].w, y.n[s].y, br);
                LW(st + la0[s], g0); LW(st + la1[s], g1);
            }
        } else {
#pragma unroll
            for (int s = 0; s < 9; ++s) {
                const u32 v = v0 + s, x = uidx(v), br = rdl(brl, 4 * s);
                const u32x4 rw = ld16_sync(ws, OFF_OVF + (x >> 4) * (1024u << 10) + (v < 10u ? 0u : (512u << 10)) + (x & 15) * 1024u + t * 16384 + lane * 16);
                const bool raw = rdl(mode, 4 * s) == 7u;
                *(LAS u32x4*)(st + la0[s]) = raw ? y.l[s] : decode_group(y.l[s].x, y.l[s].y, y.n[s].x, br);
                *(LAS u32x4*)(st + la1[s]) = raw ? rw : decode_group(y.l[s].z, y.l[s].w, y.n[s].y, br);
            }
        }
        if (__ballot(mode - 1u < 4u) != 0ull) {
            const u32 c = lane & 3;
            const u64 a = (u64)h.x | ((u64)h.y << 32), b = (u64)h.z | ((u64)h.w << 32);
            const u32 e = (u32)((c < 2 ? a : b) >> ((c < 2 ? 26u : 16u) + 19u * (c & 1u))) & 0x7FFFFu;
            const u32 l = (e >> 4) & 63u;
            const u32 adr = uaddr(hv, (e >> 3) & 1u, l & 15u, l >> 4) + 2u * (e & 7u) + 1u;
            if (lane < 36 && mode != 7u && c < mode) st[adr] = (unsigned char)(e >> 11);
        }
        lds_signal(cnt + 16 + (t & 3));
    };
    Pay9 ya, yb; u32x4 ha, hb, sn;
    ha = ldh(cls);
    waith(ha, cls); if (dw == 0) STAMP(ws, 1);
    sn = ldh(cls + 3); ldp(ya, cls);
#define HSTAMP(u) do {} while (0)
#define SN_OK hdr_ok(sn, seq16)
#define DEC_INST(u, hc, hn, yc, yn) do { \
        const int t3 = min((u) + 3, NT - 1), t6 = min((u) + 6, NT - 1); \
        if (SN_OK) { hn = sn; sn = ldh(t6); ldp(yn, t3); dec(yc, hc, (u)); asm volatile("; early" ::: "memory"); } \
        else { dec(yc, hc, (u)); waith(sn, t3); hn = sn; sn = ldh(t6); ldp(yn, t3); asm volatile("; late" ::: "memory"); } \
        HSTAMP(u); } while (0)
#pragma unroll 1
    for (int u = cls; ; u += 6) {
        DEC_INST(u, ha, hb, ya, yb);
        if (u + 3 >= NT) break;
        DEC_INST(u + 3, hb, ha, yb, ya);
        if (u + 6 >= NT) break;
    }
    if (dw == 1) STAMP(ws, 5);
#undef DEC_INST
}

__device__ __forceinline__ void mma_wave(const Params& p, LAS unsigned char* lds, int tm, int tn, int wv, int lane, u32 seq16) {
    const u32 r = lane & 31, hh = lane >> 5, fl = (r & 7u) ^ ((r >> 3) & 3u);
    u32 ak[4];
#pragma unroll
    for (int k = 0; k < 4; ++k) ak[k] = r * 128u + (((2u * k + hh) ^ fl) << 4);
    f32x16 acc[5];
#pragma unroll
    for (int m = 0; m < 5; ++m)
#pragma unroll
        for (int e = 0; e < 16; ++e) acc[m][e] = 0.f;
    const u32 boff = LDS_B + wv * 4096;
#define CYC_FROM 24
    LAS u32* const cnt = (LAS u32*)(lds + LDS_MISC);
    u32 nfull = 0;
#pragma unroll 1
    for (int u = 0; u < NT; ++u) {
        if (rfl(nfull) < 2u * (u32)((u >> 2) + 1)) lds_wait(cnt + 16 + (u & 3), 2u * (u32)((u >> 2) + 1));
        LAS unsigned char* st = lds + (u & 3) * STAGE;
        bf16x8 af[4][5], bf[4];
#define RD(k) do { bf[k] = *(const LAS bf16x8*)(st + boff + ak[k]); _Pragma("unroll") for (int m = 0; m < 5; ++m) af[k][m] = *(const LAS bf16x8*)(st + ak[k] + m * 4096); } while (0)
#define MM(k) do { _Pragma("unroll") for (int m = 0; m < 5; ++m) acc[m] = __builtin_amdgcn_mfma_f32_32x32x16_bf16(bf[k], af[k][m], acc[m], 0, 0, 0); } while (0)
        RD(0); RD(1); __builtin_amdgcn_sched_barrier(0);
        MM(0); RD(2); __builtin_amdgcn_sched_barrier(0);
        MM(1); RD(3); nfull = __hip_atomic_load(cnt + 16 + ((u + 1) & 3), __ATOMIC_RELAXED, __HIP_MEMORY_SCOPE_WORKGROUP); __builtin_amdgcn_sched_barrier(0);
        lds_signal(cnt + 24 + (u & 3));
        MM(2); MM(3);
#undef RD
#undef MM
    }
    STAMP(p.ws[p.rank], 6);
    char* const dst = p.ws[(tn >> 1) & 7];
    const u32 cg8 = 4 * (tn & 1) + wv, r32 = 5 * tm;
    const u32 ubase = OFF_Y + (u32)p.rank * Y_SRC + (cg8 * 64 + r32) * 2048u;
    u32x4 myhdr = (u32x4){0, 0, 0, 0};
#pragma unroll
    for (int m = 0; m < 5; ++m) {
        if (r32 + m < 64) {
            u32 d[8]; float mx = 0.f;
#pragma unroll
            for (int q = 0; q < 8; ++q) {
                d[q] = cvt_pk_bf16(acc[m][2 * q], acc[m][2 * q + 1]);
                mx = fmaxf(mx, fmaxf(fabsf(acc[m][2 * q]), fabsf(acc[m][2 * q + 1])));
            }
            u32x4 L, hdr; u32x2 N;
            pack_unit(d, mx, seq16, L, N, hdr);
            char* q = dst + ubase + m * 2048u;
            if (((hdr.x >> 23) & 7u) != 7u) { st16_sys(q + lane * 16, L); st8_sys(q + 1024 + lane * 8, N); }
            else { st16_sys(q + lane * 16, (u32x4){d[0], d[1], d[2], d[3]}); st16_sys(q + 1024 + lane * 16, (u32x4){d[4], d[5], d[6], d[7]}); }
            if (lane == m) myhdr = hdr;
        }
    }
    STAMP(p.ws[p.rank], 7);
    drain();
    if (lane < 5 && r32 + lane < 64) st16_sys(dst + OFF_HY + (((u32)p.rank * 8 + cg8) * 64 + r32 + lane) * 16u, myhdr);
    STAMP(p.ws[p.rank], 8);
}

__global__ void __launch_bounds__(NTHR, 3) k_main(Params p) {
    extern __shared__ __attribute__((aligned(16))) unsigned char smem[];
    LAS unsigned char* lds = (LAS unsigned char*)smem;
    const int b = blockIdx.x, wv = rfl(threadIdx.x >> 6), lane = threadIdx.x & 63;
    char* const ws = p.ws[p.rank];
    if (wv == 0) STAMP(ws, 0);
    GAS u32* cntp = (GAS u32*)(ws + OFF_CNT + b * 128);
    const u32 callv = __hip_atomic_load(cntp, __ATOMIC_RELAXED, __HIP_MEMORY_SCOPE_AGENT);
    const int xg = b & 7, slot = b >> 3;
    if (slot < 4 && wv < 2 * PW) {
        const int gw = (wv >> 1) * 32 + slot * 8 + xg;
        if (wv & 1) pusher<true>(p, lds, wv, gw, lane, callv); else pusher<false>(p, lds, wv, gw, lane, callv);
    }
    const u32 call = rfl(callv) + 1u;
    const u32 seq16 = call % 65535u + 1u;
    Wait wt{(GAS u32*)(ws + OFF_TMO), call, false};
    if (slot < 4) {
    } else if (ROLE_OF(b) == 1) {
        const int s = slot - 4, tm = s >> 1, tn = 2 * xg + (s & 1);
        if (threadIdx.x < 16) ((LAS u32*)(lds + LDS_MISC))[16 + threadIdx.x] = 0u;
        BARRIER();
        if (wv >= 4) decoder(p, lds, tm, tn, wv - 4, lane, wt, seq16); else mma_wave(p, lds, tm, tn, wv, lane, seq16);
    }
    const __amdgpu_buffer_rsrc_t rs = __builtin_amdgcn_make_buffer_rsrc(ws, 0, (int)WS_NEED, 0x00020000);
    const u32 ri = b >> 7, rj = (b >> 6) & 1, r32 = b & 63, cg8 = wv & 7;
    const u32 snd0 = (ri << 2) | (rj << 1);
    float y[16];
    LAS u32* misc = (LAS u32*)(lds + LDS_MISC);
    if (wv < 8) {
        const u32 hoff = OFF_HY + (((snd0 + (lane & 1)) * 8 + cg8) * 64 + r32) * 16u;
        u32x4 h = __builtin_amdgcn_raw_buffer_load_b128(rs, hoff, 0, 17);
        if (!hdr_ok(h, seq16)) { const u64 t0 = rtime(); u32 n = 0; do { if (spin_on(wt, t0, n)) break; h = ld16_sync(ws, hoff); } while (!hdr_ok(h, seq16)); }
        STAMP(ws, 24 + wv);
        u32x4 l[2]; u32x2 n[2];
#pragma unroll
        for (int s = 0; s < 2; ++s) {
            const u32 ub = OFF_Y + (snd0 + s) * Y_SRC + (cg8 * 64 + r32) * 2048u;
            l[s] = __builtin_amdgcn_raw_buffer_load_b128(rs, lane * 16, ub, 17);
            n[s] = __builtin_amdgcn_raw_buffer_load_b64(rs, lane * 8 + 1024, ub, 17);
        }
#pragma unroll
        for (int s = 0; s < 2; ++s) {
            const u32 w0 = rdl(h.x, s), mode = (w0 >> 23) & 7u;
            u32 d[8];
            if (mode != 7u) {
                const u32 brep = ((w0 >> 16) & 0x7Fu) * 0x01010101u;
                const u32x4 g0 = decode_group(l[s].x, l[s].y, n[s].x, brep), g1 = decode_group(l[s].z, l[s].w, n[s].y, brep);
                d[0] = g0.x; d[1] = g0.y; d[2] = g0.z; d[3] = g0.w; d[4] = g1.x; d[5] = g1.y; d[6] = g1.z; d[7] = g1.w;
                if (mode) {
                    const u64 a = (u64)w0 | ((u64)rdl(h.y, s) << 32), bb = (u64)rdl(h.z, s) | ((u64)rdl(h.w, s) << 32);
                    for (u32 c = 0; c < mode; ++c) {
                        const u32 e = hdr_exc(a, bb, c), el = (e >> 4) & 63u, j = e & 15u, hb = e >> 11, sh = (j & 1u) * 16u + 8u;
#pragma unroll
                        for (int q = 0; q < 8; ++q) if (j >> 1 == (u32)q && (u32)lane == el) d[q] = (d[q] & ~(0xFFu << sh)) | (hb << sh);
                    }
                }
            } else {
                const u32 ub = OFF_Y + (snd0 + s) * Y_SRC + (cg8 * 64 + r32) * 2048u;
                const u32x4 hi = ld16_sync(ws, ub + lane * 16 + 1024);
                d[0] = l[s].x; d[1] = l[s].y; d[2] = l[s].z; d[3] = l[s].w; d[4] = hi.x; d[5] = hi.y; d[6] = hi.z; d[7] = hi.w;
            }
#pragma unroll
            for (int q = 0; q < 8; ++q) {
                const float lo = __uint_as_float(d[q] << 16), hi = __uint_as_float(d[q] & 0xFFFF0000u);
                if (s == 0) { y[2 * q] = lo; y[2 * q + 1] = hi; } else { y[2 * q] += lo; y[2 * q + 1] += hi; }
            }
        }
        float mx = 0.f;
#pragma unroll
        for (int q = 0; q < 16; ++q) mx = fmaxf(mx, fabsf(y[q]));
        const u32 wm = wave_max_u32(__float_as_uint(mx));
        if (lane == 0) misc[wv] = wm;
        STAMPD(ws, 16 + wv);
    } else {
#pragma unroll
        for (int q = 0; q < 16; ++q) y[q] = 0.f;
    }
    BARRIER();
    if (wv == 0) {
        const u32 v = wave_max_u32(misc[lane & 7]);
        if (lane == 0) __hip_atomic_store((GAS u64*)(ws + OFF_LMAX + b * 64), ((u64)call << 32) | v, __ATOMIC_RELAXED, __HIP_MEMORY_SCOPE_AGENT);
        STAMP(ws, 11);
        const u32 go = OFF_GMAX + ((b & 7) * 8 + (lane & 7)) * 64;
        u32x2 g = ld8_sync(ws, go);
        if (__ballot(g.y == call) != ~0ull) { const u64 t0 = rtime(); u32 n = 0; do { if (spin_on(wt, t0, n)) break; g = ld8_sync(ws, go); } while (__ballot(g.y == call) != ~0ull); }
        const u32 gm = wave_max_u32(g.x);
        if (lane == 0) misc[8] = gm;
    } else if (wv == 8 && b == LEADER) {
        u32x2 g0, g1, g2, g3;
        auto poll = [&]() {
            asm volatile("global_load_dwordx2 %0, %4, %5 sc1\n\tglobal_load_dwordx2 %1, %4, %5 offset:64 sc1\n\tglobal_load_dwordx2 %2, %4, %5 offset:128 sc1\n\tglobal_load_dwordx2 %3, %4, %5 offset:192 sc1\n\ts_waitcnt vmcnt(0)"
                         : "=&v"(g0), "=&v"(g1), "=&v"(g2), "=&v"(g3) : "v"(OFF_LMAX + lane * 256), "s"(ws) : "memory");
            return __ballot(g0.y == call && g1.y == call && g2.y == call && g3.y == call) == ~0ull; };
        if (!poll()) { const u64 t0 = rtime(); u32 n = 0; do { if (spin_on(wt, t0, n)) break; } while (!poll()); }
        const u32 lm = wave_max_u32(max(max(g0.x, g1.x), max(g2.x, g3.x)));
        __hip_atomic_store((GAS u64*)(p.ws[lane >> 3] + OFF_GMAX + ((lane & 7) * 8 + (u32)p.rank) * 64), ((u64)call << 32) | lm, __ATOMIC_RELAXED, __HIP_MEMORY_SCOPE_SYSTEM);
        STAMP(ws, 14);
    }
    BARRIER();
    if (wv == 0) STAMP(ws, 12);
    if (wv < 8) {
        const float amax = __uint_as_float(misc[8]);
        const bool bad = rfl(__hip_atomic_load(wt.tmo, __ATOMIC_RELAXED, __HIP_MEMORY_SCOPE_AGENT)) == call;
        const float inv = amax > 0.f ? 448.0f / amax : 0.f;
        u32 o[4];
#pragma unroll
        for (int q = 0; q < 4; ++q) {
            float v[4];
#pragma unroll
            for (int e = 0; e < 4; ++e) v[e] = __builtin_amdgcn_fmed3f(y[4 * q + e] * inv, -448.f, 448.f);
            int t = __builtin_amdgcn_cvt_pk_fp8_f32(v[0], v[1], 0, false); t = __builtin_amdgcn_cvt_pk_fp8_f32(v[2], v[3], t, true); o[q] = (u32)t;
        }
        const u32 R = r32 * 32 + (lane & 31), row = 512 * (R >> 8) + 256 * ri + (R & 255), col = 256 * rj + 32 * cg8 + 16 * (lane >> 5);
        const u32x4 ov = bad ? (u32x4){0x7f7f7f7fu, 0x7f7f7f7fu, 0x7f7f7f7fu, 0x7f7f7f7fu} : (u32x4){o[0], o[1], o[2], o[3]};
        *(GAS u32x4*)(p.out + (size_t)row * SH + col) = ov;
    }
    if (threadIdx.x == 0) __hip_atomic_store(cntp, call, __ATOMIC_RELAXED, __HIP_MEMORY_SCOPE_AGENT);
    if (wv == 0) STAMP(ws, 13);
}

extern "C" void kernel_launch(int rank, int nranks, ncclComm_t comm, hipStream_t stream,
                              void* const* d_in, const int* in_sizes, int n_in,
                              void* d_out, int out_size, const int* in_dtypes, int out_dtype,
                              void* const* d_out_all, void* const* d_ws_all, size_t ws_bytes) {
    if (nranks != NR || (unsigned)rank >= (unsigned)NR || ws_bytes < WS_NEED || n_in < 2 || in_sizes[0] != SH * KG || in_sizes[1] != KG * SH || out_size != MG * SH) {
        fprintf(stderr, "kernel_launch: unexpected shapes\n"); return; }
    HIP_CHECK(hipFuncSetAttribute((const void*)k_main, hipFuncAttributeMaxDynamicSharedMemorySize, LDS_BYTES));
    Params p;
    p.x = (const float*)d_in[0]; p.w = (const float*)d_in[1]; p.out = (unsigned char*)d_out;
    for (int i = 0; i < NR; ++i) p.ws[i] = (char*)d_ws_all[i];
    p.rank = rank; p.pad = 0;
    k_main<<<256, NTHR, LDS_BYTES, stream>>>(p);
    HIP_CHECK(hipGetLastError());
}
```

```cpp
#include <hip/hip_runtime.h>
#include <rccl/rccl.h>
#include <cstdio>
#include <cstdint>

#define HIP_CHECK(expr) do { hipError_t _e = (expr); if (_e != hipSuccess) { \
    fprintf(stderr, "HIP error %s at %s:%d\n", hipGetErrorString(_e), __FILE__, __LINE__); \
    return; } } while (0)

typedef unsigned u32;
typedef unsigned long long u64;
typedef u32 u32x4 __attribute__((ext_vector_type(4)));
typedef u32 u32x2 __attribute__((ext_vector_type(2)));
typedef float f32x4 __attribute__((ext_vector_type(4)));
typedef float f32x16 __attribute__((ext_vector_type(16)));
typedef short bf16x8 __attribute__((ext_vector_type(8)));
#define LAS __attribute__((address_space(3)))
#define GAS __attribute__((address_space(1)))

constexpr int NR = 8, MG = 4096, KG = 4096, SH = 512;
constexpr int NT = 32;
constexpr int NTHR = 512;
constexpr u64 DEADLINE = 150000000ull;
constexpr u32 PKT = 3072;
constexpr u32 OFF_CNT = 0;
constexpr u32 OFF_TMO = 0x8000;
constexpr u32 OFF_TICK = 0x8100;
constexpr u32 OFF_LMAX = 0x10000;
constexpr u32 OFF_GMAX = 0x14000;
constexpr u32 OFF_STAMP = 0x20000;
constexpr u32 OFF_HA = 0x30000;
constexpr u32 OFF_HB = 0x40000;
constexpr u32 OFF_HY = 0x50000;
constexpr u32 OFF_IN = 2u << 20;
constexpr u32 IN_SRC = 2u << 20, IN_B = 768u << 10, IN_T = 8 * PKT;
constexpr u32 OFF_Y = 18u << 20;
constexpr u32 Y_SRC = 1u << 20;
constexpr u32 OFF_OVF = 26u << 20;
constexpr size_t WS_NEED = 34u << 20;
constexpr int STAGE = 20480, NSTAGE = 6, LDS_MISC = NSTAGE * STAGE, LDS_BYTES = 124 * 1024;
constexpr int LEADER = 248;

struct Params { const float* x; const float* w; unsigned char* out; char* ws[NR]; int rank; int pad; };

__device__ __forceinline__ u32 cvt_pk_bf16(float lo, float hi) { u32 r; asm("v_cvt_pk_bf16_f32 %0, %1, %2" : "=v"(r) : "v"(lo), "v"(hi)); return r; }
__device__ __forceinline__ void st16_sys(void* p, u32x4 v) { asm volatile("global_store_dwordx4 %0, %1, off sc0 sc1\n\ts_nop 1" :: "v"(p), "v"(v) : "memory"); }
__device__ __forceinline__ void st8_sys(void* p, u32x2 v) { asm volatile("global_store_dwordx2 %0, %1, off sc0 sc1\n\ts_nop 1" :: "v"(p), "v"(v) : "memory"); }
__device__ __forceinline__ u32x4 ld16_sync(const char* base, u32 off) { u32x4 r; asm volatile("global_load_dwordx4 %0, %1, %2 sc0 sc1\n\ts_waitcnt vmcnt(0)" : "=&v"(r) : "v"(off), "s"(base) : "memory"); return r; }
__device__ __forceinline__ u32x2 ld8_sync(const char* base, u32 off) { u32x2 r; asm volatile("global_load_dwordx2 %0, %1, %2 sc0 sc1\n\ts_waitcnt vmcnt(0)" : "=&v"(r) : "v"(off), "s"(base) : "memory"); return r; }
__device__ __forceinline__ u32 ld4_sync_agent(const void* p) { u32 r; asm volatile("global_load_dword %0, %1, off sc1\n\ts_waitcnt vmcnt(0)" : "=&v"(r) : "v"(p) : "memory"); return r; }
__device__ __forceinline__ void drain() { asm volatile("s_waitcnt vmcnt(0)" ::: "memory"); }
__device__ __forceinline__ u64 rtime() { return __builtin_amdgcn_s_memrealtime(); }
__device__ __forceinline__ u32 rfl(u32 v) { return __builtin_amdgcn_readfirstlane(v); }
__device__ __forceinline__ u32 rdl(u32 v, int l) { return __builtin_amdgcn_readlane(v, l); }
__device__ __forceinline__ u32 wave_max_u32(u32 v) {
    v = max(v, (u32)__builtin_amdgcn_update_dpp(0, (int)v, 0x111, 0xf, 0xf, false));
    v = max(v, (u32)__builtin_amdgcn_update_dpp(0, (int)v, 0x112, 0xf, 0xf, false));
    v = max(v, (u32)__builtin_amdgcn_update_dpp(0, (int)v, 0x114, 0xf, 0xf, false));
    v = max(v, (u32)__builtin_amdgcn_update_dpp(0, (int)v, 0x118, 0xf, 0xf, false));
    v = max(v, (u32)__builtin_amdgcn_update_dpp(0, (int)v, 0x142, 0xa, 0xf, false));
    v = max(v, (u32)__builtin_amdgcn_update_dpp(0, (int)v, 0x143, 0xc, 0xf, false));
    return rdl(v, 63);
}
#define BARRIER() do { asm volatile("s_waitcnt lgkmcnt(0)" ::: "memory"); __builtin_amdgcn_s_barrier(); asm volatile("" ::: "memory"); } while (0)

#define ROLE_OF(b) (((b) >> 3) < 4 ? 0 : ((b) >> 3) < 30 ? 1 : 2)
#define STAMP(ws, slot) do {} while (0)
#define STAMPD(ws, slot) do {} while (0)

#define CT() 0ull
#define CACC(a, t0, t1) do {} while (0)

__device__ __forceinline__ void pack_unit(const u32* d, float amax_lane, u32 seq16, u32x4& L, u32x2& N, u32x4& hdr) {
    u32 H[4];
#pragma unroll
    for (int q = 0; q < 4; ++q) { L[q] = __builtin_amdgcn_perm(d[2 * q + 1], d[2 * q], 0x06040200u); H[q] = __builtin_amdgcn_perm(d[2 * q + 1], d[2 * q], 0x07050301u); }
    const u32 mb = wave_max_u32(__float_as_uint(amax_lane));
    u32 h7 = (mb + 0x7FFFu + ((mb >> 16) & 1u)) >> 24;
    h7 = h7 > 127u ? 127u : h7;
    const u32 base = h7 > 7u ? h7 - 7u : 0u, brep = base * 0x01010101u;
    u32 b[4], X[4];
#pragma unroll
    for (int q = 0; q < 4; ++q) { const u32 t = (H[q] | 0x80808080u) - brep; b[q] = (t ^ 0x80808080u) & 0xF8F8F8F8u; X[q] = (H[q] & 0x80808080u) | (t & 0x07070707u); }
    N[0] = X[0] | __builtin_amdgcn_alignbit(X[1], X[1], 28); N[1] = X[2] | __builtin_amdgcn_alignbit(X[3], X[3], 28);
    u32 cnt = 0; u64 ea = 0, eb = 0;
    u64 m = __ballot((b[0] | b[1] | b[2] | b[3]) != 0);
    if (m) {
        u32 bm = 0;
#pragma unroll
        for (int q = 0; q < 4; ++q) bm |= (((((((b[q] & 0x7F7F7F7Fu) + 0x7F7F7F7Fu) | b[q]) & 0x80808080u) >> 7) * 0x01020408u) >> 24 & 0xFu) << (4 * q);
        while (m && cnt <= 4) {
            const int l = __builtin_ctzll(m); m &= m - 1;
            u32 mk = rdl(bm, l);
            const u32 h0 = rdl(H[0], l), h1 = rdl(H[1], l), h2 = rdl(H[2], l), h3 = rdl(H[3], l);
            while (mk) {
                const u32 j = (u32)__builtin_ctz(mk); mk &= mk - 1;
                const u32 hq = (j & 8u) ? ((j & 4u) ? h3 : h2) : ((j & 4u) ? h1 : h0);
                const u64 e = (u64)((u32)l * 16u + j) | ((u64)((hq >> (8u * (j & 3u))) & 0xFFu) << 11);
                if (cnt < 2) ea |= e << (26u + 19u * cnt); else if (cnt < 4) eb |= e << (16u + 19u * (cnt - 2u));
                ++cnt;
            }
        }
    }
    const u32 mode = cnt > 4 ? 7u : cnt;
    ea |= (u64)seq16 | ((u64)base << 16) | ((u64)mode << 23); eb |= (u64)seq16;
    hdr = (u32x4){(u32)ea, (u32)(ea >> 32), (u32)eb, (u32)(eb >> 32)};
}
__device__ __forceinline__ u32x4 decode_group(u32 LA, u32 LB, u32 n, u32 brep) {
    const u32 HA = (n & 0x87878787u) + brep, HB = (__builtin_amdgcn_alignbit(n, n, 4) & 0x87878787u) + brep;
    return (u32x4){__builtin_amdgcn_perm(HA, LA, 0x05010400u), __builtin_amdgcn_perm(HA, LA, 0x07030602u),
                   __builtin_amdgcn_perm(HB, LB, 0x05010400u), __builtin_amdgcn_perm(HB, LB, 0x07030602u)};
}
__device__ __forceinline__ u32 hdr_exc(u64 a, u64 b, u32 c) { return (u32)((c < 2 ? a >> (26u + 19u * c) : b >> (16u + 19u * (c - 2u)))) & 0x7FFFFu; }
__device__ __forceinline__ bool hdr_ok(const u32x4& h, u32 seq16) { return __ballot(((h.x & 0xFFFFu) == seq16) && ((h.z & 0xFFFFu) == seq16)) == ~0ull; }
__device__ __forceinline__ void patch_regs(u32* d, const u32x4& h, int hl, u32 mode, int lane) {
    const u64 a = (u64)rdl(h.x, hl) | ((u64)rdl(h.y, hl) << 32), b = (u64)rdl(h.z, hl) | ((u64)rdl(h.w, hl) << 32);
    for (u32 c = 0; c < mode; ++c) {
        const u32 e = hdr_exc(a, b, c), el = (e >> 4) & 63u, j = e & 15u, hb = e >> 11, sh = (j & 1u) * 16u + 8u;
#pragma unroll
        for (int q = 0; q < 8; ++q) if (j >> 1 == (u32)q && (u32)lane == el) d[q] = (d[q] & ~(0xFFu << sh)) | (hb << sh);
    }
}

__device__ __forceinline__ void lds_signal(LAS u32* c) {
    asm volatile("" ::: "memory");
    if ((threadIdx.x & 63) == 0) __hip_atomic_fetch_add(c, 1u, __ATOMIC_RELAXED, __HIP_MEMORY_SCOPE_WORKGROUP);
    asm volatile("" ::: "memory");
}
__device__ __forceinline__ void lds_wait(LAS u32* c, u32 target) {
    while (rfl(__hip_atomic_load(c, __ATOMIC_RELAXED, __HIP_MEMORY_SCOPE_WORKGROUP)) < target) __builtin_amdgcn_s_sleep(1);
    asm volatile("" ::: "memory");
}

struct Wait { GAS u32* tmo; u32 call; bool dead; };
__device__ __forceinline__ bool spin_on(Wait& wt, u64 t0, u32& n) {
    if (wt.dead) return true;
    if ((++n & 15u) == 0u) {
        if (rfl(ld4_sync_agent((const void*)wt.tmo)) == wt.call || rtime() - t0 > DEADLINE) {
            __hip_atomic_store(wt.tmo, wt.call, __ATOMIC_RELAXED, __HIP_MEMORY_SCOPE_AGENT); wt.dead = true; return true; }
    }
    __builtin_amdgcn_s_sleep(1);
    return false;
}
__device__ __forceinline__ void wait_hdr(u32x4& h, const char* ws, u32 off, u32 seq16, Wait& wt) {
    if (!hdr_ok(h, seq16)) { const u64 t0 = rtime(); u32 n = 0; do { if (spin_on(wt, t0, n)) break; h = ld16_sync(ws, off); } while (!hdr_ok(h, seq16)); }
}
#define HDR_READY(h) hdr_ok(h, seq16)

#define PW 2
#define SKEW 10
#define WLEAD 4
constexpr u32 PWAVES = 32 * PW;
struct Pkt { u32 isw, t, h, kh, blk; };
__device__ __forceinline__ Pkt ticket_packet(u32 n, u32 pool) {
    const u32 tau = min(2u * n + pool, 2047u);
    constexpr u32 P0 = 32u * WLEAD, P1 = P0 + 48u * SKEW, P2 = P1 + 64u * (32u - WLEAD - SKEW), P3 = P2 + 32u * WLEAD;
    static_assert(WLEAD + SKEW <= 32 && P3 + 16u * SKEW == 2048u, "slot layout");
    int j; u32 c;
    if (tau < P0) { j = (int)(tau >> 5) - WLEAD; c = tau & 31u; }
    else if (tau < P1) { const u32 q = ((tau - P0) * 1366u) >> 16; j = (int)q; c = tau - P0 - 48u * q; }
    else if (tau < P2) { j = SKEW + (int)((tau - P1) >> 6); c = (tau - P1) & 63u; }
    else if (tau < P3) { j = 32 - WLEAD + (int)((tau - P2) >> 5); c = 32u + ((tau - P2) & 31u); }
    else { j = 32 + (int)((tau - P3) >> 4); c = 48u + ((tau - P3) & 15u); }
    Pkt k;
    k.isw = c < 32u;
    k.t = (u32)(c < 32u ? j + WLEAD : c < 48u ? j : j - SKEW);
    const u32 q4 = c < 32u ? c >> 3 : (c >> 2) & 3u;
    k.h = q4 >> 1; k.kh = q4 & 1u;
    k.blk = c < 32u ? (c & 7u) : (c & 3u) + (c >= 48u ? 4u : 0u);
    return k;
}
__device__ __forceinline__ void pusher(const Params& p, LAS unsigned char* lds, int wvi, int gw, int lane, u32 callv) {
    const u32 pool = wvi & 1;
    const __amdgpu_buffer_rsrc_t rt = __builtin_amdgcn_make_buffer_rsrc(p.ws[p.rank] + OFF_TICK + pool * 128, 0, 4, 0x00020000);
    const u32 loffw = (u32)((lane >> 3) * SH + 4 * (lane & 7)) * 4u, loffx = (u32)((lane & 15) * KG + 8 * (lane >> 4)) * 4u;
    const u32 lbase = (u32)(uintptr_t)lds + (u32)wvi * 16384u;
    auto ld = [&](u32 buf, u32 n) {
        const Pkt k = ticket_packet(n, pool);
        const u32 l0 = lbase + buf * 8192u;
        u32 keep;
#define GL(i, off) "s_mov_b32 m0, %" #i "\n\ts_nop 0\n\tglobal_load_lds_dwordx4 " off "\n\t"
#define GL8(sb, o0, o1, o2, o3, o4, o5, o6, o7) asm volatile("s_mov_b32 %0, m0\n\t" \
                         GL(6, "%1, %5") GL(7, "%2, %5") GL(8, "%3, %5") GL(9, "%4, %5") GL(10, "%14, %5") GL(11, "%15, %5") GL(12, "%16, %5") GL(13, "%17, %5") \
                         "s_mov_b32 m0, %0" : "=&s"(keep) : "v"(o0), "v"(o1), "v"(o2), "v"(o3), "s"(sb), \
                           "s"(l0), "s"(l0 + 1024u), "s"(l0 + 2048u), "s"(l0 + 3072u), "s"(l0 + 4096u), "s"(l0 + 5120u), "s"(l0 + 6144u), "s"(l0 + 7168u), \
                           "v"(o4), "v"(o5), "v"(o6), "v"(o7) : "memory")
        if (k.isw) {
            const GAS char* const sb = (const GAS char*)p.w + ((2048u * k.kh + 64u * k.t) * SH + 256u * k.h + 32u * k.blk) * 4u;
            GL8(sb, loffw, loffw + 16384u, loffw + 32768u, loffw + 49152u, loffw + 65536u, loffw + 81920u, loffw + 98304u, loffw + 114688u);
        } else {
            const GAS char* const sb = (const GAS char*)p.x + ((256u * k.h + 32u * k.blk) * KG + 2048u * k.kh + 64u * k.t) * 4u;
            constexpr u32 R16 = 16u * KG * 4u;
            GL8(sb, loffx, loffx + 16u, loffx + 128u, loffx + 144u, loffx + R16, loffx + R16 + 16u, loffx + R16 + 128u, loffx + R16 + 144u);
        }
#undef GL8
#undef GL
    };
    u32 ta = gw, tb = gw + PWAVES, tc = gw + 2 * PWAVES;
    ld(0, ta); ld(1, tb);
    asm volatile("s_waitcnt vmcnt(0)" ::: "memory");
    const u32 call = rfl(callv) + 1u, seq16 = call % 65535u + 1u;
    const u32 tbase = (call - 1u) * 1024u - 3u * PWAVES;
    const u32 wcol = (16u * ((lane >> 2) & 1u) + 4u * ((lane & 31) >> 3) + (lane & 3u)) * 4u + (lane >> 5) * 1024u;
    u32x4 ph = (u32x4){0, 0, 0, 0}; char* pa = nullptr; bool have = false; u32 drawn = 0;
    auto step = [&](u32 buf, u32& n) {
        const Pkt k = ticket_packet(n, pool);
        LAS unsigned char* const lb = lds + wvi * 16384 + buf * 8192;
        asm volatile("s_waitcnt vmcnt(14)" ::: "memory");
        u32 d[16]; float mx[2] = {0.f, 0.f};
        if (k.isw) {
#pragma unroll
            for (int g = 0; g < 4; ++g)
#pragma unroll
                for (int q = 0; q < 4; ++q) {
                    const float lo = *(const LAS float*)(lb + wcol + (16 * g + 2 * q) * 128), hi = *(const LAS float*)(lb + wcol + (16 * g + 2 * q + 1) * 128);
                    d[g * 4 + q] = cvt_pk_bf16(lo, hi); mx[g >> 1] = fmaxf(mx[g >> 1], fmaxf(fabsf(lo), fabsf(hi)));
                }
        } else {
#pragma unroll
            for (int li = 0; li < 8; ++li) {
                const f32x4 v = *(const LAS f32x4*)(lb + li * 1024 + lane * 16);
                d[2 * li] = cvt_pk_bf16(v[0], v[1]); d[2 * li + 1] = cvt_pk_bf16(v[2], v[3]);
                mx[li >> 2] = fmaxf(fmaxf(mx[li >> 2], fmaxf(fabsf(v[0]), fabsf(v[1]))), fmaxf(fabsf(v[2]), fabsf(v[3])));
            }
        }
        u32 d0 = k.isw ? ((k.h << 1) | k.kh) : ((k.h << 2) | k.kh), d1 = d0 | (k.isw ? 4u : 2u);
        if ((n ^ (n >> 5)) & 1u) { const u32 x = d0; d0 = d1; d1 = x; }
        char* const w0 = p.ws[d0 & 7u]; char* const w1 = p.ws[d1 & 7u];
        const u32 po = OFF_IN + (u32)p.rank * IN_SRC + (k.isw ? IN_B : 0u) + k.t * IN_T + k.blk * PKT + lane * 16u;
        u32x4 hd[2], L[2]; u32x2 N[2];
        pack_unit(d, mx[0], seq16, L[0], N[0], hd[0]);
        pack_unit(d + 8, mx[1], seq16, L[1], N[1], hd[1]);
        const u32x4 NN = (u32x4){N[0].x, N[0].y, N[1].x, N[1].y};
        asm volatile("" :: "v"(NN), "v"(L[0]), "v"(L[1]));
        drain();
        const u32 dv = rfl(drawn);
        if (have) { tc = dv - tbase; if (lane < 4) st16_sys(pa, ph); }
        drawn = (u32)__builtin_amdgcn_raw_ptr_buffer_atomic_add_i32(1, rt, lane == 0 ? 0 : 0x7FFFFFF0, 0, 0);
#pragma unroll
        for (int u = 0; u < 2; ++u)
            if (((hd[u].x >> 23) & 7u) == 7u) {
                const u32 ov = OFF_OVF + ((u32)p.rank * 2u + k.isw) * (512u << 10) + k.t * 16384u + (2u * k.blk + u) * 1024u + lane * 16u;
                const u32x4 r1 = (u32x4){d[8 * u + 4], d[8 * u + 5], d[8 * u + 6], d[8 * u + 7]};
                L[u] = (u32x4){d[8 * u], d[8 * u + 1], d[8 * u + 2], d[8 * u + 3]};
                st16_sys(w0 + ov, r1); st16_sys(w1 + ov, r1);
            }
        st16_sys(w0 + po, L[0]); st16_sys(w1 + po, L[0]); st16_sys(w0 + po + 1024, L[1]); st16_sys(w1 + po + 1024, L[1]); st16_sys(w0 + po + 2048, NN); st16_sys(w1 + po + 2048, NN);
        asm volatile("s_waitcnt lgkmcnt(0)" ::: "memory");
        ld(buf, tc); n = tc;
        const u32 hi = k.isw ? (u32)p.rank * 16u + 2u * k.blk : (k.blk >> 2) * 64u + (u32)p.rank * 8u + (k.blk & 3u) * 2u;
        pa = ((lane & 2) ? w1 : w0) + (k.isw ? OFF_HB : OFF_HA) + k.t * 2048u + (hi + (lane & 1)) * 16u;
        ph = (lane & 1) ? hd[1] : hd[0]; have = true;
    };
#pragma unroll 1
    while (true) {
        if (ta >= 1024u) break;
        step(0, ta);
        if (tb >= 1024u) break;
        step(1, tb);
    }
    drain();
    asm volatile("" :: "v"(drawn));
    if (lane < 4) st16_sys(pa, ph);
    STAMP(p.ws[p.rank], 15);
}

#define LW(p, v) *(LAS u32x4*)(p) = (v)
__device__ __forceinline__ void decoder(const Params& p, LAS unsigned char* lds, int tm, int dw, int lane, Wait& wt, u32 seq16) {
    char* const ws = p.ws[p.rank];
    LAS u32* const cnt = (LAS u32*)(lds + LDS_MISC);
    const __amdgpu_buffer_rsrc_t rs = __builtin_amdgcn_make_buffer_rsrc(ws, 0, (int)WS_NEED, 0x00020000);
    const u32 fr = lane & 15, fq = lane >> 4;
    auto uaddr = [&](u32 v, u32 g, u32 efr, u32 efq) -> u32 { const u32 row = 16u * v + efr; return row * 128u + (((4u * g + efq) ^ (row & 7u) ^ ((row >> 3) & 3u)) << 4); };
    auto mtile = [&](u32 v) -> u32 { return min(10u * tm + v, 127u); };
    u32 po[5];
#pragma unroll
    for (int s = 0; s < 5; ++s) { const u32 x = mtile(2 * s); po[s] = OFF_IN + ((x >> 3) & 7u) * IN_SRC + (4u * (x >> 6) + ((x & 7u) >> 1)) * PKT; }
    const u32 ae0 = uaddr(0, 0, fr, fq), ae1 = uaddr(0, 1, fr, fq), ao0 = uaddr(1, 0, fr, fq) - 2048u, ao1 = uaddr(1, 1, fr, fq) - 2048u;
    const u32 hv = min((u32)lane >> 2, 9u);
    const u32 hoff = OFF_HA + mtile(hv) * 16u;
    struct PayA { u32x4 l0[5], l1[5], n[5]; };
    auto ldh = [&](int t) -> u32x4 { return __builtin_amdgcn_raw_buffer_load_b128(rs, hoff, t * 2048, 17); };
    auto ldp = [&](PayA& y, int t) {
        const u32 so = t * IN_T;
#pragma unroll
        for (int s = 0; s < 5; ++s) {
            y.l0[s] = __builtin_amdgcn_raw_buffer_load_b128(rs, lane * 16, po[s] + so, 17);
            y.l1[s] = __builtin_amdgcn_raw_buffer_load_b128(rs, lane * 16 + 1024, po[s] + so, 17);
            y.n[s] = __builtin_amdgcn_raw_buffer_load_b128(rs, lane * 16 + 2048, po[s] + so, 17);
        }
    };
    u64 c_free = 0, c_dec = 0, c_iss = 0;
    auto dec = [&](const PayA& y, const u32x4& h, int t) {
        const u32 sg = (u32)t % NSTAGE, turn = (u32)t / NSTAGE;
        LAS unsigned char* st = lds + sg * STAGE;
        const u64 q0 = CT();
        if (turn) lds_wait(cnt + 24 + sg, 4u * turn);
        const u64 q1 = CT(); CACC(c_free, q0, q1);
        const u32 mode = (h.x >> 23) & 7u, brl = ((h.x >> 16) & 0x7Fu) * 0x01010101u;
        if (__ballot(mode == 7u) == 0ull) {
#pragma unroll
            for (int s = 0; s < 5; ++s) {
                const u32 b0 = rdl(brl, 8 * s), b1 = rdl(brl, 8 * s + 4);
                LW(st + ae0 + s * 4096, decode_group(y.l0[s].x, y.l0[s].y, y.n[s].x, b0)); LW(st + ae1 + s * 4096, decode_group(y.l0[s].z, y.l0[s].w, y.n[s].y, b0));
                LW(st + ao0 + s * 4096 + 2048, decode_group(y.l1[s].x, y.l1[s].y, y.n[s].z, b1)); LW(st + ao1 + s * 4096 + 2048, decode_group(y.l1[s].z, y.l1[s].w, y.n[s].w, b1));
            }
        } else {
#pragma unroll
            for (int v = 0; v < 10; ++v) {
                const u32 x = mtile(v), br = rdl(brl, 4 * v);
                const u32x4 l = (v & 1) ? y.l1[v >> 1] : y.l0[v >> 1]; const u32 n0 = (v & 1) ? y.n[v >> 1].z : y.n[v >> 1].x, n1 = (v & 1) ? y.n[v >> 1].w : y.n[v >> 1].y;
                const u32x4 rw = ld16_sync(ws, OFF_OVF + ((x >> 3) & 7u) * (1024u << 10) + (8u * (x >> 6) + (x & 7u)) * 1024u + t * 16384 + lane * 16);
                const bool raw = rdl(mode, 4 * v) == 7u;
                *(LAS u32x4*)(st + ((v & 1) ? ao0 : ae0) + v * 2048) = raw ? l : decode_group(l.x, l.y, n0, br);
                *(LAS u32x4*)(st + ((v & 1) ? ao1 : ae1) + v * 2048) = raw ? rw : decode_group(l.z, l.w, n1, br);
            }
        }
        if (__ballot(mode - 1u < 4u) != 0ull) {
            const u32 c = lane & 3;
            const u64 a = (u64)h.x | ((u64)h.y << 32), b = (u64)h.z | ((u64)h.w << 32);
            const u32 e = (u32)((c < 2 ? a : b) >> ((c < 2 ? 26u : 16u) + 19u * (c & 1u))) & 0x7FFFFu;
            const u32 l = (e >> 4) & 63u;
            const u32 adr = uaddr(hv, (e >> 3) & 1u, l & 15u, l >> 4) + 2u * (e & 7u) + 1u;
            if (lane < 40 && mode != 7u && c < mode) st[adr] = (unsigned char)(e >> 11);
        }
        lds_signal(cnt + 16 + sg);
        const u64 q2 = CT(); CACC(c_dec, q1, q2);
    };
    PayA ya, yb; u32x4 ha, hb, sn;
    ha = ldh(dw);
    wait_hdr(ha, ws, hoff + dw * 2048, seq16, wt); if (dw == 0) STAMP(ws, 1);
    sn = ldh(dw + 4); ldp(ya, dw);
#define DEC_INST(u, hc, hn, yc, yn) do { \
        const int t4 = min((u) + 4, NT - 1), t8 = min((u) + 8, NT - 1); \
        if (HDR_READY(sn)) { const u64 i0 = CT(); hn = sn; sn = ldh(t8); ldp(yn, t4); const u64 i1 = CT(); CACC(c_iss, i0, i1); dec(yc, hc, (u)); asm volatile("; early" ::: "memory"); } \
        else { dec(yc, hc, (u)); wait_hdr(sn, ws, hoff + t4 * 2048, seq16, wt); hn = sn; sn = ldh(t8); ldp(yn, t4); asm volatile("; late" ::: "memory"); } } while (0)
#pragma unroll 1
    for (int u = dw; u < NT; u += 8) {
        DEC_INST(u, ha, hb, ya, yb);
        DEC_INST(u + 4, hb, ha, yb, ya);
    }
#undef DEC_INST
    if (dw == 3) STAMP(ws, 5);
}

__device__ __forceinline__ void mma_wave(const Params& p, LAS unsigned char* lds, int tm, int tn, int wv, int lane, Wait& wt, u32 seq16) {
    char* const ws = p.ws[p.rank];
    const __amdgpu_buffer_rsrc_t rs = __builtin_amdgcn_make_buffer_rsrc(ws, 0, (int)WS_NEED, 0x00020000);
    const u32 r = lane & 31, hh = lane >> 5, fl = (r & 7u) ^ ((r >> 3) & 3u);
    u32 ak[4];
#pragma unroll
    for (int k = 0; k < 4; ++k) ak[k] = r * 128u + (((2u * k + hh) ^ fl) << 4);
    f32x16 acc[5];
#pragma unroll
    for (int m = 0; m < 5; ++m)
#pragma unroll
        for (int e = 0; e < 16; ++e) acc[m][e] = 0.f;
    LAS u32* const cnt = (LAS u32*)(lds + LDS_MISC);
    const u32 src = tn >> 1, slab = 4 * (tn & 1) + wv;
    const u32 pB = OFF_IN + src * IN_SRC + IN_B + slab * PKT, oB = OFF_OVF + src * (1024u << 10) + (512u << 10) + 2u * slab * 1024u + lane * 16u;
    const u32 hoff = OFF_HB + (16u * src + 2u * slab + min((u32)lane >> 2, 1u)) * 16u;
    struct PayB { u32x4 l0, l1, n; };
    auto ldh = [&](int t) -> u32x4 { return __builtin_amdgcn_raw_buffer_load_b128(rs, hoff, t * 2048, 17); };
    auto ldp = [&](PayB& y, int t) {
        y.l0 = __builtin_amdgcn_raw_buffer_load_b128(rs, lane * 16, pB + t * IN_T, 17);
        y.l1 = __builtin_amdgcn_raw_buffer_load_b128(rs, lane * 16 + 1024, pB + t * IN_T, 17);
        y.n = __builtin_amdgcn_raw_buffer_load_b128(rs, lane * 16 + 2048, pB + t * IN_T, 17);
    };
    u32 nfull = 0;
    u64 c_b = 0, c_full = 0, c_mm = 0, c_iss = 0;
    auto ungrp = [&](u32 (&d)[16], const PayB& y, int g, u32 b0, u32 b1) {
        const u32x4 r = g == 0 ? decode_group(y.l0.x, y.l0.y, y.n.x, b0) : g == 1 ? decode_group(y.l0.z, y.l0.w, y.n.y, b0) : g == 2 ? decode_group(y.l1.x, y.l1.y, y.n.z, b1) : decode_group(y.l1.z, y.l1.w, y.n.w, b1);
        d[4 * g] = r.x; d[4 * g + 1] = r.y; d[4 * g + 2] = r.z; d[4 * g + 3] = r.w;
    };
    auto fix = [&](u32 (&d)[16], const PayB& y, const u32x4& h, u32 m0, u32 m1, int u) {
        if (m0 == 7u) { const u32x4 rw = ld16_sync(ws, oB + u * 16384); d[0] = y.l0.x; d[1] = y.l0.y; d[2] = y.l0.z; d[3] = y.l0.w; d[4] = rw.x; d[5] = rw.y; d[6] = rw.z; d[7] = rw.w; }
        else if (m0) patch_regs(d, h, 0, m0, lane);
        if (m1 == 7u) { const u32x4 rw = ld16_sync(ws, oB + u * 16384 + 1024); d[8] = y.l1.x; d[9] = y.l1.y; d[10] = y.l1.z; d[11] = y.l1.w; d[12] = rw.x; d[13] = rw.y; d[14] = rw.z; d[15] = rw.w; }
        else if (m1) patch_regs(d + 8, h, 4, m1, lane);
    };
    bf16x8 a0[5], a1[5];
#define SB() __builtin_amdgcn_sched_barrier(0)
#define RD(dst, st, k) do { _Pragma("unroll") for (int m = 0; m < 5; ++m) dst[m] = *(const LAS bf16x8*)((st) + ak[k] + m * 4096); SB(); } while (0)
#define M1(d, k, src, m) do { acc[m] = __builtin_amdgcn_mfma_f32_32x32x16_bf16(__builtin_bit_cast(bf16x8, (u32x4){d[4 * k], d[4 * k + 1], d[4 * k + 2], d[4 * k + 3]}), src[m], acc[m], 0, 0, 0); SB(); } while (0)
#define UH(g, pp, b0, b1) do { const u32 n_ = g == 0 ? pp.n.x : g == 1 ? pp.n.y : g == 2 ? pp.n.z : pp.n.w, br_ = g < 2 ? b0 : b1; \
                              uha = (n_ & 0x87878787u) + br_; uhb = (__builtin_amdgcn_alignbit(n_, n_, 4) & 0x87878787u) + br_; asm volatile("" : "+v"(uha), "+v"(uhb)); SB(); } while (0)
#define UP(d, g, pp) do { const u32 la_ = g == 0 ? pp.l0.x : g == 1 ? pp.l0.z : g == 2 ? pp.l1.x : pp.l1.z, lb_ = g == 0 ? pp.l0.y : g == 1 ? pp.l0.w : g == 2 ? pp.l1.y : pp.l1.w; \
                         d[4 * g] = __builtin_amdgcn_perm(uha, la_, 0x05010400u); d[4 * g + 1] = __builtin_amdgcn_perm(uha, la_, 0x07030602u); \
                         d[4 * g + 2] = __builtin_amdgcn_perm(uhb, lb_, 0x05010400u); d[4 * g + 3] = __builtin_amdgcn_perm(uhb, lb_, 0x07030602u); \
                         asm volatile("" : "+v"(d[4 * g]), "+v"(d[4 * g + 1]), "+v"(d[4 * g + 2]), "+v"(d[4 * g + 3])); SB(); } while (0)
    u32 uha, uhb;
    PayB y0, y1, y2, y3; u32x4 h0, h1, h2, h3, s0, s1; u32 da[16], db[16];
    h0 = ldh(0); h1 = ldh(1); h2 = ldh(2); h3 = ldh(3);
    wait_hdr(h0, ws, hoff, seq16, wt); ldp(y0, 0);
    wait_hdr(h1, ws, hoff + 2048, seq16, wt); ldp(y1, 1);
    wait_hdr(h2, ws, hoff + 4096, seq16, wt); s0 = ldh(4); ldp(y2, 2);
    wait_hdr(h3, ws, hoff + 6144, seq16, wt); s1 = ldh(5); ldp(y3, 3);
    {
        const u32 w0 = rdl(h0.x, 0), w1 = rdl(h0.x, 4), b0 = ((w0 >> 16) & 0x7Fu) * 0x01010101u, b1 = ((w1 >> 16) & 0x7Fu) * 0x01010101u;
        ungrp(da, y0, 0, b0, b1); ungrp(da, y0, 1, b0, b1); ungrp(da, y0, 2, b0, b1); ungrp(da, y0, 3, b0, b1);
        if (((w0 | w1) >> 23) & 7u) fix(da, y0, h0, (w0 >> 23) & 7u, (w1 >> 23) & 7u, 0);
    }
    wait_hdr(s0, ws, hoff + 4 * 2048, seq16, wt);
    lds_wait(cnt + 16, 1u);
    RD(a0, lds, 0);
#define MMA_STEP(u, dc, dn, y1_, h1_, y0_, h0_, sn, so) do { \
        const int t4 = min((u) + 4, NT - 1), t5 = min((u) + 5, NT - 1), t6 = min((u) + 6, NT - 1), t1 = min((u) + 1, NT - 1); \
        const u32 sg = (u32)(u) % NSTAGE, sg1 = (u32)t1 % NSTAGE; \
        LAS unsigned char* st = lds + sg * STAGE; \
        const u32 so4 = pB + t4 * IN_T; \
        h0_ = sn; SB(); \
        RD(a1, st, 1); \
        M1(dc, 0, a0, 0); sn = ldh(t6); SB(); \
        M1(dc, 0, a0, 1); y0_.l0 = __builtin_amdgcn_raw_buffer_load_b128(rs, lane * 16, so4, 17); SB(); \
        M1(dc, 0, a0, 2); y0_.l1 = __builtin_amdgcn_raw_buffer_load_b128(rs, lane * 16 + 1024, so4, 17); SB(); \
        M1(dc, 0, a0, 3); y0_.n = __builtin_amdgcn_raw_buffer_load_b128(rs, lane * 16 + 2048, so4, 17); SB(); \
        const u32 w0 = rdl(h1_.x, 0), w1 = rdl(h1_.x, 4); SB(); \
        M1(dc, 0, a0, 4); \
        const u32 b0 = ((w0 >> 16) & 0x7Fu) * 0x01010101u, b1 = ((w1 >> 16) & 0x7Fu) * 0x01010101u; SB(); \
        RD(a0, st, 2); \
        M1(dc, 1, a1, 0); UH(0, y1_, b0, b1); M1(dc, 1, a1, 1); UP(dn, 0, y1_); M1(dc, 1, a1, 2); UH(1, y1_, b0, b1); M1(dc, 1, a1, 3); UP(dn, 1, y1_); M1(dc, 1, a1, 4); \
        RD(a1, st, 3); \
        nfull = __hip_atomic_load(cnt + 16 + sg1, __ATOMIC_RELAXED, __HIP_MEMORY_SCOPE_WORKGROUP); SB(); \
        M1(dc, 2, a0, 0); UH(2, y1_, b0, b1); M1(dc, 2, a0, 1); UP(dn, 2, y1_); M1(dc, 2, a0, 2); UH(3, y1_, b0, b1); M1(dc, 2, a0, 3); UP(dn, 3, y1_); M1(dc, 2, a0, 4); \
        lds_signal(cnt + 24 + sg); \
        if (((w0 | w1) >> 23) & 7u) fix(dn, y1_, h1_, (w0 >> 23) & 7u, (w1 >> 23) & 7u, t1); \
        wait_hdr(so, ws, hoff + t5 * 2048, seq16, wt); \
        if (rfl(nfull) < (u32)t1 / NSTAGE + 1u) lds_wait(cnt + 16 + sg1, (u32)t1 / NSTAGE + 1u); \
        SB(); \
        RD(a0, lds + sg1 * STAGE, 0); \
        M1(dc, 3, a1, 0); M1(dc, 3, a1, 1); M1(dc, 3, a1, 2); M1(dc, 3, a1, 3); M1(dc, 3, a1, 4); } while (0)
#pragma unroll 1
    for (int u = 0; u < NT; u += 4) {
        MMA_STEP(u, da, db, y1, h1, y0, h0, s0, s1);
        MMA_STEP(u + 1, db, da, y2, h2, y1, h1, s1, s0);
        MMA_STEP(u + 2, da, db, y3, h3, y2, h2, s0, s1);
        MMA_STEP(u + 3, db, da, y0, h0, y3, h3, s1, s0);
    }
#undef UH
#undef UP
#undef M1
#undef SB
#undef MMA_STEP
#undef RD
    STAMP(ws, 6);
    char* const dst = p.ws[src & 7];
    const u32 r32 = 5 * tm;
    const u32 ubase = OFF_Y + (u32)p.rank * Y_SRC + (slab * 64 + r32) * 2048u;
    u32x4 myhdr = (u32x4){0, 0, 0, 0};
#pragma unroll
    for (int m = 0; m < 5; ++m) {
        if (r32 + m < 64) {
            u32 d[8]; float mx = 0.f;
#pragma unroll
            for (int q = 0; q < 8; ++q) {
                d[q] = cvt_pk_bf16(acc[m][2 * q], acc[m][2 * q + 1]);
                mx = fmaxf(mx, fmaxf(fabsf(acc[m][2 * q]), fabsf(acc[m][2 * q + 1])));
            }
            u32x4 L, hdr; u32x2 N;
            pack_unit(d, mx, seq16, L, N, hdr);
            char* q = dst + ubase + m * 2048u;
            if (((hdr.x >> 23) & 7u) != 7u) { st16_sys(q + lane * 16, L); st8_sys(q + 1024 + lane * 8, N); }
            else { st16_sys(q + lane * 16, (u32x4){d[0], d[1], d[2], d[3]}); st16_sys(q + 1024 + lane * 16, (u32x4){d[4], d[5], d[6], d[7]}); }
            if (lane == m) myhdr = hdr;
        }
    }
    STAMP(ws, 7);
    drain();
    if (lane < 5 && r32 + lane < 64) st16_sys(dst + OFF_HY + (((u32)p.rank * 8 + slab) * 64 + r32 + lane) * 16u, myhdr);
    STAMP(ws, 8);
}

__global__ void __launch_bounds__(NTHR, 2) k_main(Params p) {
    extern __shared__ __attribute__((aligned(16))) unsigned char smem[];
    LAS unsigned char* lds = (LAS unsigned char*)smem;
    const int b = blockIdx.x, wv = rfl(threadIdx.x >> 6), lane = threadIdx.x & 63;
    char* const ws = p.ws[p.rank];
    if (wv == 0) STAMP(ws, 0);
    GAS u32* cntp = (GAS u32*)(ws + OFF_CNT + b * 128);
    const u32 callv = __hip_atomic_load(cntp, __ATOMIC_RELAXED, __HIP_MEMORY_SCOPE_AGENT);
    const int xg = b & 7, slot = b >> 3;
    if (slot < 4 && wv < 2 * PW) {
        const int gw = (wv >> 1) * 32 + slot * 8 + xg;
        pusher(p, lds, wv, gw, lane, callv);
    }
    const u32 call = rfl(callv) + 1u;
    const u32 seq16 = call % 65535u + 1u;
    Wait wt{(GAS u32*)(ws + OFF_TMO), call, false};
    if (ROLE_OF(b) == 1) {
        const int s = slot - 4, tm = s >> 1, tn = 2 * xg + (s & 1);
        if (threadIdx.x < 16) ((LAS u32*)(lds + LDS_MISC))[16 + threadIdx.x] = 0u;
        BARRIER();
        if (wv >= 4) decoder(p, lds, tm, wv - 4, lane, wt, seq16); else mma_wave(p, lds, tm, tn, wv, lane, wt, seq16);
    }
    const __amdgpu_buffer_rsrc_t rs = __builtin_amdgcn_make_buffer_rsrc(ws, 0, (int)WS_NEED, 0x00020000);
    const u32 ri = b >> 7, rj = (b >> 6) & 1, r32 = b & 63, cg8 = wv;
    const u32 snd0 = (ri << 2) | (rj << 1);
    float y[16];
    LAS u32* misc = (LAS u32*)(lds + LDS_MISC);
    {
        const u32 hoff = OFF_HY + (((snd0 + min((u32)lane >> 2, 1u)) * 8 + cg8) * 64 + r32) * 16u;
        u32x4 h = __builtin_amdgcn_raw_buffer_load_b128(rs, hoff, 0, 17);
        if (!hdr_ok(h, seq16)) { const u64 t0 = rtime(); u32 n = 0; do { if (spin_on(wt, t0, n)) break; h = ld16_sync(ws, hoff); } while (!hdr_ok(h, seq16)); }
        STAMP(ws, 24 + wv);
        u32x4 l[2]; u32x2 n[2];
#pragma unroll
        for (int s = 0; s < 2; ++s) {
            const u32 ub = OFF_Y + (snd0 + s) * Y_SRC + (cg8 * 64 + r32) * 2048u;
            l[s] = __builtin_amdgcn_raw_buffer_load_b128(rs, lane * 16, ub, 17);
            n[s] = __builtin_amdgcn_raw_buffer_load_b64(rs, lane * 8 + 1024, ub, 17);
        }
#pragma unroll
        for (int s = 0; s < 2; ++s) {
            const u32 w0 = rdl(h.x, 4 * s), mode = (w0 >> 23) & 7u;
            u32 d[8];
            if (mode != 7u) {
                const u32 brep = ((w0 >> 16) & 0x7Fu) * 0x01010101u;
                const u32x4 g0 = decode_group(l[s].x, l[s].y, n[s].x, brep), g1 = decode_group(l[s].z, l[s].w, n[s].y, brep);
                d[0] = g0.x; d[1] = g0.y; d[2] = g0.z; d[3] = g0.w; d[4] = g1.x; d[5] = g1.y; d[6] = g1.z; d[7] = g1.w;
                if (mode) patch_regs(d, h, 4 * s, mode, lane);
            } else {
                const u32 ub = OFF_Y + (snd0 + s) * Y_SRC + (cg8 * 64 + r32) * 2048u;
                const u32x4 hi = ld16_sync(ws, ub + lane * 16 + 1024);
                d[0] = l[s].x; d[1] = l[s].y; d[2] = l[s].z; d[3] = l[s].w; d[4] = hi.x; d[5] = hi.y; d[6] = hi.z; d[7] = hi.w;
            }
#pragma unroll
            for (int q = 0; q < 8; ++q) {
                const float lo = __uint_as_float(d[q] << 16), hi = __uint_as_float(d[q] & 0xFFFF0000u);
                if (s == 0) { y[2 * q] = lo; y[2 * q + 1] = hi; } else { y[2 * q] += lo; y[2 * q + 1] += hi; }
            }
        }
        float mx = 0.f;
#pragma unroll
        for (int q = 0; q < 16; ++q) mx = fmaxf(mx, fabsf(y[q]));
        const u32 wm = wave_max_u32(__float_as_uint(mx));
        if (lane == 0) misc[wv] = wm;
        STAMPD(ws, 16 + wv);
    }
    BARRIER();
    if (wv == 0) {
        const u32 v = wave_max_u32(misc[lane & 7]);
        if (lane == 0) __hip_atomic_store((GAS u64*)(ws + OFF_LMAX + b * 64), ((u64)call << 32) | v, __ATOMIC_RELAXED, __HIP_MEMORY_SCOPE_AGENT);
        STAMP(ws, 11);
        const u32 go = OFF_GMAX + ((b & 7) * 8 + (lane & 7)) * 64;
        u32x2 g = ld8_sync(ws, go);
        if (__ballot(g.y == call) != ~0ull) { const u64 t0 = rtime(); u32 n = 0; do { if (spin_on(wt, t0, n)) break; g = ld8_sync(ws, go); } while (__ballot(g.y == call) != ~0ull); }
        const u32 gm = wave_max_u32(g.x);
        if (lane == 0) misc[8] = gm;
    } else if (wv == 1 && b == LEADER) {
        u32x2 g0, g1, g2, g3;
        auto poll = [&]() {
            asm volatile("global_load_dwordx2 %0, %4, %5 sc1\n\tglobal_load_dwordx2 %1, %4, %5 offset:64 sc1\n\tglobal_load_dwordx2 %2, %4, %5 offset:128 sc1\n\tglobal_load_dwordx2 %3, %4, %5 offset:192 sc1\n\ts_waitcnt vmcnt(0)"
                         : "=&v"(g0), "=&v"(g1), "=&v"(g2), "=&v"(g3) : "v"(OFF_LMAX + lane * 256), "s"(ws) : "memory");
            return __ballot(g0.y == call && g1.y == call && g2.y == call && g3.y == call) == ~0ull; };
        if (!poll()) { const u64 t0 = rtime(); u32 n = 0; do { if (spin_on(wt, t0, n)) break; } while (!poll()); }
        const u32 lm = wave_max_u32(max(max(g0.x, g1.x), max(g2.x, g3.x)));
        __hip_atomic_store((GAS u64*)(p.ws[lane >> 3] + OFF_GMAX + ((lane & 7) * 8 + (u32)p.rank) * 64), ((u64)call << 32) | lm, __ATOMIC_RELAXED, __HIP_MEMORY_SCOPE_SYSTEM);
        STAMP(ws, 14);
    }
    BARRIER();
    if (wv == 0) STAMP(ws, 12);
    {
        const float amax = __uint_as_float(misc[8]);
        const bool bad = rfl(__hip_atomic_load(wt.tmo, __ATOMIC_RELAXED, __HIP_MEMORY_SCOPE_AGENT)) == call;
        const float inv = amax > 0.f ? 448.0f / amax : 0.f;
        u32 o[4];
#pragma unroll
        for (int q = 0; q < 4; ++q) {
            float v[4];
#pragma unroll
            for (int e = 0; e < 4; ++e) v[e] = __builtin_amdgcn_fmed3f(y[4 * q + e] * inv, -448.f, 448.f);
            int t = __builtin_amdgcn_cvt_pk_fp8_f32(v[0], v[1], 0, false); t = __builtin_amdgcn_cvt_pk_fp8_f32(v[2], v[3], t, true); o[q] = (u32)t;
        }
        const u32 R = r32 * 32 + (lane & 31), am = R >> 4;
        const u32 row = 512 * ((am >> 3) & 7) + 256 * ri + 128 * (am >> 6) + 16 * (am & 7) + (R & 15), col = 256 * rj + 32 * cg8 + 16 * (lane >> 5);
        const u32x4 ov = bad ? (u32x4){0x7f7f7f7fu, 0x7f7f7f7fu, 0x7f7f7f7fu, 0x7f7f7f7fu} : (u32x4){o[0], o[1], o[2], o[3]};
        *(GAS u32x4*)(p.out + (size_t)row * SH + col) = ov;
    }
    if (threadIdx.x == 0) __hip_atomic_store(cntp, call, __ATOMIC_RELAXED, __HIP_MEMORY_SCOPE_AGENT);
    if (wv == 0) STAMP(ws, 13);
}

extern "C" void kernel_launch(int rank, int nranks, ncclComm_t comm, hipStream_t stream,
                              void* const* d_in, const int* in_sizes, int n_in,
                              void* d_out, int out_size, const int* in_dtypes, int out_dtype,
                              void* const* d_out_all, void* const* d_ws_all, size_t ws_bytes) {
    if (nranks != NR || (unsigned)rank >= (unsigned)NR || ws_bytes < WS_NEED || n_in < 2 || in_sizes[0] != SH * KG || in_sizes[1] != KG * SH || out_size != MG * SH) {
        fprintf(stderr, "kernel_launch: unexpected shapes\n"); return; }
    HIP_CHECK(hipFuncSetAttribute((const void*)k_main, hipFuncAttributeMaxDynamicSharedMemorySize, LDS_BYTES));
    Params p;
    p.x = (const float*)d_in[0]; p.w = (const float*)d_in[1]; p.out = (unsigned char*)d_out;
    for (int i = 0; i < NR; ++i) p.ws[i] = (char*)d_ws_all[i];
    p.rank = rank; p.pad = 0;
    k_main<<<256, NTHR, LDS_BYTES, stream>>>(p);
    HIP_CHECK(hipGetLastError());
}
```

```cpp
#include <hip/hip_runtime.h>
#include <rccl/rccl.h>
#include <cstdio>
#include <cstdint>

#define HIP_CHECK(expr) do { hipError_t _e = (expr); if (_e != hipSuccess) { \
    fprintf(stderr, "HIP error %s at %s:%d\n", hipGetErrorString(_e), __FILE__, __LINE__); \
    return; } } while (0)

typedef unsigned u32;
typedef unsigned long long u64;
typedef u32 u32x4 __attribute__((ext_vector_type(4)));
typedef u32 u32x2 __attribute__((ext_vector_type(2)));
typedef float f32x4 __attribute__((ext_vector_type(4)));
typedef float f32x16 __attribute__((ext_vector_type(16)));
typedef short bf16x8 __attribute__((ext_vector_type(8)));
#define LAS __attribute__((address_space(3)))
#define GAS __attribute__((address_space(1)))

constexpr int NR = 8, MG = 4096, KG = 4096, SH = 512;
constexpr int NT = 32;
constexpr int NTHR = 512;
constexpr u64 DEADLINE = 150000000ull;
constexpr u32 PKT = 3072;
constexpr u32 OFF_CNT = 0;
constexpr u32 OFF_TMO = 0x8000;
constexpr u32 OFF_TICK = 0x8100;
constexpr u32 OFF_LMAX = 0x10000;
constexpr u32 OFF_GMAX = 0x14000;
constexpr u32 OFF_STAMP = 0x20000;
constexpr u32 OFF_HA = 0x30000;
constexpr u32 OFF_HB = 0x40000;
constexpr u32 OFF_HY = 0x50000;
constexpr u32 OFF_IN = 2u << 20;
constexpr u32 IN_SRC = 2u << 20, IN_B = 768u << 10, IN_T = 8 * PKT;
constexpr u32 OFF_Y = 18u << 20;
constexpr u32 Y_SRC = 1u << 20;
constexpr u32 OFF_OVF = 26u << 20;
constexpr size_t WS_NEED = 34u << 20;
constexpr int STAGE = 20480, NSTAGE = 6, LDS_MISC = NSTAGE * STAGE, LDS_BYTES = 124 * 1024;
constexpr int LEADER = 0;
#define NLEAD 1

struct Params { const float* x; const float* w; unsigned char* out; char* ws[NR]; int rank; int pad; };

__device__ __forceinline__ u32 cvt_pk_bf16(float lo, float hi) { u32 r; asm("v_cvt_pk_bf16_f32 %0, %1, %2" : "=v"(r) : "v"(lo), "v"(hi)); return r; }
__device__ __forceinline__ void st16_sys(void* p, u32x4 v) { asm volatile("global_store_dwordx4 %0, %1, off sc0 sc1\n\ts_nop 1" :: "v"(p), "v"(v) : "memory"); }
__device__ __forceinline__ void st8_sys(void* p, u32x2 v) { asm volatile("global_store_dwordx2 %0, %1, off sc0 sc1\n\ts_nop 1" :: "v"(p), "v"(v) : "memory"); }
__device__ __forceinline__ u32x4 ld16_sync(const char* base, u32 off) { u32x4 r; asm volatile("global_load_dwordx4 %0, %1, %2 sc0 sc1\n\ts_waitcnt vmcnt(0)" : "=&v"(r) : "v"(off), "s"(base) : "memory"); return r; }
__device__ __forceinline__ u32x2 ld8_sync(const char* base, u32 off) { u32x2 r; asm volatile("global_load_dwordx2 %0, %1, %2 sc0 sc1\n\ts_waitcnt vmcnt(0)" : "=&v"(r) : "v"(off), "s"(base) : "memory"); return r; }
__device__ __forceinline__ u32 ld4_sync_agent(const void* p) { u32 r; asm volatile("global_load_dword %0, %1, off sc1\n\ts_waitcnt vmcnt(0)" : "=&v"(r) : "v"(p) : "memory"); return r; }
__device__ __forceinline__ void drain() { asm volatile("s_waitcnt vmcnt(0)" ::: "memory"); }
__device__ __forceinline__ u64 rtime() { return __builtin_amdgcn_s_memrealtime(); }
__device__ __forceinline__ u32 rfl(u32 v) { return __builtin_amdgcn_readfirstlane(v); }
__device__ __forceinline__ u32 rdl(u32 v, int l) { return __builtin_amdgcn_readlane(v, l); }
__device__ __forceinline__ u32 wave_max_u32(u32 v) {
    v = max(v, (u32)__builtin_amdgcn_update_dpp(0, (int)v, 0x111, 0xf, 0xf, false));
    v = max(v, (u32)__builtin_amdgcn_update_dpp(0, (int)v, 0x112, 0xf, 0xf, false));
    v = max(v, (u32)__builtin_amdgcn_update_dpp(0, (int)v, 0x114, 0xf, 0xf, false));
    v = max(v, (u32)__builtin_amdgcn_update_dpp(0, (int)v, 0x118, 0xf, 0xf, false));
    v = max(v, (u32)__builtin_amdgcn_update_dpp(0, (int)v, 0x142, 0xa, 0xf, false));
    v = max(v, (u32)__builtin_amdgcn_update_dpp(0, (int)v, 0x143, 0xc, 0xf, false));
    return rdl(v, 63);
}
#define BARRIER() do { asm volatile("s_waitcnt lgkmcnt(0)" ::: "memory"); __builtin_amdgcn_s_barrier(); asm volatile("" ::: "memory"); } while (0)

#define NPS 5
#define ROLE_OF(b) (((b) >> 3) < NPS ? 0 : ((b) >> 3) >= 6 ? 1 : 2)
#define STAMP(ws, slot) do {} while (0)
#define STAMPD(ws, slot) do {} while (0)

#define CT() 0ull
#define CACC(a, t0, t1) do {} while (0)

__device__ __forceinline__ void pack_unit(const u32* d, float amax_lane, u32 seq16, u32x4& L, u32x2& N, u32x4& hdr) {
    u32 H[4];
#pragma unroll
    for (int q = 0; q < 4; ++q) { L[q] = __builtin_amdgcn_perm(d[2 * q + 1], d[2 * q], 0x06040200u); H[q] = __builtin_amdgcn_perm(d[2 * q + 1], d[2 * q], 0x07050301u); }
    const u32 mb = wave_max_u32(__float_as_uint(amax_lane));
    u32 h7 = (mb + 0x7FFFu + ((mb >> 16) & 1u)) >> 24;
    h7 = h7 > 127u ? 127u : h7;
    const u32 base = h7 > 7u ? h7 - 7u : 0u, brep = base * 0x01010101u;
    u32 b[4], X[4];
#pragma unroll
    for (int q = 0; q < 4; ++q) { const u32 t = (H[q] | 0x80808080u) - brep; b[q] = (t ^ 0x80808080u) & 0xF8F8F8F8u; X[q] = (H[q] & 0x80808080u) | (t & 0x07070707u); }
    N[0] = X[0] | __builtin_amdgcn_alignbit(X[1], X[1], 28); N[1] = X[2] | __builtin_amdgcn_alignbit(X[3], X[3], 28);
    u32 cnt = 0; u64 ea = 0, eb = 0;
    u64 m = __ballot((b[0] | b[1] | b[2] | b[3]) != 0);
    if (m) {
        u32 bm = 0;
#pragma unroll
        for (int q = 0; q < 4; ++q) bm |= (((((((b[q] & 0x7F7F7F7Fu) + 0x7F7F7F7Fu) | b[q]) & 0x80808080u) >> 7) * 0x01020408u) >> 24 & 0xFu) << (4 * q);
        while (m && cnt <= 4) {
            const int l = __builtin_ctzll(m); m &= m - 1;
            u32 mk = rdl(bm, l);
            const u32 h0 = rdl(H[0], l), h1 = rdl(H[1], l), h2 = rdl(H[2], l), h3 = rdl(H[3], l);
            while (mk) {
                const u32 j = (u32)__builtin_ctz(mk); mk &= mk - 1;
                const u32 hq = (j & 8u) ? ((j & 4u) ? h3 : h2) : ((j & 4u) ? h1 : h0);
                const u64 e = (u64)((u32)l * 16u + j) | ((u64)((hq >> (8u * (j & 3u))) & 0xFFu) << 11);
                if (cnt < 2) ea |= e << (26u + 19u * cnt); else if (cnt < 4) eb |= e << (16u + 19u * (cnt - 2u));
                ++cnt;
            }
        }
    }
    const u32 mode = cnt > 4 ? 7u : cnt;
    ea |= (u64)seq16 | ((u64)base << 16) | ((u64)mode << 23); eb |= (u64)seq16;
    hdr = (u32x4){(u32)ea, (u32)(ea >> 32), (u32)eb, (u32)(eb >> 32)};
}
__device__ __forceinline__ u32x4 decode_group(u32 LA, u32 LB, u32 n, u32 brep) {
    const u32 HA = (n & 0x87878787u) + brep, HB = (__builtin_amdgcn_alignbit(n, n, 4) & 0x87878787u) + brep;
    return (u32x4){__builtin_amdgcn_perm(HA, LA, 0x05010400u), __builtin_amdgcn_perm(HA, LA, 0x07030602u),
                   __builtin_amdgcn_perm(HB, LB, 0x05010400u), __builtin_amdgcn_perm(HB, LB, 0x07030602u)};
}
__device__ __forceinline__ u32 hdr_exc(u64 a, u64 b, u32 c) { return (u32)((c < 2 ? a >> (26u + 19u * c) : b >> (16u + 19u * (c - 2u)))) & 0x7FFFFu; }
__device__ __forceinline__ bool hdr_ok(const u32x4& h, u32 seq16) { return __ballot((((h.x - seq16) | (h.z - seq16)) & 0x8000u) == 0u) == ~0ull; }
__device__ __forceinline__ void patch_regs(u32* d, const u32x4& h, int hl, u32 mode, int lane) {
    const u64 a = (u64)rdl(h.x, hl) | ((u64)rdl(h.y, hl) << 32), b = (u64)rdl(h.z, hl) | ((u64)rdl(h.w, hl) << 32);
    for (u32 c = 0; c < mode; ++c) {
        const u32 e = hdr_exc(a, b, c), el = (e >> 4) & 63u, j = e & 15u, hb = e >> 11, sh = (j & 1u) * 16u + 8u;
#pragma unroll
        for (int q = 0; q < 8; ++q) if (j >> 1 == (u32)q && (u32)lane == el) d[q] = (d[q] & ~(0xFFu << sh)) | (hb << sh);
    }
}

__device__ __forceinline__ void lds_signal(LAS u32* c) {
    asm volatile("s_mov_b64 exec, 1\n\tds_add_u32 %0, %1\n\ts_mov_b64 exec, -1" :: "v"((u32)(uintptr_t)c), "v"(1u) : "memory");
}
__device__ __forceinline__ void lds_wait(LAS u32* c, u32 target) {
    while (rfl(__hip_atomic_load(c, __ATOMIC_RELAXED, __HIP_MEMORY_SCOPE_WORKGROUP)) < target) __builtin_amdgcn_s_sleep(1);
    asm volatile("" ::: "memory");
}

struct Wait { GAS u32* tmo; u32 call; bool dead; };
template <int SLEEP = 1>
__device__ __forceinline__ bool spin_on(Wait& wt, u64 t0, u32& n) {
    if (wt.dead) return true;
    if ((++n & 31u) == 0u) {
        if (rfl(ld4_sync_agent((const void*)wt.tmo)) == wt.call || rtime() - t0 > DEADLINE) {
            __hip_atomic_store(wt.tmo, wt.call, __ATOMIC_RELAXED, __HIP_MEMORY_SCOPE_AGENT); wt.dead = true; return true; }
    }
    if (SLEEP) __builtin_amdgcn_s_sleep(SLEEP);
    return false;
}
__device__ __forceinline__ void wait_hdr(u32x4& h, const char* ws, u32 off, u32 seq16, Wait& wt) {
    if (!hdr_ok(h, seq16)) { const u64 t0 = rtime(); u32 n = 0; do { if (spin_on(wt, t0, n)) break; h = ld16_sync(ws, off); } while (!hdr_ok(h, seq16)); }
}
#define HDR_READY(h) hdr_ok(h, seq16)

#define PW 2
#define SKEW 12
#define NA 4
#define WLEAD 4
constexpr u32 PWAVES = 8 * NPS * PW;
struct Pkt { u32 isw, t, h, kh, blk; };
__device__ __forceinline__ Pkt ticket_packet(u32 n, u32 pool) {
    const u32 tau = min(2u * n + pool, 2047u);
    constexpr u32 XA = 4u * NA, XB = 32u - XA;
    constexpr u32 P0 = 32u * WLEAD, P1 = P0 + (32u + XA) * SKEW, P2 = P1 + 64u * (32u - WLEAD - SKEW), P3 = P2 + 32u * WLEAD;
    static_assert(WLEAD + SKEW <= 32 && P3 + XB * SKEW == 2048u, "slot layout");
    int j; u32 c;
    if (tau < P0) { j = (int)(tau >> 5) - WLEAD; c = tau & 31u; }
    else if (tau < P1) { const u32 q = (tau - P0) / (32u + XA); j = (int)q; c = tau - P0 - (32u + XA) * q; }
    else if (tau < P2) { j = SKEW + (int)((tau - P1) >> 6); c = (tau - P1) & 63u; }
    else if (tau < P3) { j = 32 - WLEAD + (int)((tau - P2) >> 5); c = 32u + ((tau - P2) & 31u); }
    else { const u32 q = (tau - P3) / XB; j = 32 + (int)q; c = 32u + XA + (tau - P3 - XB * q); }
    Pkt k;
    k.isw = c < 32u;
    k.t = (u32)(c < 32u ? j + WLEAD : c < 32u + XA ? j : j - SKEW);
    if (c < 32u) { k.h = c >> 4; k.kh = (c >> 3) & 1u; k.blk = c & 7u; }
    else if (c < 32u + XA) { const u32 i = c - 32u, q4 = i / NA; k.h = q4 >> 1; k.kh = q4 & 1u; k.blk = i - q4 * NA; }
    else { const u32 i = c - 32u - XA, q4 = i / (8u - NA); k.h = q4 >> 1; k.kh = q4 & 1u; k.blk = NA + (i - q4 * (8u - NA)); }
    return k;
}
__device__ __forceinline__ u32 mt_index(u32 src, u32 slot) { return slot < 2u * NA ? src * (2u * NA) + slot : 16u * NA + src * (16u - 2u * NA) + (slot - 2u * NA); }
__device__ __forceinline__ void mt_split(u32 x, u32& src, u32& slot) {
    if (x < 16u * NA) { src = x / (2u * NA); slot = x - src * (2u * NA); }
    else { const u32 y = x - 16u * NA; src = y / (16u - 2u * NA); slot = 2u * NA + (y - src * (16u - 2u * NA)); }
}
__device__ __forceinline__ void pusher(const Params& p, LAS unsigned char* lds, int wvi, int gw, int lane, u32 callv) {
    const u32 pool = wvi & 1;
    const __amdgpu_buffer_rsrc_t rt = __builtin_amdgcn_make_buffer_rsrc(p.ws[p.rank] + OFF_TICK + pool * 128, 0, 4, 0x00020000);
    const u32 loffw = (u32)((lane >> 3) * SH + 4 * (lane & 7)) * 4u, loffx = (u32)((lane & 15) * KG + 8 * (lane >> 4)) * 4u;
    const u32 lbase = (u32)(uintptr_t)lds + (u32)wvi * 16384u;
    auto ld = [&](u32 buf, u32 n) {
        const Pkt k = ticket_packet(n, pool);
        const u32 l0 = lbase + buf * 8192u;
        u32 keep;
#define GL(i, off) "s_mov_b32 m0, %" #i "\n\ts_nop 0\n\tglobal_load_lds_dwordx4 " off "\n\t"
#define GL8(sb, o0, o1, o2, o3, o4, o5, o6, o7) asm volatile("s_mov_b32 %0, m0\n\t" \
                         GL(6, "%1, %5") GL(7, "%2, %5") GL(8, "%3, %5") GL(9, "%4, %5") GL(10, "%14, %5") GL(11, "%15, %5") GL(12, "%16, %5") GL(13, "%17, %5") \
                         "s_mov_b32 m0, %0" : "=&s"(keep) : "v"(o0), "v"(o1), "v"(o2), "v"(o3), "s"(sb), \
                           "s"(l0), "s"(l0 + 1024u), "s"(l0 + 2048u), "s"(l0 + 3072u), "s"(l0 + 4096u), "s"(l0 + 5120u), "s"(l0 + 6144u), "s"(l0 + 7168u), \
                           "v"(o4), "v"(o5), "v"(o6), "v"(o7) : "memory")
        if (k.isw) {
            const GAS char* const sb = (const GAS char*)p.w + ((2048u * k.kh + 64u * k.t) * SH + 256u * k.h + 32u * k.blk) * 4u;
            GL8(sb, loffw, loffw + 16384u, loffw + 32768u, loffw + 49152u, loffw + 65536u, loffw + 81920u, loffw + 98304u, loffw + 114688u);
        } else {
            const GAS char* const sb = (const GAS char*)p.x + ((256u * k.h + 32u * k.blk) * KG + 2048u * k.kh + 64u * k.t) * 4u;
            constexpr u32 R16 = 16u * KG * 4u;
            GL8(sb, loffx, loffx + 16u, loffx + 128u, loffx + 144u, loffx + R16, loffx + R16 + 16u, loffx + R16 + 128u, loffx + R16 + 144u);
        }
#undef GL8
#undef GL
    };
    u32 ta = gw, tb = gw + PWAVES, tc = gw + 2 * PWAVES;
    ld(0, ta); ld(1, tb);
    asm volatile("s_waitcnt vmcnt(8)" ::: "memory");
    const u32 call = rfl(callv) + 1u, seq16 = call & 0xFFFFu;
    const u32 tbase = (call - 1u) * 1024u - 3u * PWAVES;
    const u32 wcol = (16u * ((lane >> 2) & 1u) + 4u * ((lane & 31) >> 3) + (lane & 3u)) * 4u + (lane >> 5) * 1024u;
    u32x4 ph = (u32x4){0, 0, 0, 0}; char* pa = nullptr; bool have = false; u32 drawn = 0;
    auto step = [&](u32 buf, u32& n) {
        const Pkt k = ticket_packet(n, pool);
        LAS unsigned char* const lb = lds + wvi * 16384 + buf * 8192;
        asm volatile("s_waitcnt vmcnt(14)" ::: "memory");
        u32 d[16]; float mx[2] = {0.f, 0.f};
        if (k.isw) {
#pragma unroll
            for (int g = 0; g < 4; ++g)
#pragma unroll
                for (int q = 0; q < 4; ++q) {
                    const float lo = *(const LAS float*)(lb + wcol + (16 * g + 2 * q) * 128), hi = *(const LAS float*)(lb + wcol + (16 * g + 2 * q + 1) * 128);
                    d[g * 4 + q] = cvt_pk_bf16(lo, hi); mx[g >> 1] = fmaxf(mx[g >> 1], fmaxf(fabsf(lo), fabsf(hi)));
                }
        } else {
#pragma unroll
            for (int li = 0; li < 8; ++li) {
                const f32x4 v = *(const LAS f32x4*)(lb + li * 1024 + lane * 16);
                d[2 * li] = cvt_pk_bf16(v[0], v[1]); d[2 * li + 1] = cvt_pk_bf16(v[2], v[3]);
                mx[li >> 2] = fmaxf(fmaxf(mx[li >> 2], fmaxf(fabsf(v[0]), fabsf(v[1]))), fmaxf(fabsf(v[2]), fabsf(v[3])));
            }
        }
        u32 d0 = k.isw ? ((k.h << 1) | k.kh) : ((k.h << 2) | k.kh), d1 = d0 | (k.isw ? 4u : 2u);
        if ((n ^ (n >> 5)) & 1u) { const u32 x = d0; d0 = d1; d1 = x; }
        char* const w0 = p.ws[d0 & 7u]; char* const w1 = p.ws[d1 & 7u];
        const u32 po = OFF_IN + (u32)p.rank * IN_SRC + (k.isw ? IN_B : 0u) + k.t * IN_T + k.blk * PKT + lane * 16u;
        u32x4 hd[2], L[2]; u32x2 N[2];
        pack_unit(d, mx[0], seq16, L[0], N[0], hd[0]);
        pack_unit(d + 8, mx[1], seq16, L[1], N[1], hd[1]);
        const u32x4 NN = (u32x4){N[0].x, N[0].y, N[1].x, N[1].y};
        asm volatile("" :: "v"(NN), "v"(L[0]), "v"(L[1]));
        drain();
        const u32 dv = rfl(drawn);
        if (have) { tc = dv - tbase; if (lane < 4) st16_sys(pa, ph); }
        drawn = (u32)__builtin_amdgcn_raw_ptr_buffer_atomic_add_i32(1, rt, lane == 0 ? 0 : 0x7FFFFFF0, 0, 0);
#pragma unroll
        for (int u = 0; u < 2; ++u)
            if (((hd[u].x >> 23) & 7u) == 7u) {
                const u32 ov = OFF_OVF + ((u32)p.rank * 2u + k.isw) * (512u << 10) + k.t * 16384u + (2u * k.blk + u) * 1024u + lane * 16u;
                const u32x4 r1 = (u32x4){d[8 * u + 4], d[8 * u + 5], d[8 * u + 6], d[8 * u + 7]};
                L[u] = (u32x4){d[8 * u], d[8 * u + 1], d[8 * u + 2], d[8 * u + 3]};
                st16_sys(w0 + ov, r1); st16_sys(w1 + ov, r1);
            }
        st16_sys(w0 + po, L[0]); st16_sys(w1 + po, L[0]); st16_sys(w0 + po + 1024, L[1]); st16_sys(w1 + po + 1024, L[1]); st16_sys(w0 + po + 2048, NN); st16_sys(w1 + po + 2048, NN);
        asm volatile("s_waitcnt lgkmcnt(0)" ::: "memory");
        ld(buf, tc); n = tc;
        const u32 hi = k.isw ? (u32)p.rank * 16u + 2u * k.blk : mt_index((u32)p.rank, 2u * k.blk);
        pa = ((lane & 2) ? w1 : w0) + (k.isw ? OFF_HB : OFF_HA) + k.t * 2048u + (hi + (lane & 1)) * 16u;
        ph = (lane & 1) ? hd[1] : hd[0]; have = true;
    };
#pragma unroll 1
    while (true) {
        if (ta >= 1024u) break;
        step(0, ta);
        if (tb >= 1024u) break;
        step(1, tb);
    }
    drain();
    asm volatile("" :: "v"(drawn));
    if (lane < 4) st16_sys(pa, ph);
    STAMP(p.ws[p.rank], 15);
}

#define LW(p, v) *(LAS u32x4*)(p) = (v)
__device__ __forceinline__ void decoder(const Params& p, LAS unsigned char* lds, int tm, int dw, int lane, Wait& wt, u32 seq16) {
    char* const ws = p.ws[p.rank];
    LAS u32* const cnt = (LAS u32*)(lds + LDS_MISC);
    const __amdgpu_buffer_rsrc_t rs = __builtin_amdgcn_make_buffer_rsrc(ws, 0, (int)WS_NEED, 0x00020000);
    const u32 fr = lane & 15, fq = lane >> 4;
    auto uaddr = [&](u32 v, u32 g, u32 efr, u32 efq) -> u32 { const u32 row = 16u * v + efr; return row * 128u + (((4u * g + efq) ^ (row & 7u) ^ ((row >> 3) & 3u)) << 4); };
    auto mtile = [&](u32 v) -> u32 { return min(10u * tm + v, 127u); };
    u32 po[5];
#pragma unroll
    for (int s = 0; s < 5; ++s) { u32 sr, sl; mt_split(mtile(2 * s), sr, sl); po[s] = OFF_IN + sr * IN_SRC + (sl >> 1) * PKT; }
    const u32 ae0 = uaddr(0, 0, fr, fq), ae1 = uaddr(0, 1, fr, fq), ao0 = uaddr(1, 0, fr, fq) - 2048u, ao1 = uaddr(1, 1, fr, fq) - 2048u;
    const u32 hv = min((u32)lane >> 2, 9u);
    const u32 hoff = OFF_HA + mtile(hv) * 16u;
    struct PayA { u32x4 l0[5], l1[5], n[5]; };
    auto ldh = [&](int t) -> u32x4 { return __builtin_amdgcn_raw_buffer_load_b128(rs, hoff, t * 2048, 17); };
    auto ldp = [&](PayA& y, int t) {
        const u32 so = t * IN_T;
#pragma unroll
        for (int s = 0; s < 5; ++s) {
            y.l0[s] = __builtin_amdgcn_raw_buffer_load_b128(rs, lane * 16, po[s] + so, 17);
            y.l1[s] = __builtin_amdgcn_raw_buffer_load_b128(rs, lane * 16 + 1024, po[s] + so, 17);
            y.n[s] = __builtin_amdgcn_raw_buffer_load_b128(rs, lane * 16 + 2048, po[s] + so, 17);
        }
    };
    u64 c_free = 0, c_dec = 0, c_iss = 0;
    auto freec = [&](int t) -> u32 { return __hip_atomic_load(cnt + 24 + (u32)t % NSTAGE, __ATOMIC_RELAXED, __HIP_MEMORY_SCOPE_WORKGROUP); };
    auto dec = [&](const PayA& y, const u32x4& h, int t, u32 nfree) {
        const u32 sg = (u32)t % NSTAGE, turn = (u32)t / NSTAGE;
        LAS unsigned char* st = lds + sg * STAGE;
        const u64 q0 = CT();
        if (rfl(nfree) < 4u * turn) lds_wait(cnt + 24 + sg, 4u * turn);
        const u64 q1 = CT(); CACC(c_free, q0, q1);
        const u32 mode = (h.x >> 23) & 7u, brl = ((h.x >> 16) & 0x7Fu) * 0x01010101u;
        if (__ballot(mode == 7u) == 0ull) {
#pragma unroll
            for (int s = 0; s < 5; ++s) {
                const u32 b0 = rdl(brl, 8 * s), b1 = rdl(brl, 8 * s + 4);
                LW(st + ae0 + s * 4096, decode_group(y.l0[s].x, y.l0[s].y, y.n[s].x, b0)); LW(st + ae1 + s * 4096, decode_group(y.l0[s].z, y.l0[s].w, y.n[s].y, b0));
                LW(st + ao0 + s * 4096 + 2048, decode_group(y.l1[s].x, y.l1[s].y, y.n[s].z, b1)); LW(st + ao1 + s * 4096 + 2048, decode_group(y.l1[s].z, y.l1[s].w, y.n[s].w, b1));
            }
        } else {
#pragma unroll
            for (int v = 0; v < 10; ++v) {
                u32 sr, sl; mt_split(mtile(v), sr, sl);
                const u32 br = rdl(brl, 4 * v);
                const u32x4 l = (v & 1) ? y.l1[v >> 1] : y.l0[v >> 1]; const u32 n0 = (v & 1) ? y.n[v >> 1].z : y.n[v >> 1].x, n1 = (v & 1) ? y.n[v >> 1].w : y.n[v >> 1].y;
                const u32x4 rw = ld16_sync(ws, OFF_OVF + sr * (1024u << 10) + sl * 1024u + t * 16384 + lane * 16);
                const bool raw = rdl(mode, 4 * v) == 7u;
                *(LAS u32x4*)(st + ((v & 1) ? ao0 : ae0) + v * 2048) = raw ? l : decode_group(l.x, l.y, n0, br);
                *(LAS u32x4*)(st + ((v & 1) ? ao1 : ae1) + v * 2048) = raw ? rw : decode_group(l.z, l.w, n1, br);
            }
        }
        if (__ballot(mode - 1u < 4u) != 0ull) {
            const u32 c = lane & 3;
            const u64 a = (u64)h.x | ((u64)h.y << 32), b = (u64)h.z | ((u64)h.w << 32);
            const u32 e = (u32)((c < 2 ? a : b) >> ((c < 2 ? 26u : 16u) + 19u * (c & 1u))) & 0x7FFFFu;
            const u32 l = (e >> 4) & 63u;
            const u32 adr = uaddr(hv, (e >> 3) & 1u, l & 15u, l >> 4) + 2u * (e & 7u) + 1u;
            if (lane < 40 && mode != 7u && c < mode) st[adr] = (unsigned char)(e >> 11);
        }
        lds_signal(cnt + 16 + sg);
        const u64 q2 = CT(); CACC(c_dec, q1, q2);
    };
    PayA ya, yb; u32x4 ha, hb, sn;
    ha = ldh(dw);
    wait_hdr(ha, ws, hoff + dw * 2048, seq16, wt); if (dw == 0) STAMP(ws, 1);
    sn = ldh(dw + 4); ldp(ya, dw);
#define DEC_INST(u, hc, hn, yc, yn) do { \
        const int t4 = min((u) + 4, NT - 1), t8 = min((u) + 8, NT - 1); \
        const u32 nf = freec(u); \
        if (HDR_READY(sn)) { const u64 i0 = CT(); hn = sn; sn = ldh(t8); ldp(yn, t4); const u64 i1 = CT(); CACC(c_iss, i0, i1); dec(yc, hc, (u), nf); asm volatile("; early" ::: "memory"); } \
        else { dec(yc, hc, (u), nf); wait_hdr(sn, ws, hoff + t4 * 2048, seq16, wt); hn = sn; sn = ldh(t8); ldp(yn, t4); asm volatile("; late" ::: "memory"); } } while (0)
#pragma unroll 1
    for (int u = dw; u < NT; u += 8) {
        DEC_INST(u, ha, hb, ya, yb);
        DEC_INST(u + 4, hb, ha, yb, ya);
    }
#undef DEC_INST
    if (dw == 3) STAMP(ws, 5);
}

__device__ __forceinline__ void mma_wave(const Params& p, LAS unsigned char* lds, int tm, int tn, int wv, int lane, Wait& wt, u32 seq16) {
    char* const ws = p.ws[p.rank];
    const __amdgpu_buffer_rsrc_t rs = __builtin_amdgcn_make_buffer_rsrc(ws, 0, (int)WS_NEED, 0x00020000);
    const u32 r = lane & 31, hh = lane >> 5, fl = (r & 7u) ^ ((r >> 3) & 3u);
    u32 ak[4];
#pragma unroll
    for (int k = 0; k < 4; ++k) ak[k] = r * 128u + (((2u * k + hh) ^ fl) << 4);
    f32x16 acc[5];
#pragma unroll
    for (int m = 0; m < 5; ++m)
#pragma unroll
        for (int e = 0; e < 16; ++e) acc[m][e] = 0.f;
    LAS u32* const cnt = (LAS u32*)(lds + LDS_MISC);
    const u32 src = tn >> 1, slab = 4 * (tn & 1) + wv;
    const u32 pB = OFF_IN + src * IN_SRC + IN_B + slab * PKT, oB = OFF_OVF + src * (1024u << 10) + (512u << 10) + 2u * slab * 1024u + lane * 16u;
    const u32 hoff = OFF_HB + (16u * src + 2u * slab + min((u32)lane >> 2, 1u)) * 16u;
    struct PayB { u32x4 l0, l1, n; };
    auto ldh = [&](int t) -> u32x4 { return __builtin_amdgcn_raw_buffer_load_b128(rs, hoff, t * 2048, 17); };
    auto ldp = [&](PayB& y, int t) {
        y.l0 = __builtin_amdgcn_raw_buffer_load_b128(rs, lane * 16, pB + t * IN_T, 17);
        y.l1 = __builtin_amdgcn_raw_buffer_load_b128(rs, lane * 16 + 1024, pB + t * IN_T, 17);
        y.n = __builtin_amdgcn_raw_buffer_load_b128(rs, lane * 16 + 2048, pB + t * IN_T, 17);
    };
    u32 nfull = 0;
    u64 c_b = 0, c_full = 0, c_mm = 0, c_iss = 0;
    auto ungrp = [&](u32 (&d)[16], const PayB& y, int g, u32 b0, u32 b1) {
        const u32x4 r = g == 0 ? decode_group(y.l0.x, y.l0.y, y.n.x, b0) : g == 1 ? decode_group(y.l0.z, y.l0.w, y.n.y, b0) : g == 2 ? decode_group(y.l1.x, y.l1.y, y.n.z, b1) : decode_group(y.l1.z, y.l1.w, y.n.w, b1);
        d[4 * g] = r.x; d[4 * g + 1] = r.y; d[4 * g + 2] = r.z; d[4 * g + 3] = r.w;
    };
    auto fix = [&](u32 (&d)[16], const PayB& y, const u32x4& h, u32 m0, u32 m1, int u) {
        if (m0 == 7u) { const u32x4 rw = ld16_sync(ws, oB + u * 16384); d[0] = y.l0.x; d[1] = y.l0.y; d[2] = y.l0.z; d[3] = y.l0.w; d[4] = rw.x; d[5] = rw.y; d[6] = rw.z; d[7] = rw.w; }
        else if (m0) patch_regs(d, h, 0, m0, lane);
        if (m1 == 7u) { const u32x4 rw = ld16_sync(ws, oB + u * 16384 + 1024); d[8] = y.l1.x; d[9] = y.l1.y; d[10] = y.l1.z; d[11] = y.l1.w; d[12] = rw.x; d[13] = rw.y; d[14] = rw.z; d[15] = rw.w; }
        else if (m1) patch_regs(d + 8, h, 4, m1, lane);
    };
    bf16x8 a0[5], a1[5];
#define SB() __builtin_amdgcn_sched_barrier(0)
#define RD(dst, st, k) do { _Pragma("unroll") for (int m = 0; m < 5; ++m) dst[m] = *(const LAS bf16x8*)((st) + ak[k] + m * 4096); SB(); } while (0)
#define WG5() do { __builtin_amdgcn_s_waitcnt(0xC57F); SB(); } while (0)
#define M1(d, k, src, m) do { acc[m] = __builtin_amdgcn_mfma_f32_32x32x16_bf16(__builtin_bit_cast(bf16x8, (u32x4){d[4 * k], d[4 * k + 1], d[4 * k + 2], d[4 * k + 3]}), src[m], acc[m], 0, 0, 0); SB(); } while (0)
#define UH(g, pp, b0, b1) do { const u32 n_ = g == 0 ? pp.n.x : g == 1 ? pp.n.y : g == 2 ? pp.n.z : pp.n.w, br_ = g < 2 ? b0 : b1; \
                              uha = (n_ & 0x87878787u) + br_; uhb = (__builtin_amdgcn_alignbit(n_, n_, 4) & 0x87878787u) + br_; asm volatile("" : "+v"(uha), "+v"(uhb)); SB(); } while (0)
#define UP(d, g, pp) do { const u32 la_ = g == 0 ? pp.l0.x : g == 1 ? pp.l0.z : g == 2 ? pp.l1.x : pp.l1.z, lb_ = g == 0 ? pp.l0.y : g == 1 ? pp.l0.w : g == 2 ? pp.l1.y : pp.l1.w; \
                         d[4 * g] = __builtin_amdgcn_perm(uha, la_, 0x05010400u); d[4 * g + 1] = __builtin_amdgcn_perm(uha, la_, 0x07030602u); \
                         d[4 * g + 2] = __builtin_amdgcn_perm(uhb, lb_, 0x05010400u); d[4 * g + 3] = __builtin_amdgcn_perm(uhb, lb_, 0x07030602u); \
                         asm volatile("" : "+v"(d[4 * g]), "+v"(d[4 * g + 1]), "+v"(d[4 * g + 2]), "+v"(d[4 * g + 3])); SB(); } while (0)
    u32 uha, uhb;
    PayB y0, y1, y2, y3; u32x4 h0, h1, h2, h3, s0, s1; u32 da[16], db[16];
    h0 = ldh(0); h1 = ldh(1); h2 = ldh(2); h3 = ldh(3);
    wait_hdr(h0, ws, hoff, seq16, wt); ldp(y0, 0);
    wait_hdr(h1, ws, hoff + 2048, seq16, wt); ldp(y1, 1);
    wait_hdr(h2, ws, hoff + 4096, seq16, wt); s0 = ldh(4); ldp(y2, 2);
    wait_hdr(h3, ws, hoff + 6144, seq16, wt); s1 = ldh(5); ldp(y3, 3);
    {
        const u32 w0 = rdl(h0.x, 0), w1 = rdl(h0.x, 4), b0 = ((w0 >> 16) & 0x7Fu) * 0x01010101u, b1 = ((w1 >> 16) & 0x7Fu) * 0x01010101u;
        ungrp(da, y0, 0, b0, b1); ungrp(da, y0, 1, b0, b1); ungrp(da, y0, 2, b0, b1); ungrp(da, y0, 3, b0, b1);
        if (((w0 | w1) >> 23) & 7u) fix(da, y0, h0, (w0 >> 23) & 7u, (w1 >> 23) & 7u, 0);
    }
    wait_hdr(s0, ws, hoff + 4 * 2048, seq16, wt);
    lds_wait(cnt + 16, 1u);
    RD(a0, lds, 0);
    u32 sg = 0, turn = 0;
#define R1(dst, st, k, m) do { dst[m] = *(const LAS bf16x8*)((st) + ak[k] + m * 4096); SB(); } while (0)
#define MMA_STEP(u, dc, dn, y1_, h1_, y0_, h0_, sn, so) do { \
        const int t4 = min((u) + 4, NT - 1), t5 = min((u) + 5, NT - 1), t6 = min((u) + 6, NT - 1), t1 = min((u) + 1, NT - 1); \
        const bool lastu = (u) + 1 >= NT, wrap = sg == NSTAGE - 1; \
        const u32 sg1 = lastu ? sg : wrap ? 0u : sg + 1u, turn1 = (!lastu && wrap) ? turn + 1u : turn; \
        LAS unsigned char* st = lds + sg * STAGE; \
        const u32 so4 = pB + t4 * IN_T; \
        h0_ = sn; SB(); \
        M1(dc, 0, a0, 0); R1(a1, st, 1, 0); sn = ldh(t6); SB(); \
        M1(dc, 0, a0, 1); R1(a1, st, 1, 1); y0_.l0 = __builtin_amdgcn_raw_buffer_load_b128(rs, lane * 16, so4, 17); SB(); \
        M1(dc, 0, a0, 2); R1(a1, st, 1, 2); y0_.l1 = __builtin_amdgcn_raw_buffer_load_b128(rs, lane * 16 + 1024, so4, 17); SB(); \
        M1(dc, 0, a0, 3); R1(a1, st, 1, 3); y0_.n = __builtin_amdgcn_raw_buffer_load_b128(rs, lane * 16 + 2048, so4, 17); SB(); \
        M1(dc, 0, a0, 4); R1(a1, st, 1, 4); \
        const u32 w0 = rdl(h1_.x, 0), w1 = rdl(h1_.x, 4); \
        const u32 b0 = ((w0 >> 16) & 0x7Fu) * 0x01010101u, b1 = ((w1 >> 16) & 0x7Fu) * 0x01010101u; SB(); \
        M1(dc, 1, a1, 0); R1(a0, st, 2, 0); UH(0, y1_, b0, b1); M1(dc, 1, a1, 1); R1(a0, st, 2, 1); UP(dn, 0, y1_); M1(dc, 1, a1, 2); R1(a0, st, 2, 2); UH(1, y1_, b0, b1); \
        M1(dc, 1, a1, 3); R1(a0, st, 2, 3); UP(dn, 1, y1_); M1(dc, 1, a1, 4); R1(a0, st, 2, 4); \
        nfull = __hip_atomic_load(cnt + 16 + sg1, __ATOMIC_RELAXED, __HIP_MEMORY_SCOPE_WORKGROUP); SB(); \
        M1(dc, 2, a0, 0); R1(a1, st, 3, 0); UH(2, y1_, b0, b1); M1(dc, 2, a0, 1); R1(a1, st, 3, 1); UP(dn, 2, y1_); M1(dc, 2, a0, 2); R1(a1, st, 3, 2); UH(3, y1_, b0, b1); \
        M1(dc, 2, a0, 3); R1(a1, st, 3, 3); UP(dn, 3, y1_); M1(dc, 2, a0, 4); R1(a1, st, 3, 4); \
        lds_signal(cnt + 24 + sg); SB(); \
          \
        M1(dc, 3, a1, 0); \
        if (rfl(nfull) < turn1 + 1u) lds_wait(cnt + 16 + sg1, turn1 + 1u); \
        SB(); \
        LAS unsigned char* st1 = lds + sg1 * STAGE; \
        M1(dc, 3, a1, 1); R1(a0, st1, 0, 0); R1(a0, st1, 0, 1); \
        M1(dc, 3, a1, 2); R1(a0, st1, 0, 2); R1(a0, st1, 0, 3); \
        wait_hdr(so, ws, hoff + t5 * 2048, seq16, wt); \
        SB(); \
        M1(dc, 3, a1, 3); R1(a0, st1, 0, 4); \
        if (((w0 | w1) >> 23) & 7u) fix(dn, y1_, h1_, (w0 >> 23) & 7u, (w1 >> 23) & 7u, t1); \
        SB(); \
        M1(dc, 3, a1, 4); \
        sg = sg1; turn = turn1; } while (0)
#pragma unroll 1
    for (int u = 0; u < NT; u += 4) {
        MMA_STEP(u, da, db, y1, h1, y0, h0, s0, s1);
        MMA_STEP(u + 1, db, da, y2, h2, y1, h1, s1, s0);
        MMA_STEP(u + 2, da, db, y3, h3, y2, h2, s0, s1);
        MMA_STEP(u + 3, db, da, y0, h0, y3, h3, s1, s0);
    }
#undef UH
#undef WG5
#undef R1
#undef UP
#undef M1
#undef SB
#undef MMA_STEP
#undef RD
    STAMP(ws, 6);
    char* const dst = p.ws[src & 7];
    const u32 r32 = 5 * tm;
    const u32 ubase = OFF_Y + (u32)p.rank * Y_SRC + (slab * 64 + r32) * 2048u;
    u32x4 myhdr = (u32x4){0, 0, 0, 0};
#pragma unroll
    for (int m = 0; m < 5; ++m) {
        if (r32 + m < 64) {
            u32 d[8]; float mx = 0.f;
#pragma unroll
            for (int q = 0; q < 8; ++q) {
                d[q] = cvt_pk_bf16(acc[m][2 * q], acc[m][2 * q + 1]);
                mx = fmaxf(mx, fmaxf(fabsf(acc[m][2 * q]), fabsf(acc[m][2 * q + 1])));
            }
            u32x4 L, hdr; u32x2 N;
            pack_unit(d, mx, seq16, L, N, hdr);
            char* q = dst + ubase + m * 2048u;
            if (((hdr.x >> 23) & 7u) != 7u) { st16_sys(q + lane * 16, L); st8_sys(q + 1024 + lane * 8, N); }
            else { st16_sys(q + lane * 16, (u32x4){d[0], d[1], d[2], d[3]}); st16_sys(q + 1024 + lane * 16, (u32x4){d[4], d[5], d[6], d[7]}); }
            if (lane == m) myhdr = hdr;
        }
    }
    STAMP(ws, 7);
    drain();
    if (lane < 5 && r32 + lane < 64) st16_sys(dst + OFF_HY + (((u32)p.rank * 8 + slab) * 64 + r32 + lane) * 16u, myhdr);
    STAMP(ws, 8);
}

__global__ void __launch_bounds__(NTHR, 2) k_main(Params p) {
    extern __shared__ __attribute__((aligned(16))) unsigned char smem[];
    LAS unsigned char* lds = (LAS unsigned char*)smem;
    const int b = blockIdx.x, wv = rfl(threadIdx.x >> 6), lane = threadIdx.x & 63;
    char* const ws = p.ws[p.rank];
    if (wv == 0) STAMP(ws, 0);
    GAS u32* cntp = (GAS u32*)(ws + OFF_CNT + b * 128);
    const u32 callv = __hip_atomic_load(cntp, __ATOMIC_RELAXED, __HIP_MEMORY_SCOPE_AGENT);
    const int xg = b & 7, slot = b >> 3;
    if (slot < NPS && wv < 2 * PW) {
        const int gw = (wv >> 1) * (8 * NPS) + slot * 8 + xg;
        pusher(p, lds, wv, gw, lane, callv);
    }
    const u32 call = rfl(callv) + 1u;
    const u32 seq16 = call & 0xFFFFu;
    Wait wt{(GAS u32*)(ws + OFF_TMO), call, false};
    if (ROLE_OF(b) == 1) {
        const int s = slot - 6, tm = s >> 1, tn = 2 * xg + (s & 1);
        if (threadIdx.x < 16) ((LAS u32*)(lds + LDS_MISC))[16 + threadIdx.x] = 0u;
        BARRIER();
        if (wv >= 4) decoder(p, lds, tm, wv - 4, lane, wt, seq16); else mma_wave(p, lds, tm, tn, wv, lane, wt, seq16);
    }
    const __amdgpu_buffer_rsrc_t rs = __builtin_amdgcn_make_buffer_rsrc(ws, 0, (int)WS_NEED, 0x00020000);
    const u32 ri = b >> 7, rj = (b >> 6) & 1, r32 = b & 63, cg8 = wv;
    const u32 snd0 = (ri << 2) | (rj << 1);
    float y[16];
    LAS u32* misc = (LAS u32*)(lds + LDS_MISC);
    {
        const u32 hoff = OFF_HY + (((snd0 + min((u32)lane >> 2, 1u)) * 8 + cg8) * 64 + r32) * 16u;
        u32x4 h = __builtin_amdgcn_raw_buffer_load_b128(rs, hoff, 0, 17);
        if (!hdr_ok(h, seq16)) { const u64 t0 = rtime(); u32 n = 0; do { if (spin_on(wt, t0, n)) break; h = ld16_sync(ws, hoff); } while (!hdr_ok(h, seq16)); }
        STAMP(ws, 24 + wv);
        u32x4 l[2]; u32x2 n[2];
#pragma unroll
        for (int s = 0; s < 2; ++s) {
            const u32 ub = OFF_Y + (snd0 + s) * Y_SRC + (cg8 * 64 + r32) * 2048u;
            l[s] = __builtin_amdgcn_raw_buffer_load_b128(rs, lane * 16, ub, 17);
            n[s] = __builtin_amdgcn_raw_buffer_load_b64(rs, lane * 8 + 1024, ub, 17);
        }
#pragma unroll
        for (int s = 0; s < 2; ++s) {
            const u32 w0 = rdl(h.x, 4 * s), mode = (w0 >> 23) & 7u;
            u32 d[8];
            if (mode != 7u) {
                const u32 brep = ((w0 >> 16) & 0x7Fu) * 0x01010101u;
                const u32x4 g0 = decode_group(l[s].x, l[s].y, n[s].x, brep), g1 = decode_group(l[s].z, l[s].w, n[s].y, brep);
                d[0] = g0.x; d[1] = g0.y; d[2] = g0.z; d[3] = g0.w; d[4] = g1.x; d[5] = g1.y; d[6] = g1.z; d[7] = g1.w;
                if (mode) patch_regs(d, h, 4 * s, mode, lane);
            } else {
                const u32 ub = OFF_Y + (snd0 + s) * Y_SRC + (cg8 * 64 + r32) * 2048u;
                const u32x4 hi = ld16_sync(ws, ub + lane * 16 + 1024);
                d[0] = l[s].x; d[1] = l[s].y; d[2] = l[s].z; d[3] = l[s].w; d[4] = hi.x; d[5] = hi.y; d[6] = hi.z; d[7] = hi.w;
            }
#pragma unroll
            for (int q = 0; q < 8; ++q) {
                const float lo = __uint_as_float(d[q] << 16), hi = __uint_as_float(d[q] & 0xFFFF0000u);
                if (s == 0) { y[2 * q] = lo; y[2 * q + 1] = hi; } else { y[2 * q] += lo; y[2 * q + 1] += hi; }
            }
        }
        float mx = 0.f;
#pragma unroll
        for (int q = 0; q < 16; ++q) mx = fmaxf(mx, fabsf(y[q]));
        const u32 wm = wave_max_u32(__float_as_uint(mx));
        if (lane == 0) misc[wv] = wm;
        STAMPD(ws, 16 + wv);
    }
    BARRIER();
    if (wv == 0) {
        const u32 v = wave_max_u32(misc[lane & 7]);
        if (lane == 0) __hip_atomic_store((GAS u64*)(ws + OFF_LMAX + b * 64), ((u64)call << 32) | v, __ATOMIC_RELAXED, __HIP_MEMORY_SCOPE_AGENT);
        STAMP(ws, 11);
        const u32 go = OFF_GMAX + ((b & 7) * 8 + (lane & 7)) * 64;
        u32x2 g = ld8_sync(ws, go);
        if (__ballot((int)(g.y - call) >= 0) != ~0ull) { const u64 t0 = rtime(); u32 n = 0; do { if (spin_on<0>(wt, t0, n)) break; g = ld8_sync(ws, go); } while (__ballot((int)(g.y - call) >= 0) != ~0ull); }
        const u32 gm = wave_max_u32(g.x);
        if (lane == 0) misc[8] = gm;
    } else if (wv <= NLEAD && b == LEADER) {
        if (wv == 2) __builtin_amdgcn_s_sleep(6); else if (wv == 3) __builtin_amdgcn_s_sleep(12);
        u32x2 g0, g1, g2, g3;
        auto poll = [&]() {
            asm volatile("global_load_dwordx2 %0, %4, %5 sc1\n\tglobal_load_dwordx2 %1, %4, %5 offset:64 sc1\n\tglobal_load_dwordx2 %2, %4, %5 offset:128 sc1\n\tglobal_load_dwordx2 %3, %4, %5 offset:192 sc1\n\ts_waitcnt vmcnt(0)"
                         : "=&v"(g0), "=&v"(g1), "=&v"(g2), "=&v"(g3) : "v"(OFF_LMAX + lane * 256), "s"(ws) : "memory");
            return __ballot((int)((g0.y - call) | (g1.y - call) | (g2.y - call) | (g3.y - call)) >= 0) == ~0ull; };
        if (!poll()) { const u64 t0 = rtime(); u32 n = 0; do { if (spin_on<0>(wt, t0, n)) break; } while (!poll()); }
        const u32 lm = wave_max_u32(max(max(g0.x, g1.x), max(g2.x, g3.x)));
        __hip_atomic_store((GAS u64*)(p.ws[lane >> 3] + OFF_GMAX + ((lane & 7) * 8 + (u32)p.rank) * 64), ((u64)call << 32) | lm, __ATOMIC_RELAXED, __HIP_MEMORY_SCOPE_SYSTEM);
        STAMP(ws, 14);
    }
    BARRIER();
    if (wv == 0) STAMP(ws, 12);
    {
        const float amax = __uint_as_float(misc[8]);
        const bool bad = rfl(__hip_atomic_load(wt.tmo, __ATOMIC_RELAXED, __HIP_MEMORY_SCOPE_AGENT)) == call;
        const float inv = amax > 0.f ? 448.0f / amax : 0.f;
        u32 o[4];
#pragma unroll
        for (int q = 0; q < 4; ++q) {
            float v[4];
#pragma unroll
            for (int e = 0; e < 4; ++e) v[e] = __builtin_amdgcn_fmed3f(y[4 * q + e] * inv, -448.f, 448.f);
            int t = __builtin_amdgcn_cvt_pk_fp8_f32(v[0], v[1], 0, false); t = __builtin_amdgcn_cvt_pk_fp8_f32(v[2], v[3], t, true); o[q] = (u32)t;
        }
        const u32 R = r32 * 32 + (lane & 31);
        u32 sr, sl; mt_split(R >> 4, sr, sl);
        const u32 row = 512 * sr + 256 * ri + 16 * sl + (R & 15), col = 256 * rj + 32 * cg8 + 16 * (lane >> 5);
        const u32x4 ov = bad ? (u32x4){0x7f7f7f7fu, 0x7f7f7f7fu, 0x7f7f7f7fu, 0x7f7f7f7fu} : (u32x4){o[0], o[1], o[2], o[3]};
        *(GAS u32x4*)(p.out + (size_t)row * SH + col) = ov;
    }
    if (threadIdx.x == 0) __hip_atomic_store(cntp, call, __ATOMIC_RELAXED, __HIP_MEMORY_SCOPE_AGENT);
    if (wv == 0) STAMP(ws, 13);
}

extern "C" void kernel_launch(int rank, int nranks, ncclComm_t comm, hipStream_t stream,
                              void* const* d_in, const int* in_sizes, int n_in,
                              void* d_out, int out_size, const int* in_dtypes, int out_dtype,
                              void* const* d_out_all, void* const* d_ws_all, size_t ws_bytes) {
    if (nranks != NR || (unsigned)rank >= (unsigned)NR || ws_bytes < WS_NEED || n_in < 2 || in_sizes[0] != SH * KG || in_sizes[1] != KG * SH || out_size != MG * SH) {
        fprintf(stderr, "kernel_launch: unexpected shapes\n"); return; }
    HIP_CHECK(hipFuncSetAttribute((const void*)k_main, hipFuncAttributeMaxDynamicSharedMemorySize, LDS_BYTES));
    Params p;
    p.x = (const float*)d_in[0]; p.w = (const float*)d_in[1]; p.out = (unsigned char*)d_out;
    for (int i = 0; i < NR; ++i) p.ws[i] = (char*)d_ws_all[i];
    p.rank = rank; p.pad = 0;
    k_main<<<256, NTHR, LDS_BYTES, stream>>>(p);
    HIP_CHECK(hipGetLastError());
}
```

```cpp
#include <hip/hip_runtime.h>
#include <rccl/rccl.h>
#include <cstdio>
#include <cstdint>

#define HIP_CHECK(expr) do { hipError_t _e = (expr); if (_e != hipSuccess) { \
    fprintf(stderr, "HIP error %s at %s:%d\n", hipGetErrorString(_e), __FILE__, __LINE__); \
    return; } } while (0)

typedef unsigned u32;
typedef unsigned long long u64;
typedef u32 u32x4 __attribute__((ext_vector_type(4)));
typedef u32 u32x2 __attribute__((ext_vector_type(2)));
typedef float f32x4 __attribute__((ext_vector_type(4)));
typedef float f32x16 __attribute__((ext_vector_type(16)));
typedef short bf16x8 __attribute__((ext_vector_type(8)));
#define LAS __attribute__((address_space(3)))
#define GAS __attribute__((address_space(1)))

constexpr int NR = 8, MG = 4096, KG = 4096, SH = 512;
constexpr int NT = 32;
constexpr int NTHR = 512;
constexpr u64 DEADLINE = 150000000ull;
constexpr u32 PKT = 3072;
constexpr u32 OFF_CNT = 0;
constexpr u32 OFF_TMO = 0x8000;
constexpr u32 OFF_TICK = 0x8100;
constexpr u32 OFF_LMAX = 0x10000;
constexpr u32 OFF_GMAX = 0x14000;
constexpr u32 OFF_HA = 0x30000;
constexpr u32 OFF_HB = 0x40000;
constexpr u32 OFF_HY = 0x50000;
constexpr u32 OFF_IN = 2u << 20;
constexpr u32 IN_SRC = 2u << 20, IN_B = 768u << 10, IN_T = 8 * PKT;
constexpr u32 OFF_Y = 18u << 20;
constexpr u32 Y_SRC = 1u << 20;
constexpr u32 OFF_OVF = 26u << 20;
constexpr size_t WS_NEED = 34u << 20;
constexpr int STAGE = 20480, NSTAGE = 6, LDS_MISC = NSTAGE * STAGE, LDS_BYTES = 124 * 1024;
constexpr int LEADER = 0;

struct Params { const float* x; const float* w; unsigned char* out; char* ws[NR]; int rank; int pad; };

__device__ __forceinline__ u32 cvt_pk_bf16(float lo, float hi) { u32 r; asm("v_cvt_pk_bf16_f32 %0, %1, %2" : "=v"(r) : "v"(lo), "v"(hi)); return r; }
__device__ __forceinline__ void st16_sys(void* p, u32x4 v) { asm volatile("global_store_dwordx4 %0, %1, off sc0 sc1\n\ts_nop 1" :: "v"(p), "v"(v) : "memory"); }
__device__ __forceinline__ void st8_sys(void* p, u32x2 v) { asm volatile("global_store_dwordx2 %0, %1, off sc0 sc1\n\ts_nop 1" :: "v"(p), "v"(v) : "memory"); }
__device__ __forceinline__ u32x4 ld16_sync(const char* base, u32 off) { u32x4 r; asm volatile("global_load_dwordx4 %0, %1, %2 sc0 sc1\n\ts_waitcnt vmcnt(0)" : "=&v"(r) : "v"(off), "s"(base) : "memory"); return r; }
__device__ __forceinline__ u32x2 ld8_sync(const char* base, u32 off) { u32x2 r; asm volatile("global_load_dwordx2 %0, %1, %2 sc0 sc1\n\ts_waitcnt vmcnt(0)" : "=&v"(r) : "v"(off), "s"(base) : "memory"); return r; }
__device__ __forceinline__ u32 ld4_sync_agent(const void* p) { u32 r; asm volatile("global_load_dword %0, %1, off sc1\n\ts_waitcnt vmcnt(0)" : "=&v"(r) : "v"(p) : "memory"); return r; }
__device__ __forceinline__ void drain() { asm volatile("s_waitcnt vmcnt(0)" ::: "memory"); }
__device__ __forceinline__ u64 rtime() { return __builtin_amdgcn_s_memrealtime(); }
__device__ __forceinline__ u32 rfl(u32 v) { return __builtin_amdgcn_readfirstlane(v); }
__device__ __forceinline__ u32 rdl(u32 v, int l) { return __builtin_amdgcn_readlane(v, l); }
__device__ __forceinline__ u32 wave_max_u32(u32 v) {
    v = max(v, (u32)__builtin_amdgcn_update_dpp(0, (int)v, 0x111, 0xf, 0xf, false));
    v = max(v, (u32)__builtin_amdgcn_update_dpp(0, (int)v, 0x112, 0xf, 0xf, false));
    v = max(v, (u32)__builtin_amdgcn_update_dpp(0, (int)v, 0x114, 0xf, 0xf, false));
    v = max(v, (u32)__builtin_amdgcn_update_dpp(0, (int)v, 0x118, 0xf, 0xf, false));
    v = max(v, (u32)__builtin_amdgcn_update_dpp(0, (int)v, 0x142, 0xa, 0xf, false));
    v = max(v, (u32)__builtin_amdgcn_update_dpp(0, (int)v, 0x143, 0xc, 0xf, false));
    return rdl(v, 63);
}
#define BARRIER() do { asm volatile("s_waitcnt lgkmcnt(0)" ::: "memory"); __builtin_amdgcn_s_barrier(); asm volatile("" ::: "memory"); } while (0)

constexpr int NPS = 5;

__device__ __forceinline__ void pack_unit(const u32* d, float amax_lane, u32 seq16, u32x4& L, u32x2& N, u32x4& hdr) {
    u32 H[4];
#pragma unroll
    for (int q = 0; q < 4; ++q) { L[q] = __builtin_amdgcn_perm(d[2 * q + 1], d[2 * q], 0x06040200u); H[q] = __builtin_amdgcn_perm(d[2 * q + 1], d[2 * q], 0x07050301u); }
    const u32 mb = wave_max_u32(__float_as_uint(amax_lane));
    u32 h7 = (mb + 0x7FFFu + ((mb >> 16) & 1u)) >> 24;
    h7 = h7 > 127u ? 127u : h7;
    const u32 base = h7 > 7u ? h7 - 7u : 0u, brep = base * 0x01010101u;
    u32 b[4], X[4];
#pragma unroll
    for (int q = 0; q < 4; ++q) { const u32 t = (H[q] | 0x80808080u) - brep; b[q] = (t ^ 0x80808080u) & 0xF8F8F8F8u; X[q] = (H[q] & 0x80808080u) | (t & 0x07070707u); }
    N[0] = X[0] | __builtin_amdgcn_alignbit(X[1], X[1], 28); N[1] = X[2] | __builtin_amdgcn_alignbit(X[3], X[3], 28);
    u32 cnt = 0; u64 ea = 0, eb = 0;
    u64 m = __ballot((b[0] | b[1] | b[2] | b[3]) != 0);
    if (m) {
        u32 bm = 0;
#pragma unroll
        for (int q = 0; q < 4; ++q) bm |= (((((((b[q] & 0x7F7F7F7Fu) + 0x7F7F7F7Fu) | b[q]) & 0x80808080u) >> 7) * 0x01020408u) >> 24 & 0xFu) << (4 * q);
        while (m && cnt <= 4) {
            const int l = __builtin_ctzll(m); m &= m - 1;
            u32 mk = rdl(bm, l);
            const u32 h0 = rdl(H[0], l), h1 = rdl(H[1], l), h2 = rdl(H[2], l), h3 = rdl(H[3], l);
            while (mk) {
                const u32 j = (u32)__builtin_ctz(mk); mk &= mk - 1;
                const u32 hq = (j & 8u) ? ((j & 4u) ? h3 : h2) : ((j & 4u) ? h1 : h0);
                const u64 e = (u64)((u32)l * 16u + j) | ((u64)((hq >> (8u * (j & 3u))) & 0xFFu) << 11);
                if (cnt < 2) ea |= e << (26u + 19u * cnt); else if (cnt < 4) eb |= e << (16u + 19u * (cnt - 2u));
                ++cnt;
            }
        }
    }
    const u32 mode = cnt > 4 ? 7u : cnt;
    ea |= (u64)seq16 | ((u64)base << 16) | ((u64)mode << 23); eb |= (u64)seq16;
    hdr = (u32x4){(u32)ea, (u32)(ea >> 32), (u32)eb, (u32)(eb >> 32)};
}
__device__ __forceinline__ u32x4 decode_group(u32 LA, u32 LB, u32 n, u32 brep) {
    const u32 HA = (n & 0x87878787u) + brep, HB = (__builtin_amdgcn_alignbit(n, n, 4) & 0x87878787u) + brep;
    return (u32x4){__builtin_amdgcn_perm(HA, LA, 0x05010400u), __builtin_amdgcn_perm(HA, LA, 0x07030602u),
                   __builtin_amdgcn_perm(HB, LB, 0x05010400u), __builtin_amdgcn_perm(HB, LB, 0x07030602u)};
}
__device__ __forceinline__ u32 hdr_exc(u64 a, u64 b, u32 c) { return (u32)((c < 2 ? a >> (26u + 19u * c) : b >> (16u + 19u * (c - 2u)))) & 0x7FFFFu; }
__device__ __forceinline__ bool hdr_ok(const u32x4& h, u32 seq16) { return __ballot((((h.x - seq16) | (h.z - seq16)) & 0x8000u) == 0u) == ~0ull; }
__device__ __forceinline__ void patch_regs(u32* d, const u32x4& h, int hl, u32 mode, int lane) {
    const u64 a = (u64)rdl(h.x, hl) | ((u64)rdl(h.y, hl) << 32), b = (u64)rdl(h.z, hl) | ((u64)rdl(h.w, hl) << 32);
    for (u32 c = 0; c < mode; ++c) {
        const u32 e = hdr_exc(a, b, c), el = (e >> 4) & 63u, j = e & 15u, hb = e >> 11, sh = (j & 1u) * 16u + 8u;
#pragma unroll
        for (int q = 0; q < 8; ++q) if (j >> 1 == (u32)q && (u32)lane == el) d[q] = (d[q] & ~(0xFFu << sh)) | (hb << sh);
    }
}

__device__ __forceinline__ void lds_signal(LAS u32* c) {
    asm volatile("s_mov_b64 exec, 1\n\tds_add_u32 %0, %1\n\ts_mov_b64 exec, -1" :: "v"((u32)(uintptr_t)c), "v"(1u) : "memory");
}
__device__ __forceinline__ void lds_wait(LAS u32* c, u32 target) {
    while (rfl(__hip_atomic_load(c, __ATOMIC_RELAXED, __HIP_MEMORY_SCOPE_WORKGROUP)) < target) __builtin_amdgcn_s_sleep(1);
    asm volatile("" ::: "memory");
}

struct Wait { GAS u32* tmo; u32 call; bool dead; };
template <int SLEEP = 1>
__device__ __forceinline__ bool spin_on(Wait& wt, u64 t0, u32& n) {
    if (wt.dead) return true;
    if ((++n & 31u) == 0u) {
        if (rfl(ld4_sync_agent((const void*)wt.tmo)) == wt.call || rtime() - t0 > DEADLINE) {
            __hip_atomic_store(wt.tmo, wt.call, __ATOMIC_RELAXED, __HIP_MEMORY_SCOPE_AGENT); wt.dead = true; return true; }
    }
    if (SLEEP) __builtin_amdgcn_s_sleep(SLEEP);
    return false;
}
__device__ __forceinline__ void wait_hdr(u32x4& h, const char* ws, u32 off, u32 seq16, Wait& wt) {
    if (!hdr_ok(h, seq16)) { const u64 t0 = rtime(); u32 n = 0; do { if (spin_on(wt, t0, n)) break; h = ld16_sync(ws, off); } while (!hdr_ok(h, seq16)); }
}

constexpr int PW = 2;
constexpr int SKEW = 12;
constexpr u32 NA = 4;
constexpr int WLEAD = 4;
constexpr u32 PWAVES = 8 * NPS * PW;
struct Pkt { u32 isw, t, h, kh, blk; };
__device__ __forceinline__ Pkt ticket_packet(u32 n, u32 pool) {
    const u32 tau = min(2u * n + pool, 2047u);
    constexpr u32 XA = 4u * NA, XB = 32u - XA;
    constexpr u32 P0 = 32u * WLEAD, P1 = P0 + (32u + XA) * SKEW, P2 = P1 + 64u * (32u - WLEAD - SKEW), P3 = P2 + 32u * WLEAD;
    static_assert(WLEAD + SKEW <= 32 && P3 + XB * SKEW == 2048u, "slot layout");
    int j; u32 c;
    if (tau < P0) { j = (int)(tau >> 5) - WLEAD; c = tau & 31u; }
    else if (tau < P1) { const u32 q = (tau - P0) / (32u + XA); j = (int)q; c = tau - P0 - (32u + XA) * q; }
    else if (tau < P2) { j = SKEW + (int)((tau - P1) >> 6); c = (tau - P1) & 63u; }
    else if (tau < P3) { j = 32 - WLEAD + (int)((tau - P2) >> 5); c = 32u + ((tau - P2) & 31u); }
    else { const u32 q = (tau - P3) / XB; j = 32 + (int)q; c = 32u + XA + (tau - P3 - XB * q); }
    Pkt k;
    k.isw = c < 32u;
    k.t = (u32)(c < 32u ? j + WLEAD : c < 32u + XA ? j : j - SKEW);
    if (c < 32u) { k.h = c >> 4; k.kh = (c >> 3) & 1u; k.blk = c & 7u; }
    else if (c < 32u + XA) { const u32 i = c - 32u, q4 = i / NA; k.h = q4 >> 1; k.kh = q4 & 1u; k.blk = i - q4 * NA; }
    else { const u32 i = c - 32u - XA, q4 = i / (8u - NA); k.h = q4 >> 1; k.kh = q4 & 1u; k.blk = NA + (i - q4 * (8u - NA)); }
    return k;
}
__device__ __forceinline__ u32 mt_index(u32 src, u32 slot) { return slot < 2u * NA ? src * (2u * NA) + slot : 16u * NA + src * (16u - 2u * NA) + (slot - 2u * NA); }
__device__ __forceinline__ void mt_split(u32 x, u32& src, u32& slot) {
    if (x < 16u * NA) { src = x / (2u * NA); slot = x - src * (2u * NA); }
    else { const u32 y = x - 16u * NA; src = y / (16u - 2u * NA); slot = 2u * NA + (y - src * (16u - 2u * NA)); }
}
__device__ __forceinline__ void pusher(const Params& p, LAS unsigned char* lds, int wvi, int gw, int lane, u32 callv) {
    const u32 pool = wvi & 1;
    const __amdgpu_buffer_rsrc_t rt = __builtin_amdgcn_make_buffer_rsrc(p.ws[p.rank] + OFF_TICK + pool * 128, 0, 4, 0x00020000);
    const u32 loffw = (u32)((lane >> 3) * SH + 4 * (lane & 7)) * 4u, loffx = (u32)((lane & 15) * KG + 8 * (lane >> 4)) * 4u;
    const u32 lbase = (u32)(uintptr_t)lds + (u32)wvi * 16384u;
    auto ld = [&](u32 buf, u32 n) {
        const Pkt k = ticket_packet(n, pool);
        const u32 l0 = lbase + buf * 8192u;
        u32 keep;
#define GL(i, off) "s_mov_b32 m0, %" #i "\n\ts_nop 0\n\tglobal_load_lds_dwordx4 " off "\n\t"
#define GL8(sb, o0, o1, o2, o3, o4, o5, o6, o7) asm volatile("s_mov_b32 %0, m0\n\t" \
                         GL(6, "%1, %5") GL(7, "%2, %5") GL(8, "%3, %5") GL(9, "%4, %5") GL(10, "%14, %5") GL(11, "%15, %5") GL(12, "%16, %5") GL(13, "%17, %5") \
                         "s_mov_b32 m0, %0" : "=&s"(keep) : "v"(o0), "v"(o1), "v"(o2), "v"(o3), "s"(sb), \
                           "s"(l0), "s"(l0 + 1024u), "s"(l0 + 2048u), "s"(l0 + 3072u), "s"(l0 + 4096u), "s"(l0 + 5120u), "s"(l0 + 6144u), "s"(l0 + 7168u), \
                           "v"(o4), "v"(o5), "v"(o6), "v"(o7) : "memory")
        if (k.isw) {
            const GAS char* const sb = (const GAS char*)p.w + ((2048u * k.kh + 64u * k.t) * SH + 256u * k.h + 32u * k.blk) * 4u;
            GL8(sb, loffw, loffw + 16384u, loffw + 32768u, loffw + 49152u, loffw + 65536u, loffw + 81920u, loffw + 98304u, loffw + 114688u);
        } else {
            const GAS char* const sb = (const GAS char*)p.x + ((256u * k.h + 32u * k.blk) * KG + 2048u * k.kh + 64u * k.t) * 4u;
            constexpr u32 R16 = 16u * KG * 4u;
            GL8(sb, loffx, loffx + 16u, loffx + 128u, loffx + 144u, loffx + R16, loffx + R16 + 16u, loffx + R16 + 128u, loffx + R16 + 144u);
        }
#undef GL8
#undef GL
    };
    u32 ta = gw, tb = gw + PWAVES, tc = gw + 2 * PWAVES;
    ld(0, ta); ld(1, tb);
    asm volatile("s_waitcnt vmcnt(8)" ::: "memory");
    const u32 call = rfl(callv) + 1u, seq16 = call & 0xFFFFu;
    const u32 tbase = (call - 1u) * 1024u - 3u * PWAVES;
    const u32 wcol = (16u * ((lane >> 2) & 1u) + 4u * ((lane & 31) >> 3) + (lane & 3u)) * 4u + (lane >> 5) * 1024u;
    u32x4 ph = (u32x4){0, 0, 0, 0}; char* pa = nullptr; bool have = false; u32 drawn = 0;
    u32 buf = 0;
#pragma unroll 1
    while (ta < 1024u) {
        const u32 n = ta;
        const Pkt k = ticket_packet(n, pool);
        LAS unsigned char* const lb = lds + wvi * 16384 + buf * 8192;
        asm volatile("s_waitcnt vmcnt(14)" ::: "memory");
        u32 d[16]; float mx[2] = {0.f, 0.f};
        if (k.isw) {
#pragma unroll
            for (int g = 0; g < 4; ++g)
#pragma unroll
                for (int q = 0; q < 4; ++q) {
                    const float lo = *(const LAS float*)(lb + wcol + (16 * g + 2 * q) * 128), hi = *(const LAS float*)(lb + wcol + (16 * g + 2 * q + 1) * 128);
                    d[g * 4 + q] = cvt_pk_bf16(lo, hi); mx[g >> 1] = fmaxf(mx[g >> 1], fmaxf(fabsf(lo), fabsf(hi)));
                }
        } else {
#pragma unroll
            for (int li = 0; li < 8; ++li) {
                const f32x4 v = *(const LAS f32x4*)(lb + li * 1024 + lane * 16);
                d[2 * li] = cvt_pk_bf16(v[0], v[1]); d[2 * li + 1] = cvt_pk_bf16(v[2], v[3]);
                mx[li >> 2] = fmaxf(fmaxf(mx[li >> 2], fmaxf(fabsf(v[0]), fabsf(v[1]))), fmaxf(fabsf(v[2]), fabsf(v[3])));
            }
        }
        u32 d0 = k.isw ? ((k.h << 1) | k.kh) : ((k.h << 2) | k.kh), d1 = d0 | (k.isw ? 4u : 2u);
        if ((n ^ (n >> 5)) & 1u) { const u32 x = d0; d0 = d1; d1 = x; }
        char* const w0 = p.ws[d0 & 7u]; char* const w1 = p.ws[d1 & 7u];
        const u32 po = OFF_IN + (u32)p.rank * IN_SRC + (k.isw ? IN_B : 0u) + k.t * IN_T + k.blk * PKT + lane * 16u;
        u32x4 hd[2], L[2]; u32x2 N[2];
        pack_unit(d, mx[0], seq16, L[0], N[0], hd[0]);
        pack_unit(d + 8, mx[1], seq16, L[1], N[1], hd[1]);
        const u32x4 NN = (u32x4){N[0].x, N[0].y, N[1].x, N[1].y};
        asm volatile("" :: "v"(NN), "v"(L[0]), "v"(L[1]));
        drain();
        const u32 dv = rfl(drawn);
        if (have) { tc = dv - tbase; if (lane < 4) st16_sys(pa, ph); }
        drawn = (u32)__builtin_amdgcn_raw_ptr_buffer_atomic_add_i32(1, rt, lane == 0 ? 0 : 0x7FFFFFF0, 0, 0);
#pragma unroll
        for (int u = 0; u < 2; ++u)
            if (((hd[u].x >> 23) & 7u) == 7u) {
                const u32 ov = OFF_OVF + ((u32)p.rank * 2u + k.isw) * (512u << 10) + k.t * 16384u + (2u * k.blk + u) * 1024u + lane * 16u;
                const u32x4 r1 = (u32x4){d[8 * u + 4], d[8 * u + 5], d[8 * u + 6], d[8 * u + 7]};
                L[u] = (u32x4){d[8 * u], d[8 * u + 1], d[8 * u + 2], d[8 * u + 3]};
                st16_sys(w0 + ov, r1); st16_sys(w1 + ov, r1);
            }
        st16_sys(w0 + po, L[0]); st16_sys(w1 + po, L[0]); st16_sys(w0 + po + 1024, L[1]); st16_sys(w1 + po + 1024, L[1]); st16_sys(w0 + po + 2048, NN); st16_sys(w1 + po + 2048, NN);
        asm volatile("s_waitcnt lgkmcnt(0)" ::: "memory");
        ld(buf, tc); ta = tb; tb = tc; buf ^= 1u;
        const u32 hi = k.isw ? (u32)p.rank * 16u + 2u * k.blk : mt_index((u32)p.rank, 2u * k.blk);
        pa = ((lane & 2) ? w1 : w0) + (k.isw ? OFF_HB : OFF_HA) + k.t * 2048u + (hi + (lane & 1)) * 16u;
        ph = (lane & 1) ? hd[1] : hd[0]; have = true;
    }
    drain();
    asm volatile("" :: "v"(drawn));
    if (lane < 4) st16_sys(pa, ph);
}

__device__ __forceinline__ void decoder(const Params& p, LAS unsigned char* lds, int tm, int dw, int lane, Wait& wt, u32 seq16) {
    char* const ws = p.ws[p.rank];
    LAS u32* const cnt = (LAS u32*)(lds + LDS_MISC);
    const __amdgpu_buffer_rsrc_t rs = __builtin_amdgcn_make_buffer_rsrc(ws, 0, (int)WS_NEED, 0x00020000);
    const u32 fr = lane & 15, fq = lane >> 4;
    auto uaddr = [&](u32 v, u32 g, u32 efr, u32 efq) -> u32 { const u32 row = 16u * v + efr; return row * 128u + (((4u * g + efq) ^ (row & 7u) ^ ((row >> 3) & 3u)) << 4); };
    auto mtile = [&](u32 v) -> u32 { return min(10u * tm + v, 127u); };
    u32 po[5];
#pragma unroll
    for (int s = 0; s < 5; ++s) { u32 sr, sl; mt_split(mtile(2 * s), sr, sl); po[s] = OFF_IN + sr * IN_SRC + (sl >> 1) * PKT; }
    const u32 ae0 = uaddr(0, 0, fr, fq), ae1 = uaddr(0, 1, fr, fq), ao0 = uaddr(1, 0, fr, fq) - 2048u, ao1 = uaddr(1, 1, fr, fq) - 2048u;
    const u32 hv = min((u32)lane >> 2, 9u);
    const u32 hoff = OFF_HA + mtile(hv) * 16u;
    struct PayA { u32x4 l0[5], l1[5], n[5]; };
    auto ldh = [&](int t) -> u32x4 { return __builtin_amdgcn_raw_buffer_load_b128(rs, hoff, t * 2048, 17); };
    auto ldp = [&](PayA& y, int t) {
        const u32 so = t * IN_T;
#pragma unroll
        for (int s = 0; s < 5; ++s) {
            y.l0[s] = __builtin_amdgcn_raw_buffer_load_b128(rs, lane * 16, po[s] + so, 17);
            y.l1[s] = __builtin_amdgcn_raw_buffer_load_b128(rs, lane * 16 + 1024, po[s] + so, 17);
            y.n[s] = __builtin_amdgcn_raw_buffer_load_b128(rs, lane * 16 + 2048, po[s] + so, 17);
        }
    };
    auto freec = [&](int t) -> u32 { return __hip_atomic_load(cnt + 24 + (u32)t % NSTAGE, __ATOMIC_RELAXED, __HIP_MEMORY_SCOPE_WORKGROUP); };
    auto dec = [&](const PayA& y, const u32x4& h, int t, u32 nfree) {
        const u32 sg = (u32)t % NSTAGE, turn = (u32)t / NSTAGE;
        LAS unsigned char* st = lds + sg * STAGE;
        if (rfl(nfree) < 4u * turn) lds_wait(cnt + 24 + sg, 4u * turn);
        const u32 mode = (h.x >> 23) & 7u, brl = ((h.x >> 16) & 0x7Fu) * 0x01010101u;
        if (__ballot(mode == 7u) == 0ull) {
#pragma unroll
            for (int s = 0; s < 5; ++s) {
                const u32 b0 = rdl(brl, 8 * s), b1 = rdl(brl, 8 * s + 4);
                *(LAS u32x4*)(st + ae0 + s * 4096) = decode_group(y.l0[s].x, y.l0[s].y, y.n[s].x, b0); *(LAS u32x4*)(st + ae1 + s * 4096) = decode_group(y.l0[s].z, y.l0[s].w, y.n[s].y, b0);
                *(LAS u32x4*)(st + ao0 + s * 4096 + 2048) = decode_group(y.l1[s].x, y.l1[s].y, y.n[s].z, b1); *(LAS u32x4*)(st + ao1 + s * 4096 + 2048) = decode_group(y.l1[s].z, y.l1[s].w, y.n[s].w, b1);
            }
        } else {
#pragma unroll
            for (int v = 0; v < 10; ++v) {
                u32 sr, sl; mt_split(mtile(v), sr, sl);
                const u32 br = rdl(brl, 4 * v);
                const u32x4 l = (v & 1) ? y.l1[v >> 1] : y.l0[v >> 1]; const u32 n0 = (v & 1) ? y.n[v >> 1].z : y.n[v >> 1].x, n1 = (v & 1) ? y.n[v >> 1].w : y.n[v >> 1].y;
                const u32x4 rw = ld16_sync(ws, OFF_OVF + sr * (1024u << 10) + sl * 1024u + t * 16384 + lane * 16);
                const bool raw = rdl(mode, 4 * v) == 7u;
                *(LAS u32x4*)(st + ((v & 1) ? ao0 : ae0) + v * 2048) = raw ? l : decode_group(l.x, l.y, n0, br);
                *(LAS u32x4*)(st + ((v & 1) ? ao1 : ae1) + v * 2048) = raw ? rw : decode_group(l.z, l.w, n1, br);
            }
        }
        if (__ballot(mode - 1u < 4u) != 0ull) {
            const u32 c = lane & 3;
            const u64 a = (u64)h.x | ((u64)h.y << 32), b = (u64)h.z | ((u64)h.w << 32);
            const u32 e = (u32)((c < 2 ? a : b) >> ((c < 2 ? 26u : 16u) + 19u * (c & 1u))) & 0x7FFFFu;
            const u32 l = (e >> 4) & 63u;
            const u32 adr = uaddr(hv, (e >> 3) & 1u, l & 15u, l >> 4) + 2u * (e & 7u) + 1u;
            if (lane < 40 && mode != 7u && c < mode) st[adr] = (unsigned char)(e >> 11);
        }
        lds_signal(cnt + 16 + sg);
    };
    PayA ya, yb; u32x4 ha, hb, sn;
    ha = ldh(dw);
    wait_hdr(ha, ws, hoff + dw * 2048, seq16, wt);
    sn = ldh(dw + 4); ldp(ya, dw);
#define DEC_INST(u, hc, hn, yc, yn) do { \
        const int t4 = min((u) + 4, NT - 1), t8 = min((u) + 8, NT - 1); \
        const u32 nf = freec(u); \
        if (hdr_ok(sn, seq16)) { hn = sn; sn = ldh(t8); ldp(yn, t4); dec(yc, hc, (u), nf); asm volatile("; early" ::: "memory"); } \
        else { dec(yc, hc, (u), nf); wait_hdr(sn, ws, hoff + t4 * 2048, seq16, wt); hn = sn; sn = ldh(t8); ldp(yn, t4); asm volatile("; late" ::: "memory"); } } while (0)
#pragma unroll 1
    for (int u = dw; u < NT; u += 8) {
        DEC_INST(u, ha, hb, ya, yb);
        DEC_INST(u + 4, hb, ha, yb, ya);
    }
#undef DEC_INST
}

__device__ __forceinline__ void mma_wave(const Params& p, LAS unsigned char* lds, int tm, int tn, int wv, int lane, Wait& wt, u32 seq16) {
    char* const ws = p.ws[p.rank];
    const __amdgpu_buffer_rsrc_t rs = __builtin_amdgcn_make_buffer_rsrc(ws, 0, (int)WS_NEED, 0x00020000);
    const u32 r = lane & 31, hh = lane >> 5, fl = (r & 7u) ^ ((r >> 3) & 3u);
    u32 ak[4];
#pragma unroll
    for (int k = 0; k < 4; ++k) ak[k] = r * 128u + (((2u * k + hh) ^ fl) << 4);
    f32x16 acc[5];
#pragma unroll
    for (int m = 0; m < 5; ++m)
#pragma unroll
        for (int e = 0; e < 16; ++e) acc[m][e] = 0.f;
    LAS u32* const cnt = (LAS u32*)(lds + LDS_MISC);
    const u32 src = tn >> 1, slab = 4 * (tn & 1) + wv;
    const u32 pB = OFF_IN + src * IN_SRC + IN_B + slab * PKT, oB = OFF_OVF + src * (1024u << 10) + (512u << 10) + 2u * slab * 1024u + lane * 16u;
    const u32 hoff = OFF_HB + (16u * src + 2u * slab + min((u32)lane >> 2, 1u)) * 16u;
    struct PayB { u32x4 l0, l1, n; };
    auto ldh = [&](int t) -> u32x4 { return __builtin_amdgcn_raw_buffer_load_b128(rs, hoff, t * 2048, 17); };
    auto ldp = [&](PayB& y, int t) {
        y.l0 = __builtin_amdgcn_raw_buffer_load_b128(rs, lane * 16, pB + t * IN_T, 17);
        y.l1 = __builtin_amdgcn_raw_buffer_load_b128(rs, lane * 16 + 1024, pB + t * IN_T, 17);
        y.n = __builtin_amdgcn_raw_buffer_load_b128(rs, lane * 16 + 2048, pB + t * IN_T, 17);
    };
    u32 nfull = 0;
    auto ungrp = [&](u32 (&d)[16], const PayB& y, int g, u32 b0, u32 b1) {
        const u32x4 r = g == 0 ? decode_group(y.l0.x, y.l0.y, y.n.x, b0) : g == 1 ? decode_group(y.l0.z, y.l0.w, y.n.y, b0) : g == 2 ? decode_group(y.l1.x, y.l1.y, y.n.z, b1) : decode_group(y.l1.z, y.l1.w, y.n.w, b1);
        d[4 * g] = r.x; d[4 * g + 1] = r.y; d[4 * g + 2] = r.z; d[4 * g + 3] = r.w;
    };
    auto fix = [&](u32 (&d)[16], const PayB& y, const u32x4& h, u32 m0, u32 m1, int u) {
        if (m0 == 7u) { const u32x4 rw = ld16_sync(ws, oB + u * 16384); d[0] = y.l0.x; d[1] = y.l0.y; d[2] = y.l0.z; d[3] = y.l0.w; d[4] = rw.x; d[5] = rw.y; d[6] = rw.z; d[7] = rw.w; }
        else if (m0) patch_regs(d, h, 0, m0, lane);
        if (m1 == 7u) { const u32x4 rw = ld16_sync(ws, oB + u * 16384 + 1024); d[8] = y.l1.x; d[9] = y.l1.y; d[10] = y.l1.z; d[11] = y.l1.w; d[12] = rw.x; d[13] = rw.y; d[14] = rw.z; d[15] = rw.w; }
        else if (m1) patch_regs(d + 8, h, 4, m1, lane);
    };
    bf16x8 a0[5], a1[5];
#define SB() __builtin_amdgcn_sched_barrier(0)
#define RD(dst, st, k) do { _Pragma("unroll") for (int m = 0; m < 5; ++m) dst[m] = *(const LAS bf16x8*)((st) + ak[k] + m * 4096); SB(); } while (0)
#define M1(d, k, src, m) do { acc[m] = __builtin_amdgcn_mfma_f32_32x32x16_bf16(__builtin_bit_cast(bf16x8, (u32x4){d[4 * k], d[4 * k + 1], d[4 * k + 2], d[4 * k + 3]}), src[m], acc[m], 0, 0, 0); SB(); } while (0)
#define UH(g, pp, b0, b1) do { const u32 n_ = g == 0 ? pp.n.x : g == 1 ? pp.n.y : g == 2 ? pp.n.z : pp.n.w, br_ = g < 2 ? b0 : b1; \
                              uha = (n_ & 0x87878787u) + br_; uhb = (__builtin_amdgcn_alignbit(n_, n_, 4) & 0x87878787u) + br_; asm volatile("" : "+v"(uha), "+v"(uhb)); SB(); } while (0)
#define UP(d, g, pp) do { const u32 la_ = g == 0 ? pp.l0.x : g == 1 ? pp.l0.z : g == 2 ? pp.l1.x : pp.l1.z, lb_ = g == 0 ? pp.l0.y : g == 1 ? pp.l0.w : g == 2 ? pp.l1.y : pp.l1.w; \
                         d[4 * g] = __builtin_amdgcn_perm(uha, la_, 0x05010400u); d[4 * g + 1] = __builtin_amdgcn_perm(uha, la_, 0x07030602u); \
                         d[4 * g + 2] = __builtin_amdgcn_perm(uhb, lb_, 0x05010400u); d[4 * g + 3] = __builtin_amdgcn_perm(uhb, lb_, 0x07030602u); \
                         asm volatile("" : "+v"(d[4 * g]), "+v"(d[4 * g + 1]), "+v"(d[4 * g + 2]), "+v"(d[4 * g + 3])); SB(); } while (0)
    u32 uha, uhb;
    PayB y0, y1, y2, y3; u32x4 h0, h1, h2, h3, s0, s1; u32 da[16], db[16];
    h0 = ldh(0); h1 = ldh(1); h2 = ldh(2); h3 = ldh(3);
    wait_hdr(h0, ws, hoff, seq16, wt); ldp(y0, 0);
    wait_hdr(h1, ws, hoff + 2048, seq16, wt); ldp(y1, 1);
    wait_hdr(h2, ws, hoff + 4096, seq16, wt); s0 = ldh(4); ldp(y2, 2);
    wait_hdr(h3, ws, hoff + 6144, seq16, wt); s1 = ldh(5); ldp(y3, 3);
    {
        const u32 w0 = rdl(h0.x, 0), w1 = rdl(h0.x, 4), b0 = ((w0 >> 16) & 0x7Fu) * 0x01010101u, b1 = ((w1 >> 16) & 0x7Fu) * 0x01010101u;
        ungrp(da, y0, 0, b0, b1); ungrp(da, y0, 1, b0, b1); ungrp(da, y0, 2, b0, b1); ungrp(da, y0, 3, b0, b1);
        if (((w0 | w1) >> 23) & 7u) fix(da, y0, h0, (w0 >> 23) & 7u, (w1 >> 23) & 7u, 0);
    }
    wait_hdr(s0, ws, hoff + 4 * 2048, seq16, wt);
    lds_wait(cnt + 16, 1u);
    RD(a0, lds, 0);
    u32 sg = 0, turn = 0;
#define R1(dst, st, k, m) do { dst[m] = *(const LAS bf16x8*)((st) + ak[k] + m * 4096); SB(); } while (0)
#define MMA_STEP(u, dc, dn, y1_, h1_, y0_, h0_, sn, so) do { \
        const int t4 = min((u) + 4, NT - 1), t5 = min((u) + 5, NT - 1), t6 = min((u) + 6, NT - 1), t1 = min((u) + 1, NT - 1); \
        const bool lastu = (u) + 1 >= NT, wrap = sg == NSTAGE - 1; \
        const u32 sg1 = lastu ? sg : wrap ? 0u : sg + 1u, turn1 = (!lastu && wrap) ? turn + 1u : turn; \
        LAS unsigned char* st = lds + sg * STAGE; \
        const u32 so4 = pB + t4 * IN_T; \
        h0_ = sn; SB(); \
        M1(dc, 0, a0, 0); R1(a1, st, 1, 0); sn = ldh(t6); SB(); \
        M1(dc, 0, a0, 1); R1(a1, st, 1, 1); y0_.l0 = __builtin_amdgcn_raw_buffer_load_b128(rs, lane * 16, so4, 17); SB(); \
        M1(dc, 0, a0, 2); R1(a1, st, 1, 2); y0_.l1 = __builtin_amdgcn_raw_buffer_load_b128(rs, lane * 16 + 1024, so4, 17); SB(); \
        M1(dc, 0, a0, 3); R1(a1, st, 1, 3); y0_.n = __builtin_amdgcn_raw_buffer_load_b128(rs, lane * 16 + 2048, so4, 17); SB(); \
        M1(dc, 0, a0, 4); R1(a1, st, 1, 4); \
        const u32 w0 = rdl(h1_.x, 0), w1 = rdl(h1_.x, 4); \
        const u32 b0 = ((w0 >> 16) & 0x7Fu) * 0x01010101u, b1 = ((w1 >> 16) & 0x7Fu) * 0x01010101u; SB(); \
        M1(dc, 1, a1, 0); R1(a0, st, 2, 0); UH(0, y1_, b0, b1); M1(dc, 1, a1, 1); R1(a0, st, 2, 1); UP(dn, 0, y1_); M1(dc, 1, a1, 2); R1(a0, st, 2, 2); UH(1, y1_, b0, b1); \
        M1(dc, 1, a1, 3); R1(a0, st, 2, 3); UP(dn, 1, y1_); M1(dc, 1, a1, 4); R1(a0, st, 2, 4); \
        nfull = __hip_atomic_load(cnt + 16 + sg1, __ATOMIC_RELAXED, __HIP_MEMORY_SCOPE_WORKGROUP); SB(); \
        M1(dc, 2, a0, 0); R1(a1, st, 3, 0); UH(2, y1_, b0, b1); M1(dc, 2, a0, 1); R1(a1, st, 3, 1); UP(dn, 2, y1_); M1(dc, 2, a0, 2); R1(a1, st, 3, 2); UH(3, y1_, b0, b1); \
        M1(dc, 2, a0, 3); R1(a1, st, 3, 3); UP(dn, 3, y1_); M1(dc, 2, a0, 4); R1(a1, st, 3, 4); \
        lds_signal(cnt + 24 + sg); SB(); \
          \
        M1(dc, 3, a1, 0); \
        if (rfl(nfull) < turn1 + 1u) lds_wait(cnt + 16 + sg1, turn1 + 1u); \
        SB(); \
        LAS unsigned char* st1 = lds + sg1 * STAGE; \
        M1(dc, 3, a1, 1); R1(a0, st1, 0, 0); R1(a0, st1, 0, 1); \
        M1(dc, 3, a1, 2); R1(a0, st1, 0, 2); R1(a0, st1, 0, 3); \
        wait_hdr(so, ws, hoff + t5 * 2048, seq16, wt); \
        SB(); \
        M1(dc, 3, a1, 3); R1(a0, st1, 0, 4); \
        if (((w0 | w1) >> 23) & 7u) fix(dn, y1_, h1_, (w0 >> 23) & 7u, (w1 >> 23) & 7u, t1); \
        SB(); \
        M1(dc, 3, a1, 4); \
        sg = sg1; turn = turn1; } while (0)
#pragma unroll 1
    for (int u = 0; u < NT; u += 4) {
        MMA_STEP(u, da, db, y1, h1, y0, h0, s0, s1);
        MMA_STEP(u + 1, db, da, y2, h2, y1, h1, s1, s0);
        MMA_STEP(u + 2, da, db, y3, h3, y2, h2, s0, s1);
        MMA_STEP(u + 3, db, da, y0, h0, y3, h3, s1, s0);
    }
#undef UH
#undef R1
#undef UP
#undef M1
#undef SB
#undef MMA_STEP
#undef RD
    char* const dst = p.ws[src & 7];
    const u32 r32 = 5 * tm;
    const u32 ubase = OFF_Y + (u32)p.rank * Y_SRC + (slab * 64 + r32) * 2048u;
    u32x4 myhdr = (u32x4){0, 0, 0, 0};
#pragma unroll
    for (int m = 0; m < 5; ++m) {
        if (r32 + m < 64) {
            u32 d[8]; float mx = 0.f;
#pragma unroll
            for (int q = 0; q < 8; ++q) {
                d[q] = cvt_pk_bf16(acc[m][2 * q], acc[m][2 * q + 1]);
                mx = fmaxf(mx, fmaxf(fabsf(acc[m][2 * q]), fabsf(acc[m][2 * q + 1])));
            }
            u32x4 L, hdr; u32x2 N;
            pack_unit(d, mx, seq16, L, N, hdr);
            char* q = dst + ubase + m * 2048u;
            if (((hdr.x >> 23) & 7u) != 7u) { st16_sys(q + lane * 16, L); st8_sys(q + 1024 + lane * 8, N); }
            else { st16_sys(q + lane * 16, (u32x4){d[0], d[1], d[2], d[3]}); st16_sys(q + 1024 + lane * 16, (u32x4){d[4], d[5], d[6], d[7]}); }
            if (lane == m) myhdr = hdr;
        }
    }
    drain();
    if (lane < 5 && r32 + lane < 64) st16_sys(dst + OFF_HY + (((u32)p.rank * 8 + slab) * 64 + r32 + lane) * 16u, myhdr);
}

__global__ void __launch_bounds__(NTHR, 2) k_main(Params p) {
    extern __shared__ __attribute__((aligned(16))) unsigned char smem[];
    LAS unsigned char* lds = (LAS unsigned char*)smem;
    const int b = blockIdx.x, wv = rfl(threadIdx.x >> 6), lane = threadIdx.x & 63;
    char* const ws = p.ws[p.rank];
    GAS u32* cntp = (GAS u32*)(ws + OFF_CNT + b * 128);
    const u32 callv = __hip_atomic_load(cntp, __ATOMIC_RELAXED, __HIP_MEMORY_SCOPE_AGENT);
    const int xg = b & 7, slot = b >> 3;
    if (slot < NPS && wv < 2 * PW) {
        const int gw = (wv >> 1) * (8 * NPS) + slot * 8 + xg;
        pusher(p, lds, wv, gw, lane, callv);
    }
    const u32 call = rfl(callv) + 1u;
    const u32 seq16 = call & 0xFFFFu;
    Wait wt{(GAS u32*)(ws + OFF_TMO), call, false};
    if (slot >= 6) {
        const int s = slot - 6, tm = s >> 1, tn = 2 * xg + (s & 1);
        if (threadIdx.x < 16) ((LAS u32*)(lds + LDS_MISC))[16 + threadIdx.x] = 0u;
        BARRIER();
        if (wv >= 4) decoder(p, lds, tm, wv - 4, lane, wt, seq16); else mma_wave(p, lds, tm, tn, wv, lane, wt, seq16);
    }
    const __amdgpu_buffer_rsrc_t rs = __builtin_amdgcn_make_buffer_rsrc(ws, 0, (int)WS_NEED, 0x00020000);
    const u32 ri = b >> 7, rj = (b >> 6) & 1, r32 = b & 63, cg8 = wv;
    const u32 snd0 = (ri << 2) | (rj << 1);
    float y[16];
    LAS u32* misc = (LAS u32*)(lds + LDS_MISC);
    {
        const u32 hoff = OFF_HY + (((snd0 + min((u32)lane >> 2, 1u)) * 8 + cg8) * 64 + r32) * 16u;
        u32x4 h = __builtin_amdgcn_raw_buffer_load_b128(rs, hoff, 0, 17);
        if (!hdr_ok(h, seq16)) { const u64 t0 = rtime(); u32 n = 0; do { if (spin_on(wt, t0, n)) break; h = ld16_sync(ws, hoff); } while (!hdr_ok(h, seq16)); }
        u32x4 l[2]; u32x2 n[2];
#pragma unroll
        for (int s = 0; s < 2; ++s) {
            const u32 ub = OFF_Y + (snd0 + s) * Y_SRC + (cg8 * 64 + r32) * 2048u;
            l[s] = __builtin_amdgcn_raw_buffer_load_b128(rs, lane * 16, ub, 17);
            n[s] = __builtin_amdgcn_raw_buffer_load_b64(rs, lane * 8 + 1024, ub, 17);
        }
#pragma unroll
        for (int s = 0; s < 2; ++s) {
            const u32 w0 = rdl(h.x, 4 * s), mode = (w0 >> 23) & 7u;
            u32 d[8];
            if (mode != 7u) {
                const u32 brep = ((w0 >> 16) & 0x7Fu) * 0x01010101u;
                const u32x4 g0 = decode_group(l[s].x, l[s].y, n[s].x, brep), g1 = decode_group(l[s].z, l[s].w, n[s].y, brep);
                d[0] = g0.x; d[1] = g0.y; d[2] = g0.z; d[3] = g0.w; d[4] = g1.x; d[5] = g1.y; d[6] = g1.z; d[7] = g1.w;
                if (mode) patch_regs(d, h, 4 * s, mode, lane);
            } else {
                const u32 ub = OFF_Y + (snd0 + s) * Y_SRC + (cg8 * 64 + r32) * 2048u;
                const u32x4 hi = ld16_sync(ws, ub + lane * 16 + 1024);
                d[0] = l[s].x; d[1] = l[s].y; d[2] = l[s].z; d[3] = l[s].w; d[4] = hi.x; d[5] = hi.y; d[6] = hi.z; d[7] = hi.w;
            }
#pragma unroll
            for (int q = 0; q < 8; ++q) {
                const float lo = __uint_as_float(d[q] << 16), hi = __uint_as_float(d[q] & 0xFFFF0000u);
                if (s == 0) { y[2 * q] = lo; y[2 * q + 1] = hi; } else { y[2 * q] += lo; y[2 * q + 1] += hi; }
            }
        }
        float mx = 0.f;
#pragma unroll
        for (int q = 0; q < 16; ++q) mx = fmaxf(mx, fabsf(y[q]));
        const u32 wm = wave_max_u32(__float_as_uint(mx));
        if (lane == 0) misc[wv] = wm;
    }
    BARRIER();
    if (wv == 0) {
        const u32 v = wave_max_u32(misc[lane & 7]);
        if (lane == 0) __hip_atomic_store((GAS u64*)(ws + OFF_LMAX + b * 64), ((u64)call << 32) | v, __ATOMIC_RELAXED, __HIP_MEMORY_SCOPE_AGENT);
        const u32 go = OFF_GMAX + ((b & 7) * 8 + (lane & 7)) * 64;
        u32x2 g = ld8_sync(ws, go);
        if (__ballot((int)(g.y - call) >= 0) != ~0ull) { const u64 t0 = rtime(); u32 n = 0; do { if (spin_on<0>(wt, t0, n)) break; g = ld8_sync(ws, go); } while (__ballot((int)(g.y - call) >= 0) != ~0ull); }
        const u32 gm = wave_max_u32(g.x);
        if (lane == 0) misc[8] = gm;
    } else if (wv == 1 && b == LEADER) {
        u32x2 g0, g1, g2, g3;
        auto poll = [&]() {
            asm volatile("global_load_dwordx2 %0, %4, %5 sc1\n\tglobal_load_dwordx2 %1, %4, %5 offset:64 sc1\n\tglobal_load_dwordx2 %2, %4, %5 offset:128 sc1\n\tglobal_load_dwordx2 %3, %4, %5 offset:192 sc1\n\ts_waitcnt vmcnt(0)"
                         : "=&v"(g0), "=&v"(g1), "=&v"(g2), "=&v"(g3) : "v"(OFF_LMAX + lane * 256), "s"(ws) : "memory");
            return __ballot((int)((g0.y - call) | (g1.y - call) | (g2.y - call) | (g3.y - call)) >= 0) == ~0ull; };
        if (!poll()) { const u64 t0 = rtime(); u32 n = 0; do { if (spin_on<0>(wt, t0, n)) break; } while (!poll()); }
        const u32 lm = wave_max_u32(max(max(g0.x, g1.x), max(g2.x, g3.x)));
        __hip_atomic_store((GAS u64*)(p.ws[lane >> 3] + OFF_GMAX + ((lane & 7) * 8 + (u32)p.rank) * 64), ((u64)call << 32) | lm, __ATOMIC_RELAXED, __HIP_MEMORY_SCOPE_SYSTEM);
    }
    BARRIER();
    {
        const float amax = __uint_as_float(misc[8]);
        const bool bad = rfl(__hip_atomic_load(wt.tmo, __ATOMIC_RELAXED, __HIP_MEMORY_SCOPE_AGENT)) == call;
        const float inv = amax > 0.f ? 448.0f / amax : 0.f;
        u32 o[4];
#pragma unroll
        for (int q = 0; q < 4; ++q) {
            float v[4];
#pragma unroll
            for (int e = 0; e < 4; ++e) v[e] = __builtin_amdgcn_fmed3f(y[4 * q + e] * inv, -448.f, 448.f);
            int t = __builtin_amdgcn_cvt_pk_fp8_f32(v[0], v[1], 0, false); t = __builtin_amdgcn_cvt_pk_fp8_f32(v[2], v[3], t, true); o[q] = (u32)t;
        }
        const u32 R = r32 * 32 + (lane & 31);
        u32 sr, sl; mt_split(R >> 4, sr, sl);
        const u32 row = 512 * sr + 256 * ri + 16 * sl + (R & 15), col = 256 * rj + 32 * cg8 + 16 * (lane >> 5);
        const u32x4 ov = bad ? (u32x4){0x7f7f7f7fu, 0x7f7f7f7fu, 0x7f7f7f7fu, 0x7f7f7f7fu} : (u32x4){o[0], o[1], o[2], o[3]};
        *(GAS u32x4*)(p.out + (size_t)row * SH + col) = ov;
    }
    if (threadIdx.x == 0) __hip_atomic_store(cntp, call, __ATOMIC_RELAXED, __HIP_MEMORY_SCOPE_AGENT);
}

extern "C" void kernel_launch(int rank, int nranks, ncclComm_t comm, hipStream_t stream,
                              void* const* d_in, const int* in_sizes, int n_in,
                              void* d_out, int out_size, const int* in_dtypes, int out_dtype,
                              void* const* d_out_all, void* const* d_ws_all, size_t ws_bytes) {
    if (nranks != NR || (unsigned)rank >= (unsigned)NR || ws_bytes < WS_NEED || n_in < 2 || in_sizes[0] != SH * KG || in_sizes[1] != KG * SH || out_size != MG * SH) {
        fprintf(stderr, "kernel_launch: unexpected shapes\n"); return; }
    HIP_CHECK(hipFuncSetAttribute((const void*)k_main, hipFuncAttributeMaxDynamicSharedMemorySize, LDS_BYTES));
    Params p;
    p.x = (const float*)d_in[0]; p.w = (const float*)d_in[1]; p.out = (unsigned char*)d_out;
    for (int i = 0; i < NR; ++i) p.ws[i] = (char*)d_ws_all[i];
    p.rank = rank; p.pad = 0;
    k_main<<<256, NTHR, LDS_BYTES, stream>>>(p);
    HIP_CHECK(hipGetLastError());
}
```
